```python
import numpy as np

kernel_name = "gpu_ag_gemm_m4096_k4096_n4096_bf16_gelu"


ABSMAX_RTOL = 0.008

COMPUTE_DTYPE = "bfloat16"
EPILOGUE = "gelu"
IN_QUANT = None
OUT_QUANT = None
IN_DTYPES = {"x": "bfloat16", "w_mat": "bfloat16"}
WORLD_SIZE = 8

def _to_bf16(x):
    u = np.ascontiguousarray(x, dtype=np.float32).view(np.uint32)
    return ((u + 0x7FFF + ((u >> 16) & 1)) & 0xFFFF0000).view(np.float32)

def setup_inputs():

    x = _to_bf16(np.random.randn(4096, 4096).astype(np.float32))
    w_mat = _to_bf16((np.random.randn(4096, 4096) / np.sqrt(4096)).astype(np.float32))
    return {"x": x, "w_mat": w_mat}

def _epilogue(y):
    c = 0.7978845608028654
    return (0.5 * y * (1.0 + np.tanh(c * (y + 0.044715 * y ** 3)))).astype(np.float32)

def reference(x, w_mat):
    y = (x @ w_mat).astype(np.float32)
    return _epilogue(y)
```

```cpp
#include <hip/hip_runtime.h>
#include <rccl/rccl.h>
#include <cstdio>
#include <cstdint>
#define LOAD_AUX 17

namespace agg {
typedef unsigned u32;
typedef unsigned long long u64;
typedef u32 u32x4 __attribute__((ext_vector_type(4)));
typedef u32 u32x2 __attribute__((ext_vector_type(2)));
typedef float f32x4 __attribute__((ext_vector_type(4)));
typedef short s16x4 __attribute__((ext_vector_type(4)));
typedef short s16x8 __attribute__((ext_vector_type(8)));
typedef __bf16 bf16x8 __attribute__((ext_vector_type(8)));
#define LAS __attribute__((address_space(3)))
#define GAS __attribute__((address_space(1)))
typedef GAS u32 gu32;

constexpr int NRANK = 8, KDIM = 4096, SROWS = 512, SCOLS = 512;
constexpr int KC = 256, NKC = KDIM / KC;
#define P_A0 192
#define P_A1 144
constexpr int A0 = P_A0, A1 = P_A1, B0 = 256 - A0, B1 = 256 - A1, NA = A0 + A1;
static_assert(A0 % 16 == 0 && A1 % 16 == 0 && NA <= SROWS && B0 + B1 + NA == SROWS, "row split");
constexpr int BK = 64, NST = KDIM / BK, NBUF = 4, STAGE_B = 32768;
constexpr int LDS_CTL = NBUF * STAGE_B, LDS_BYTES = LDS_CTL + 64;
constexpr int NTHREADS = 576, NWG = 256;

constexpr size_t OFF_CNT = 0;
constexpr size_t OFF_TMO = 32768;
constexpr size_t OFF_DONE = OFF_TMO + 128;
constexpr size_t OFF_XRDY = 40960;
constexpr size_t OFF_WRDY = OFF_XRDY + 16 * NKC * 128;
constexpr size_t OFF_DBG = 131072;
constexpr size_t DBG_BYTES = 1048576 - OFF_DBG;
constexpr size_t A_BYTES = (size_t)2048 * KDIM * 2, W_BYTES = (size_t)KDIM * 512 * 2;
constexpr size_t OFF_A = 1048576;
constexpr size_t OFF_W = OFF_A + 2 * A_BYTES;
constexpr size_t WS_NEED = OFF_W + 2 * W_BYTES;
static_assert(OFF_WRDY + 4 * NKC * 128 <= OFF_DBG, "ctl");
constexpr u64 DEADLINE = 30000000ull;

struct Params {
    const uint16_t* x; const uint16_t* w;
    char* out[NRANK]; char* ws[NRANK];
    int rank;
    int ci, wi, cx, wx;
    int stag_i, stag_x;
    int mode;
    unsigned char sched[32];
};

__device__ __forceinline__ u64 rtc() { return __builtin_amdgcn_s_memrealtime(); }
__device__ __forceinline__ int uni(int v) { return __builtin_amdgcn_readfirstlane(v); }

__device__ __forceinline__ void map_unit(int unit, int u, int& rank, int& r) {
    const int lr = unit * 16, G = lr >> 10; int i = lr & 1023;
    if (G == 0) { if (i < 4 * A0) { const int j = i / A0; r = i - j * A0; rank = 4 * u + j; }
                  else { i -= 4 * A0; const int j = i / B0; r = NA + (i - j * B0); rank = 4 * (1 - u) + j; } }
    else { if (i < 4 * A1) { const int j = i / A1; r = A0 + i - j * A1; rank = 4 * u + j; }
           else { i -= 4 * A1; const int j = i / B1; r = NA + B0 + (i - j * B1); rank = 4 * (1 - u) + j; } }
}

#define STAMP(idx) do {} while (0)

__device__ __forceinline__ bool wait_ge(gu32* word, u32 want, gu32* tmo, u32 call) {
    const u64 t0 = rtc();
    for (;;) {
        const u32 v = (u32)uni((int)__hip_atomic_load(word, __ATOMIC_RELAXED, __HIP_MEMORY_SCOPE_SYSTEM));
        if ((int)(v - want) >= 0) return true;
        if ((u32)uni((int)__hip_atomic_load(tmo, __ATOMIC_RELAXED, __HIP_MEMORY_SCOPE_AGENT)) == call || rtc() - t0 > DEADLINE) {
            __hip_atomic_store(tmo, call, __ATOMIC_RELAXED, __HIP_MEMORY_SCOPE_AGENT);
            return false;
        }
        __builtin_amdgcn_s_sleep(2);
    }
}

__device__ __forceinline__ void st16_sys(char* ptr, u32x4 v) {
    asm volatile("global_store_dwordx4 %0, %1, off sc0 sc1\n\ts_nop 1" :: "v"(ptr), "v"(v) : "memory");
}
__device__ __forceinline__ void st16_sys_s(u32 voff, u32x4 v, const char* sbase) {
    asm volatile("global_store_dwordx4 %0, %1, %2 sc0 sc1\n\ts_nop 1" :: "v"(voff), "v"(v), "s"(sbase) : "memory");
}

__device__ __forceinline__ float gelu1(float y) {
    const float k1 = -2.0f * 0.7978845608028654f * 1.4426950408889634f, k2 = k1 * 0.044715f;
    const float t = y * y, e = __builtin_amdgcn_exp2f(y * __builtin_fmaf(t, k2, k1));
    return y * __builtin_amdgcn_rcpf(1.0f + e);
}
__device__ __forceinline__ u32 cvt_pk(float lo, float hi) { u32 r; asm("v_cvt_pk_bf16_f32 %0, %1, %2" : "=v"(r) : "v"(lo), "v"(hi)); return r; }

__device__ __forceinline__ void gemm_role(const Params& p, LAS char* lds, u32 call, int xcd, int slot) {
    const int g = p.rank, u = g >> 2, v = g & 3, par = call & 1;
    const int wave = uni(threadIdx.x >> 6), lane = threadIdx.x & 63;
    const int q = xcd >> 1, helper = xcd & 1, rsel = slot >> 2, ct = slot & 3;
    const int rt = (rsel >> 1) * 8 + 2 * q + (rsel & 1);
    char* const ws = p.ws[g];
    gu32* tmo = (gu32*)(ws + OFF_TMO);
    LAS u32* ctl = (LAS u32*)(lds + LDS_CTL);
#define BAR() __builtin_amdgcn_s_barrier()
    if (wave == 8) {
        int nrem = 0;
        for (int i = 0; i < 8; ++i) { int rk, r; map_unit(rt * 8 + i, u, rk, r); nrem += (rk != g); }
        gu32* xr = (gu32*)(ws + OFF_XRDY + (size_t)rt * NKC * 128);
        gu32* wr = (gu32*)(ws + OFF_WRDY + (size_t)ct * NKC * 128);
        const u32 xwant = call * (u32)nrem, wwant = call * 32u;
        bool ok = true;
        STAMP(0);
        if (!(p.mode & 2)) {
            ok = wait_ge(xr, xwant, tmo, call) && ok;
            if (helper) ok = wait_ge(wr, wwant, tmo, call) && ok;
        }
        STAMP(1);
        BAR();
        for (int s = 0; s < NST; ++s) {
            if (((s + 3) & 3) == 0 && s + 3 < NST) {
                const int kc = (s + 3) >> 2;
                if (!(p.mode & 2)) {
                    ok = wait_ge(xr + kc * 32, xwant, tmo, call) && ok;
                    if (helper) ok = wait_ge(wr + kc * 32, wwant, tmo, call) && ok;
                }
                STAMP(1 + kc);
            }
            BAR();
        }
        const u32 t = (u32)uni((int)__hip_atomic_load(tmo, __ATOMIC_RELAXED, __HIP_MEMORY_SCOPE_AGENT));
        if (lane == 0) ctl[0] = (!ok || t == call) ? 1u : 0u;
        asm volatile("s_waitcnt lgkmcnt(0)" ::: "memory");
        BAR();
        BAR();
        if (helper && lane == 0) {
            const int h = 4 * (1 - u) + ct;
            __hip_atomic_fetch_add((gu32*)(p.ws[h] + OFF_DONE), 1u, __ATOMIC_RELAXED, __HIP_MEMORY_SCOPE_SYSTEM);
        }
        STAMP(20);
        return;
    }
    const int wm = wave >> 2, wn = wave & 3;
    const char* asrc; {
        int rk, r; map_unit(rt * 8 + wave, u, rk, r);
        const char* base = (rk == g) ? (const char*)p.x + (size_t)r * 8192 : ws + OFF_A + (size_t)par * A_BYTES + (size_t)(rt * 128 + wave * 16) * 8192;
        asrc = base;
    }
    const char* bsrc = helper ? ws + OFF_W + (size_t)par * W_BYTES + ct * 256 : (const char*)p.w + ct * 256;
    u32 aoff[2], boff[2];
#pragma unroll
    for (int i = 0; i < 2; ++i) {
        const int ar = 8 * i + (lane >> 3);
        aoff[i] = (u32)ar * 8192u + (u32)(((lane & 7) ^ ((ar >> 1) & 7)) * 16);
        const int kr = 8 * wave + 4 * i + (lane >> 4);
        const int sw = ((kr & 3) << 2) | ((kr >> 2) & 3);
        boff[i] = (u32)kr * 1024u + (u32)(((lane & 15) ^ sw) * 16);
    }
    auto issue = [&](int s) {
        LAS char* st = lds + (s & 3) * STAGE_B;
        const char* a = asrc + (size_t)s * 128;
        const char* b = bsrc + (size_t)s * 65536;
#pragma unroll
        for (int i = 0; i < 2; ++i)
            __builtin_amdgcn_global_load_lds((const GAS void*)(a + aoff[i]), (LAS void*)(st + (16 * wave + 8 * i) * 128), 16, 0, LOAD_AUX);
#pragma unroll
        for (int i = 0; i < 2; ++i)
            __builtin_amdgcn_global_load_lds((const GAS void*)(b + boff[i]), (LAS void*)(st + 16384 + (8 * wave + 4 * i) * 256), 16, 0, LOAD_AUX);
    };
    const int fr = lane & 15, fg = lane >> 4;
    u32 xa[2];
#pragma unroll
    for (int ks = 0; ks < 2; ++ks) xa[ks] = (u32)((64 * wm + fr) * 128 + (((4 * ks + fg) ^ ((fr >> 1) & 7)) * 16));
    u32 wa[2][2];
    { const int tq = (lane & 15) >> 2, tp = lane & 3;
#pragma unroll
      for (int j = 0; j < 2; ++j)
#pragma unroll
        for (int t = 0; t < 2; ++t) {
            const int kr = 8 * fg + 4 * t + tq;
            const int sw = ((kr & 3) << 2) | ((kr >> 2) & 3);
            const int ch = 4 * wn + 2 * j + (tp >> 1);
            wa[j][t] = (u32)(16384 + kr * 256 + ((ch ^ sw) * 16) + 8 * (tp & 1));
        } }
    f32x4 acc[4][2];
#pragma unroll
    for (int i = 0; i < 4; ++i)
#pragma unroll
        for (int j = 0; j < 2; ++j) acc[i][j] = (f32x4){0.f, 0.f, 0.f, 0.f};
    const u32 ldsb = (u32)(size_t)lds;
    auto compute = [&](int s) {
        const u32 sb = ldsb + (u32)(s & 3) * STAGE_B;
        const u32 w00 = wa[0][0] + sb, w01 = wa[0][1] + sb, w10 = wa[1][0] + sb, w11 = wa[1][1] + sb, x0 = xa[0] + sb, x1 = xa[1] + sb;
        s16x4 a00, a01, a10, a11, b00, b01, b10, b11; bf16x8 xf0[4], xf1[4];
        asm volatile("ds_read_b64_tr_b16 %0, %8\n\tds_read_b64_tr_b16 %1, %9\n\tds_read_b64_tr_b16 %2, %10\n\tds_read_b64_tr_b16 %3, %11\n\t"
                     "ds_read_b128 %4, %12\n\tds_read_b128 %5, %12 offset:2048\n\tds_read_b128 %6, %12 offset:4096\n\tds_read_b128 %7, %12 offset:6144"
                     : "=&v"(a00), "=&v"(a01), "=&v"(a10), "=&v"(a11), "=&v"(xf0[0]), "=&v"(xf0[1]), "=&v"(xf0[2]), "=&v"(xf0[3])
                     : "v"(w00), "v"(w01), "v"(w10), "v"(w11), "v"(x0) : "memory");
        asm volatile("ds_read_b64_tr_b16 %0, %8 offset:8192\n\tds_read_b64_tr_b16 %1, %9 offset:8192\n\tds_read_b64_tr_b16 %2, %10 offset:8192\n\tds_read_b64_tr_b16 %3, %11 offset:8192\n\t"
                     "ds_read_b128 %4, %12\n\tds_read_b128 %5, %12 offset:2048\n\tds_read_b128 %6, %12 offset:4096\n\tds_read_b128 %7, %12 offset:6144"
                     : "=&v"(b00), "=&v"(b01), "=&v"(b10), "=&v"(b11), "=&v"(xf1[0]), "=&v"(xf1[1]), "=&v"(xf1[2]), "=&v"(xf1[3])
                     : "v"(w00), "v"(w01), "v"(w10), "v"(w11), "v"(x1) : "memory");
        asm volatile("s_waitcnt lgkmcnt(8)" : "+v"(a00), "+v"(a01), "+v"(a10), "+v"(a11), "+v"(xf0[0]), "+v"(xf0[1]), "+v"(xf0[2]), "+v"(xf0[3]) :: "memory");
        const bf16x8 wf00 = __builtin_bit_cast(bf16x8, __builtin_shufflevector(a00, a01, 0, 1, 2, 3, 4, 5, 6, 7));
        const bf16x8 wf01 = __builtin_bit_cast(bf16x8, __builtin_shufflevector(a10, a11, 0, 1, 2, 3, 4, 5, 6, 7));
        __builtin_amdgcn_s_setprio(1);
#pragma unroll
        for (int i = 0; i < 4; ++i) {
            acc[i][0] = __builtin_amdgcn_mfma_f32_16x16x32_bf16(wf00, xf0[i], acc[i][0], 0, 0, 0);
            acc[i][1] = __builtin_amdgcn_mfma_f32_16x16x32_bf16(wf01, xf0[i], acc[i][1], 0, 0, 0);
        }
        __builtin_amdgcn_s_setprio(0);
        __builtin_amdgcn_sched_barrier(0);
        asm volatile("s_waitcnt lgkmcnt(0)" : "+v"(b00), "+v"(b01), "+v"(b10), "+v"(b11), "+v"(xf1[0]), "+v"(xf1[1]), "+v"(xf1[2]), "+v"(xf1[3]) :: "memory");
        const bf16x8 wf10 = __builtin_bit_cast(bf16x8, __builtin_shufflevector(b00, b01, 0, 1, 2, 3, 4, 5, 6, 7));
        const bf16x8 wf11 = __builtin_bit_cast(bf16x8, __builtin_shufflevector(b10, b11, 0, 1, 2, 3, 4, 5, 6, 7));
        __builtin_amdgcn_s_setprio(1);
#pragma unroll
        for (int i = 0; i < 4; ++i) {
            acc[i][0] = __builtin_amdgcn_mfma_f32_16x16x32_bf16(wf10, xf1[i], acc[i][0], 0, 0, 0);
            acc[i][1] = __builtin_amdgcn_mfma_f32_16x16x32_bf16(wf11, xf1[i], acc[i][1], 0, 0, 0);
        }
        __builtin_amdgcn_s_setprio(0);
    };
#define WAITV(n) asm volatile("s_waitcnt vmcnt(" #n ")" ::: "memory")
#define WAITL0() asm volatile("s_waitcnt lgkmcnt(0)" ::: "memory")
    BAR();
    STAMP(0);
    issue(0); issue(1); issue(2);
    for (int s = 0; s < NST - 3; ++s) {
        WAITV(8); WAITL0(); BAR();
        issue(s + 3);
        compute(s);
    }
    WAITV(8); WAITL0(); BAR(); compute(NST - 3);
    WAITV(4); WAITL0(); BAR(); compute(NST - 2);
    WAITV(0); WAITL0(); BAR(); compute(NST - 1);
    STAMP(1);
    BAR();
    const bool bad = ctl[0] != 0u;
    char* obase = helper ? p.out[4 * (1 - u) + ct] + v * 256 : p.out[g] + ct * 256;
    const int cbyte = (32 * wn + (fg & 1) * 16 + (fg >> 1) * 8) * 2;
#pragma unroll
    for (int i = 0; i < 4; ++i) {
        int rk, r; map_unit(rt * 8 + wm * 4 + i, u, rk, r);
        const int grow = rk * SROWS + r + fr;
        u32 a0 = cvt_pk(gelu1(acc[i][0][0]), gelu1(acc[i][0][1])), a1 = cvt_pk(gelu1(acc[i][0][2]), gelu1(acc[i][0][3]));
        u32 b0 = cvt_pk(gelu1(acc[i][1][0]), gelu1(acc[i][1][1])), b1 = cvt_pk(gelu1(acc[i][1][2]), gelu1(acc[i][1][3]));
        auto s0 = __builtin_amdgcn_permlane16_swap(a0, b0, false, false);
        auto s1 = __builtin_amdgcn_permlane16_swap(a1, b1, false, false);
        u32x4 o = (u32x4){s0[0], s1[0], s0[1], s1[1]};
        if (bad) o = (u32x4){0x7fc07fc0u, 0x7fc07fc0u, 0x7fc07fc0u, 0x7fc07fc0u};
        char* dst = obase + (size_t)grow * 1024 + cbyte;
        if (helper) st16_sys(dst, o); else *(GAS u32x4*)dst = o;
    }
    WAITV(0);
    STAMP(2);
    BAR();
}

__device__ __forceinline__ void stagger(int ticks16) { if (ticks16 > 0) { const u64 t = rtc() + (u64)(ticks16 >> 4); while (rtc() < t) __builtin_amdgcn_s_sleep(1); } }

__device__ __forceinline__ void send_ingroup(const Params& p, u32 call, int id, int nw) {
    const int g = p.rank, u = g >> 2, v = g & 3, par = call & 1, lane = threadIdx.x & 63;
    const char* d0 = p.ws[4 * u + ((v + 1) & 3)] + OFF_A + (size_t)par * A_BYTES;
    const char* d1 = p.ws[4 * u + ((v + 2) & 3)] + OFF_A + (size_t)par * A_BYTES;
    const char* d2 = p.ws[4 * u + ((v + 3) & 3)] + OFF_A + (size_t)par * A_BYTES;
    char* fl = p.ws[4 * u + ((v + 1 + (lane < 3 ? lane : 0)) & 3)] + OFF_XRDY;
    const u32 loff = (u32)(lane >> 5) * 8192u + (u32)(lane & 31) * 16u;
    int t = id, step = 0;
    stagger(id * p.stag_i);
    STAMP(0);
    u32x4 d[8];
    int n = 0;
    auto locate = [&](int& G, int& kc) -> bool {
        for (;;) { if (step >= 32) return false; const int sc = p.sched[step]; G = sc >> 4; kc = sc & 15; const int cnt = G ? A1 / 16 : A0 / 16; if (t < cnt) return true; t -= cnt; ++step; } };
    int G, kc;
    bool have = locate(G, kc);
    if (have) { const char* s = (const char*)p.x + (size_t)((G ? A0 : 0) + 16 * t) * 8192 + kc * 512;
#pragma unroll
        for (int i = 0; i < 8; ++i) d[i] = *(const GAS u32x4*)(s + loff + i * 16384); }
    while (have) {
        const int lr0 = G * 1024 + v * (G ? A1 : A0) + 16 * t;
        const u32 doff = (u32)lr0 * 8192u + (u32)kc * 512u + loff;
#pragma unroll
        for (int i = 0; i < 8; ++i) { st16_sys_s(doff + i * 16384, d[i], d0); st16_sys_s(doff + i * 16384, d[i], d1); st16_sys_s(doff + i * 16384, d[i], d2); }
        const u32 foff = (u32)((lr0 >> 7) * NKC + kc) * 128u;
        t += nw; have = locate(G, kc);
        if (have) { const char* s = (const char*)p.x + (size_t)((G ? A0 : 0) + 16 * t) * 8192 + kc * 512;
#pragma unroll
            for (int i = 0; i < 8; ++i) d[i] = *(const GAS u32x4*)(s + loff + i * 16384); }
        asm volatile("s_waitcnt vmcnt(0)" ::: "memory");
        if (lane < 3) __hip_atomic_fetch_add((gu32*)(fl + foff), 1u, __ATOMIC_RELAXED, __HIP_MEMORY_SCOPE_SYSTEM);
        ++n; if (n < 30) STAMP(1 + n);
    }
    STAMP(1);
}

__device__ __forceinline__ void send_cross(const Params& p, u32 call, int id, int nw) {
    const int g = p.rank, u = g >> 2, v = g & 3, par = call & 1, lane = threadIdx.x & 63;
    const int o = 4 * (1 - u);
    const char* d0 = p.ws[o + 0] + OFF_A + (size_t)par * A_BYTES;
    const char* d1 = p.ws[o + 1] + OFF_A + (size_t)par * A_BYTES;
    const char* d2 = p.ws[o + 2] + OFF_A + (size_t)par * A_BYTES;
    const char* d3 = p.ws[o + 3] + OFF_A + (size_t)par * A_BYTES;
    char* wd = p.ws[o + (lane >> 4)] + OFF_W + (size_t)par * W_BYTES + v * 256 + (lane & 15) * 16;
    char* fl = p.ws[o + (lane & 3)];
    const u32 loff = (u32)(lane >> 5) * 8192u + (u32)(lane & 31) * 16u;
    int t = id, step = 0;
    stagger(id * p.stag_x);
    STAMP(0);
    u32x4 d[8];
    int n = 0;
    auto locate = [&](int& G, int& kc) -> bool {
        for (;;) { if (step >= 32) return false; const int sc = p.sched[step]; G = sc >> 4; kc = sc & 15; const int cnt = G ? B1 / 16 : B0 / 16 + 32; if (t < cnt) return true; t -= cnt; ++step; } };
    auto load = [&](int G, int kc) {
        const int nb = G ? B1 / 16 : B0 / 16;
        if (t < nb) { const char* s = (const char*)p.x + (size_t)(NA + (G ? B0 : 0) + 16 * t) * 8192 + kc * 512;
#pragma unroll
            for (int i = 0; i < 8; ++i) d[i] = *(const GAS u32x4*)(s + loff + i * 16384); }
        else { const char* s = (const char*)p.w + (size_t)(kc * KC + 8 * (t - nb)) * 1024;
#pragma unroll
            for (int i = 0; i < 8; ++i) d[i] = *(const GAS u32x4*)(s + lane * 16 + i * 1024); }
    };
    int G, kc;
    bool have = locate(G, kc);
    if (have) load(G, kc);
    while (have) {
        const int nb = G ? B1 / 16 : B0 / 16;
        u32 foff;
        if (t < nb) {
            const int lr0 = G * 1024 + 4 * (G ? A1 : A0) + v * (G ? B1 : B0) + 16 * t;
            const u32 doff = (u32)lr0 * 8192u + (u32)kc * 512u + loff;
#pragma unroll
            for (int i = 0; i < 8; ++i) { st16_sys_s(doff + i * 16384, d[i], d0); st16_sys_s(doff + i * 16384, d[i], d1); st16_sys_s(doff + i * 16384, d[i], d2); st16_sys_s(doff + i * 16384, d[i], d3); }
            foff = (u32)OFF_XRDY + (u32)((lr0 >> 7) * NKC + kc) * 128u;
        } else {
            char* dd = wd + (size_t)(kc * KC + 8 * (t - nb)) * 1024;
#pragma unroll
            for (int i = 0; i < 8; ++i) st16_sys(dd + i * 1024, d[i]);
            foff = (u32)OFF_WRDY + (u32)(v * NKC + kc) * 128u;
        }
        t += nw; have = locate(G, kc);
        if (have) load(G, kc);
        asm volatile("s_waitcnt vmcnt(0)" ::: "memory");
        if (lane < 4) __hip_atomic_fetch_add((gu32*)(fl + foff), 1u, __ATOMIC_RELAXED, __HIP_MEMORY_SCOPE_SYSTEM);
        ++n; if (n < 30) STAMP(1 + n);
    }
    STAMP(1);
}

__global__ __launch_bounds__(NTHREADS, 1) void ag_gemm_kernel(Params p) {
    extern __shared__ __attribute__((aligned(16))) char smem[];
    LAS char* lds = (LAS char*)smem;
    const int b = blockIdx.x, xcd = b & 7, slot = b >> 3, wave = uni(threadIdx.x >> 6), lane = threadIdx.x & 63;
    gu32* cnt = (gu32*)(p.ws[p.rank] + OFF_CNT + (size_t)b * 128) + wave;
    const u32 call = (u32)uni((int)*cnt) + 1u;
    if (lane == 0) *cnt = call;
    if (slot < 16) {
        if (p.mode & 1) return;
        gemm_role(p, lds, call, xcd, slot);
        return;
    }
    const int c = (slot - 16) * 8 + xcd;
    if (c < p.ci) { if (wave < p.wi && !(p.mode & 2)) send_ingroup(p, call, wave * p.ci + c, p.ci * p.wi); }
    else if (c < p.ci + p.cx) { if (wave < p.wx && !(p.mode & 2)) send_cross(p, call, wave * p.cx + (c - p.ci), p.cx * p.wx); }
    else if (c == 127 && wave == 0) {
        gu32* tmo = (gu32*)(p.ws[p.rank] + OFF_TMO);
        STAMP(0);
        if (p.mode & 1) {
            const int g = p.rank, u = g >> 2;
            for (int step = 0; step < 32; ++step) { const int sc = p.sched[step], G = sc >> 4, kc = sc & 15;
                for (int rt = G * 8; rt < G * 8 + 8; ++rt) { int nrem = 0;
                    for (int i = 0; i < 8; ++i) { int rk, r; map_unit(rt * 8 + i, u, rk, r); nrem += (rk != g); }
                    wait_ge((gu32*)(p.ws[g] + OFF_XRDY + (size_t)(rt * NKC + kc) * 128), call * (u32)nrem, tmo, call); }
                if (G == 0) for (int j = 0; j < 4; ++j) wait_ge((gu32*)(p.ws[g] + OFF_WRDY + (size_t)(j * NKC + kc) * 128), call * 32u, tmo, call);
                STAMP(1 + step);
            }
        } else if (!(p.mode & 2)) {
            if (!wait_ge((gu32*)(p.ws[p.rank] + OFF_DONE), call * 64u, tmo, call)) {
                if (lane == 0) *(GAS uint16_t*)p.out[p.rank] = 0x7fc0;
            }
        }
        STAMP(39);
    }
}
}

struct AggKnobs { int ci = 16, wi = 2, cx = 48, wx = 2, stag_i = 0, stag_x = 0, mode = 0, n1 = 8; };
static AggKnobs g_knobs;

extern "C" void kernel_launch(int rank, int nranks, ncclComm_t comm, hipStream_t stream,
                              void* const* d_in, const int* in_sizes, int n_in, void* d_out, int out_size,
                              const int* in_dtypes, int out_dtype, void* const* d_out_all, void* const* d_ws_all, size_t ws_bytes) {
    using namespace agg;
    if (nranks != NRANK || (unsigned)rank >= (unsigned)NRANK || ws_bytes < WS_NEED || n_in != 2 || in_sizes[0] != SROWS * KDIM || in_sizes[1] != KDIM * SCOLS || out_size != 4096 * SCOLS) {
        fprintf(stderr, "ag_gemm: unexpected shapes\n"); return; }
    static bool init[NRANK] = {};
    if (!init[rank]) { (void)hipFuncSetAttribute((const void*)ag_gemm_kernel, hipFuncAttributeMaxDynamicSharedMemorySize, LDS_BYTES); init[rank] = true; }
    Params p{};
    p.x = (const uint16_t*)d_in[0]; p.w = (const uint16_t*)d_in[1];
    for (int i = 0; i < NRANK; ++i) { p.out[i] = (char*)d_out_all[i]; p.ws[i] = (char*)d_ws_all[i]; }
    p.rank = rank;
    p.ci = g_knobs.ci; p.wi = g_knobs.wi; p.cx = g_knobs.cx; p.wx = g_knobs.wx; p.stag_i = g_knobs.stag_i; p.stag_x = g_knobs.stag_x; p.mode = g_knobs.mode;
    { const int n1 = g_knobs.n1; int j = 0, c0 = 0, c1 = 0;
      for (; j < 16 + n1; ++j) { const bool g1 = ((j + 1) * n1) / (16 + n1) > (j * n1) / (16 + n1); if (g1) p.sched[j] = (unsigned char)(16 | c1++); else p.sched[j] = (unsigned char)c0++; }
      for (; j < 32; ++j) p.sched[j] = (unsigned char)(16 | c1++); }
    hipLaunchKernelGGL(ag_gemm_kernel, dim3(NWG), dim3(NTHREADS), LDS_BYTES, stream, p);
}
```

```cpp
#include <hip/hip_runtime.h>
#include <rccl/rccl.h>
#include <cstdio>
#include <cstdint>
#define LOAD_AUX 17
#define EPI_PAT 1
#define PFD 0

namespace agg {
typedef unsigned u32;
typedef unsigned long long u64;
typedef u32 u32x4 __attribute__((ext_vector_type(4)));
typedef u32 u32x2 __attribute__((ext_vector_type(2)));
typedef float f32x4 __attribute__((ext_vector_type(4)));
typedef short s16x4 __attribute__((ext_vector_type(4)));
typedef short s16x8 __attribute__((ext_vector_type(8)));
typedef __bf16 bf16x8 __attribute__((ext_vector_type(8)));
#define LAS __attribute__((address_space(3)))
#define GAS __attribute__((address_space(1)))
typedef GAS u32 gu32;

constexpr int NRANK = 8, KDIM = 4096, SROWS = 512, SCOLS = 512;
constexpr int KC = 256, NKC = KDIM / KC;
#define P_A0 192
#define P_A1 144
constexpr int A0 = P_A0, A1 = P_A1, B0 = 256 - A0, B1 = 256 - A1, NA = A0 + A1;
static_assert(A0 % 16 == 0 && A1 % 16 == 0 && NA <= SROWS && B0 + B1 + NA == SROWS, "row split");
constexpr int BK = 64, NST = KDIM / BK, NBUF = 4, STAGE_B = 32768;
constexpr int LDS_CTL = NBUF * STAGE_B, LDS_BYTES = LDS_CTL + 64;
constexpr int NTHREADS = 576, NWG = 256;

constexpr size_t OFF_CNT = 0;
constexpr size_t OFF_TMO = 32768;
constexpr size_t OFF_DONE = OFF_TMO + 128;
constexpr size_t OFF_DUMMY = OFF_TMO + 256;
constexpr size_t OFF_XRDY = 40960;
constexpr size_t OFF_WRDY = OFF_XRDY + 16 * 128;
constexpr size_t OFF_DBG = 131072;
constexpr size_t DBG_BYTES = 1048576 - OFF_DBG;
constexpr int NSLOT = 2048 + 512;
constexpr size_t OFF_HDR = 1048576;
constexpr size_t A_BYTES = (size_t)2048 * KDIM * 2, W_BYTES = (size_t)KDIM * 512 * 2;
constexpr size_t OFF_A = 2097152;
constexpr size_t OFF_W = OFF_A + A_BYTES;
constexpr size_t PK_BYTES = (size_t)NSLOT * 8192;
constexpr size_t OFF_PK = OFF_W + W_BYTES;
constexpr size_t WS_NEED = OFF_PK + 2 * PK_BYTES;
static_assert(OFF_WRDY + 4 * 128 <= OFF_DBG && OFF_HDR + 2 * NSLOT * 128 <= OFF_A, "ctl");
static_assert(WS_NEED <= (size_t)65 * 1048576, "workspace");
constexpr u64 DEADLINE = 30000000ull;

struct Params {
    const uint16_t* x; const uint16_t* w;
    char* out[NRANK]; char* ws[NRANK];
    int rank;
    int nsw, wpw;
    int nuw, upw;
    int stag;
    int mode;
    unsigned gmask;
};

__device__ __forceinline__ u64 rtc() { return __builtin_amdgcn_s_memrealtime(); }
__device__ __forceinline__ int uni(int v) { return __builtin_amdgcn_readfirstlane(v); }

__device__ __forceinline__ void map_unit(int unit, int u, int& rank, int& r) {
    const int lr = unit * 16, G = lr >> 10; int i = lr & 1023;
    if (G == 0) { if (i < 4 * A0) { const int j = i / A0; r = i - j * A0; rank = 4 * u + j; }
                  else { i -= 4 * A0; const int j = i / B0; r = NA + (i - j * B0); rank = 4 * (1 - u) + j; } }
    else { if (i < 4 * A1) { const int j = i / A1; r = A0 + i - j * A1; rank = 4 * u + j; }
           else { i -= 4 * A1; const int j = i / B1; r = NA + B0 + (i - j * B1); rank = 4 * (1 - u) + j; } }
}

#define STAMP(idx) do {} while (0)
#define STAMPC(idx) do {} while (0)

__device__ __forceinline__ bool wait_ge(gu32* word, u32 want, gu32* tmo, u32 call) {
    const u64 t0 = rtc();
    for (;;) {
        const u32 v = (u32)uni((int)__hip_atomic_load(word, __ATOMIC_RELAXED, __HIP_MEMORY_SCOPE_SYSTEM));
        if ((int)(v - want) >= 0) return true;
        if ((u32)uni((int)__hip_atomic_load(tmo, __ATOMIC_RELAXED, __HIP_MEMORY_SCOPE_AGENT)) == call || rtc() - t0 > DEADLINE) {
            __hip_atomic_store(tmo, call, __ATOMIC_RELAXED, __HIP_MEMORY_SCOPE_AGENT);
            return false;
        }
        __builtin_amdgcn_s_sleep(2);
    }
}

__device__ __forceinline__ void st16_sys(char* ptr, u32x4 v) {
    asm volatile("global_store_dwordx4 %0, %1, off sc0 sc1\n\ts_nop 1" :: "v"(ptr), "v"(v) : "memory");
}
__device__ __forceinline__ void st16_sys_s(u32 voff, u32x4 v, const char* sbase) {
    asm volatile("global_store_dwordx4 %0, %1, %2 sc0 sc1\n\ts_nop 1" :: "v"(voff), "v"(v), "s"(sbase) : "memory");
}

__device__ __forceinline__ float gelu1(float y) {
    const float k1 = -2.0f * 0.7978845608028654f * 1.4426950408889634f, k2 = k1 * 0.044715f;
    const float t = y * y, e = __builtin_amdgcn_exp2f(y * __builtin_fmaf(t, k2, k1));
    return y * __builtin_amdgcn_rcpf(1.0f + e);
}
__device__ __forceinline__ u32 cvt_pk(float lo, float hi) { u32 r; asm("v_cvt_pk_bf16_f32 %0, %1, %2" : "=v"(r) : "v"(lo), "v"(hi)); return r; }

#define BAR() __builtin_amdgcn_s_barrier()
#define WAITV(n) asm volatile("s_waitcnt vmcnt(" #n ")" ::: "memory")
#define WAITL0() asm volatile("s_waitcnt lgkmcnt(0)" ::: "memory")
#define SB() __builtin_amdgcn_sched_barrier(0)
#define EPI_R(it) (lane & 15)
#define EPI_C(it) (4 * (it) + (lane >> 4))
struct Frags { s16x4 w[2][2][2]; bf16x8 x[2][2]; };
typedef float f32x16 __attribute__((ext_vector_type(16)));
__device__ __forceinline__ const char* uniptr(const char* q) { const u64 a = (u64)q; return (const char*)(((u64)(u32)uni((int)(u32)(a >> 32)) << 32) | (u32)uni((int)(u32)a)); }

__device__ __forceinline__ void gemm_role(const Params& p, LAS char* lds, u32 call, int xcd, int slot) {
    const int g = p.rank, u = g >> 2, v = g & 3;
    const int wave = uni(threadIdx.x >> 6), lane = threadIdx.x & 63;
    const int q = xcd >> 1, helper = xcd & 1, rsel = slot >> 2, ct = slot & 3;
    const int rt = (rsel >> 1) * 8 + 2 * q + (rsel & 1);
    char* const ws = p.ws[g];
    gu32* tmo = (gu32*)(ws + OFF_TMO);
    LAS u32* ctl = (LAS u32*)(lds + LDS_CTL);
    if (wave == 8) {
        int nrem = 0;
        for (int i = 0; i < 8; ++i) { int rk, r; map_unit(rt * 8 + i, u, rk, r); nrem += (rk != g); }
        const bool wl = helper && lane >= 16;
        gu32* word = (gu32*)(ws + (wl ? OFF_WRDY + (size_t)ct * 128 : OFF_XRDY + (size_t)rt * 128)) + (lane & 15);
        const u32 want = call * (wl ? 8u : (u32)nrem);
        bool ok = true; int ready = (p.mode & 2) ? 16 : 0;
        auto need = [&](int c) {
            if (ready > c) return;
            const u64 t0 = rtc();
            for (;;) {
                u32 val = want;
                if (lane < 32) val = __hip_atomic_load(word, __ATOMIC_RELAXED, __HIP_MEMORY_SCOPE_SYSTEM);
                const u64 m = __ballot((int)(val - want) >= 0);
                const u32 mm = (u32)m & (u32)(m >> 16) & 0xffffu;
                ready = __builtin_ctz(~mm);
                if (ready > c) return;
                if ((u32)uni((int)__hip_atomic_load(tmo, __ATOMIC_RELAXED, __HIP_MEMORY_SCOPE_AGENT)) == call || rtc() - t0 > DEADLINE) {
                    __hip_atomic_store(tmo, call, __ATOMIC_RELAXED, __HIP_MEMORY_SCOPE_AGENT); ok = false; ready = 16; return; }
                __builtin_amdgcn_s_sleep(1);
            }
        };
        STAMP(0);
        need(0);
        STAMP(1);
        BAR(); BAR();
        for (int s = 0; s < NST - 1; ++s) {
            if ((s & 3) == 0 && s + 4 < NST) { need((s >> 2) + 1); STAMP(2 + (s >> 2)); }
            if (s == NST - 8) {
                const u32 t = (u32)uni((int)__hip_atomic_load(tmo, __ATOMIC_RELAXED, __HIP_MEMORY_SCOPE_AGENT));
                if (lane == 0) ctl[0] = (!ok || t == call) ? 1u : 0u;
            }
            BAR();
        }
        WAITL0();
        BAR();
        BAR();
        BAR();
        if (helper && lane == 0) {
            const int h = 4 * (1 - u) + ct;
            __hip_atomic_fetch_add((gu32*)(p.ws[h] + OFF_DONE), 1u, __ATOMIC_RELAXED, __HIP_MEMORY_SCOPE_SYSTEM);
        }
        STAMP(20);
        return;
    }
    const int kh = wave >> 2, wm = (wave >> 1) & 1, wn = wave & 1;
    int urk, urow; map_unit(rt * 8 + wave, u, urk, urow);
    const char* asrc = uniptr((urk == g) ? (const char*)p.x + (size_t)urow * 8192 : ws + OFF_A + (size_t)(rt * 128 + wave * 16) * 8192);
    const char* bsrc = uniptr(helper ? ws + OFF_W + ct * 256 : (const char*)p.w + ct * 256);
    u32 aoff[2], boff[2];
#pragma unroll
    for (int i = 0; i < 2; ++i) {
        const int ar = 8 * i + (lane >> 3);
        aoff[i] = (u32)ar * 8192u + (u32)(((lane & 7) ^ ((ar >> 1) & 7)) * 16);
        const int kr = 8 * wave + 4 * i + (lane >> 4);
        const int sw = ((kr & 3) << 2) | ((kr >> 2) & 3);
        boff[i] = (u32)kr * 1024u + (u32)(((lane & 15) ^ sw) * 16);
    }
    auto issue1 = [&](int s, int buf, int i) {
        if (i < 2) __builtin_amdgcn_global_load_lds((const GAS void*)(asrc + (size_t)s * 128 + aoff[i]), (LAS void*)(lds + buf * 16384 + (16 * wave + 8 * i) * 128), 16, 0, LOAD_AUX);
        else __builtin_amdgcn_global_load_lds((const GAS void*)(bsrc + (size_t)s * 65536 + boff[i - 2]), (LAS void*)(lds + 65536 + buf * 16384 + (8 * wave + 4 * (i - 2)) * 256), 16, 0, LOAD_AUX);
    };
    auto issue = [&](int s) { issue1(s, s & 3, 0); issue1(s, s & 3, 1); issue1(s, s & 3, 2); issue1(s, s & 3, 3); };
    const u32 ldsb = (u32)(size_t)lds;
    const int l31 = lane & 31, lh = lane >> 5;
    u32 xa[2], wa[2][2];
#pragma unroll
    for (int ks = 0; ks < 2; ++ks) xa[ks] = ldsb + (u32)((64 * wm + l31) * 128 + (((4 * kh + 2 * ks + lh) ^ ((l31 >> 1) & 7)) * 16));
    { const int blk = (lane >> 4) & 1, tq = (lane & 15) >> 2, tp = lane & 3;
#pragma unroll
      for (int c = 0; c < 2; ++c)
#pragma unroll
        for (int t = 0; t < 2; ++t) {
            const int kr = 32 * kh + 8 * lh + 4 * t + tq;
            const int sw = ((kr & 3) << 2) | ((kr >> 2) & 3);
            const int ch = 8 * wn + 4 * c + 2 * blk + (tp >> 1);
            wa[c][t] = ldsb + (u32)(65536 + kr * 256 + ((ch ^ sw) * 16) + 8 * (tp & 1));
        } }
    f32x16 acc[2][2];
#pragma unroll
    for (int i = 0; i < 2; ++i)
#pragma unroll
        for (int c = 0; c < 2; ++c)
#pragma unroll
            for (int e = 0; e < 16; ++e) acc[i][c][e] = 0.f;
#define RDW(f, BUF, ks, c) asm volatile("ds_read_b64_tr_b16 %0, %2 offset:%c4\n\tds_read_b64_tr_b16 %1, %3 offset:%c4" : "=&v"(f.w[ks][c][0]), "=&v"(f.w[ks][c][1]) : "v"(wa[c][0]), "v"(wa[c][1]), "i"((BUF) * 16384 + (ks) * 4096) : "memory")
#define RDX(f, BUF, ks, i) asm volatile("ds_read_b128 %0, %1 offset:%c2" : "=v"(f.x[ks][i]) : "v"(xa[ks]), "i"((BUF) * 16384 + (i) * 4096) : "memory")
#define LANDED(f) asm volatile("s_waitcnt lgkmcnt(0)" : "+v"(f.w[0][0][0]), "+v"(f.w[0][0][1]), "+v"(f.w[0][1][0]), "+v"(f.w[0][1][1]), "+v"(f.w[1][0][0]), "+v"(f.w[1][0][1]), "+v"(f.w[1][1][0]), "+v"(f.w[1][1][1]), \
                     "+v"(f.x[0][0]), "+v"(f.x[0][1]), "+v"(f.x[1][0]), "+v"(f.x[1][1]) :: "memory")
#define MF(f, ks, i, c) do { acc[i][c] = __builtin_amdgcn_mfma_f32_32x32x16_bf16(__builtin_bit_cast(bf16x8, __builtin_shufflevector(f.w[ks][c][0], f.w[ks][c][1], 0, 1, 2, 3, 4, 5, 6, 7)), f.x[ks][i], acc[i][c], 0, 0, 0); SB(); } while (0)
#define STEP(cur, nxt, s, CB, NB, ISSUE) do { BAR(); SB(); \
        MF(cur, 0, 0, 0); RDW(nxt, NB, 0, 0); if (ISSUE) issue1((s) + 4, CB, 0); SB(); \
        MF(cur, 0, 0, 1); RDX(nxt, NB, 0, 0); RDX(nxt, NB, 0, 1); SB(); \
        MF(cur, 0, 1, 0); RDW(nxt, NB, 0, 1); if (ISSUE) issue1((s) + 4, CB, 1); SB(); \
        MF(cur, 0, 1, 1); RDW(nxt, NB, 1, 0); if (ISSUE) issue1((s) + 4, CB, 2); SB(); \
        MF(cur, 1, 0, 0); RDX(nxt, NB, 1, 0); RDX(nxt, NB, 1, 1); SB(); \
        MF(cur, 1, 0, 1); RDW(nxt, NB, 1, 1); if (ISSUE) issue1((s) + 4, CB, 3); SB(); \
        MF(cur, 1, 1, 0); MF(cur, 1, 1, 1); \
        LANDED(nxt); } while (0)
    Frags P, Q;
    BAR();
    STAMP(0); STAMPC(30);
    issue(0); issue(1); issue(2);
    WAITV(8); BAR();
    issue(3);
    RDW(P, 0, 0, 0); RDX(P, 0, 0, 0); RDX(P, 0, 0, 1); RDW(P, 0, 0, 1); RDW(P, 0, 1, 0); RDX(P, 0, 1, 0); RDX(P, 0, 1, 1); RDW(P, 0, 1, 1); LANDED(P);
    for (int s = 0; s < NST - 4; s += 4) {
        WAITV(8); STEP(P, Q, s, 0, 1, true);
        WAITV(8); STEP(Q, P, s + 1, 1, 2, true);
        WAITV(8); STEP(P, Q, s + 2, 2, 3, true);
        WAITV(8); STEP(Q, P, s + 3, 3, 0, true);
    }
    WAITV(8); STEP(P, Q, NST - 4, 0, 1, false);
    WAITV(4); STEP(Q, P, NST - 3, 1, 2, false);
    WAITV(0); STEP(P, Q, NST - 2, 2, 3, false);
    MF(Q, 0, 0, 0); MF(Q, 0, 0, 1); MF(Q, 0, 1, 0); MF(Q, 0, 1, 1); MF(Q, 1, 0, 0); MF(Q, 1, 0, 1); MF(Q, 1, 1, 0); MF(Q, 1, 1, 1);
    STAMP(1); STAMPC(31);
    LAS char* xslot = lds + (wave < 6 ? wave * 8192 : 65536 + (wave - 6) * 8192);
    LAS char* pslot = lds + ((wave ^ 4) < 6 ? (wave ^ 4) * 8192 : 65536 + ((wave ^ 4) - 6) * 8192);
#pragma unroll
    for (int c = 0; c < 2; ++c)
#pragma unroll
        for (int e = 0; e < 4; ++e) { const f32x16& a = kh ? acc[0][c] : acc[1][c];
            ((LAS f32x4*)xslot)[(c * 4 + e) * 64 + lane] = (f32x4){a[4 * e], a[4 * e + 1], a[4 * e + 2], a[4 * e + 3]}; }
    WAITL0(); BAR();
    const bool bad = ctl[0] != 0u;
    LAS char* otile = lds + 98304;
    { const int row = 64 * wm + 32 * kh + l31;
      f32x4 o[2][4];
#pragma unroll
      for (int c = 0; c < 2; ++c)
#pragma unroll
        for (int e = 0; e < 4; ++e) o[c][e] = ((LAS f32x4*)pslot)[(c * 4 + e) * 64 + lane];
      u32x2 pk[2][4];
#pragma unroll
      for (int c = 0; c < 2; ++c)
#pragma unroll
        for (int e = 0; e < 4; ++e) {
            const f32x16& a = kh ? acc[1][c] : acc[0][c];
            pk[c][e] = (u32x2){cvt_pk(gelu1(a[4 * e] + o[c][e][0]), gelu1(a[4 * e + 1] + o[c][e][1])), cvt_pk(gelu1(a[4 * e + 2] + o[c][e][2]), gelu1(a[4 * e + 3] + o[c][e][3]))};
        }
#pragma unroll
      for (int c = 0; c < 2; ++c)
#pragma unroll
        for (int e = 0; e < 4; ++e) {
            const int ch = 8 * wn + 4 * c + e;
            *(LAS u32x2*)(otile + row * 256 + ((ch ^ (row & 15)) * 16) + lh * 8) = pk[c][e];
        } }
    WAITL0(); BAR();
    { char* obase = (helper ? p.out[4 * (1 - u) + ct] + v * 256 : p.out[g] + ct * 256) + (size_t)(urk * SROWS + urow) * 1024;
      u32x4 ov[4];
#pragma unroll
      for (int it = 0; it < 4; ++it) {
          const int r = EPI_R(it), row = 16 * wave + r, ch = EPI_C(it);
          ov[it] = *(LAS u32x4*)(otile + row * 256 + ((ch ^ (row & 15)) * 16));
          if (bad) ov[it] = (u32x4){0x7fc07fc0u, 0x7fc07fc0u, 0x7fc07fc0u, 0x7fc07fc0u};
      }
#pragma unroll
      for (int it = 0; it < 4; ++it) {
          char* dst = obase + (size_t)EPI_R(it) * 1024 + EPI_C(it) * 16;
          if (helper) st16_sys(dst, ov[it]); else *(GAS u32x4*)dst = ov[it];
      } }
    if (helper) WAITV(0);
    STAMP(2);
    BAR();
}

struct Stream {
    u32 gmask; int step, t, G, kc; bool cross;
    __device__ __forceinline__ bool locate() {
        for (;;) { if (step >= 32) return false;
            G = (gmask >> step) & 1; const int c1 = __builtin_popcount(gmask & ((1u << step) - 1u)); kc = G ? c1 : step - c1;
            const int cn = cross ? (G ? B1 / 16 : B0 / 16 + 8) : (G ? A1 / 16 : A0 / 16); if (t < cn) return true; t -= cn; ++step; } }
    __device__ __forceinline__ bool isW() const { return cross && G == 0 && t >= B0 / 16; }
    __device__ __forceinline__ int lrow(int sv) const { return G * 1024 + (cross ? 4 * (G ? A1 : A0) + sv * (G ? B1 : B0) : sv * (G ? A1 : A0)) + 16 * t; }
    __device__ __forceinline__ int slot(int sv) const { return isW() ? 2048 + sv * 128 + kc * 8 + (t - B0 / 16) : (lrow(sv) >> 4) * 16 + kc; }
};

__device__ __forceinline__ void stagger(int ticks16) { if (ticks16 > 0) { const u64 t = rtc() + (u64)(ticks16 >> 4); while (rtc() < t) __builtin_amdgcn_s_sleep(1); } }
__device__ __forceinline__ u32 vperm(u32 s0, u32 s1, u32 sel) { return __builtin_amdgcn_perm(s0, s1, sel); }
__device__ __forceinline__ u32 pkmax(u32 a, u32 b) { u32 r; asm("v_pk_max_u16 %0, %1, %2" : "=v"(r) : "v"(a), "v"(b)); return r; }
__device__ __forceinline__ u32 wave_max(u32 x) {
    x = max(x, (u32)__builtin_amdgcn_update_dpp(0, (int)x, 0x111, 0xf, 0xf, true));
    x = max(x, (u32)__builtin_amdgcn_update_dpp(0, (int)x, 0x112, 0xf, 0xf, true));
    x = max(x, (u32)__builtin_amdgcn_update_dpp(0, (int)x, 0x114, 0xf, 0xf, true));
    x = max(x, (u32)__builtin_amdgcn_update_dpp(0, (int)x, 0x118, 0xf, 0xf, true));
    const u32 a = (u32)__builtin_amdgcn_readlane((int)x, 15), b = (u32)__builtin_amdgcn_readlane((int)x, 31), c = (u32)__builtin_amdgcn_readlane((int)x, 47), d = (u32)__builtin_amdgcn_readlane((int)x, 63);
    return max(max(a, b), max(c, d));
}
struct Raw { u32x4 d[8]; };
#define GLOAD(dst, voff, sbase) asm volatile("global_load_dwordx4 %0, %1, %2" : "=v"(dst) : "v"(voff), "s"(sbase) : "memory")
#define GLOAD_SYS(dst, voff, sbase, off) asm volatile("global_load_dwordx4 %0, %1, %2 offset:" #off " sc0 sc1" : "=v"(dst) : "v"(voff), "s"(sbase) : "memory")
#define PIN8(R) "+v"(R.d[0]), "+v"(R.d[1]), "+v"(R.d[2]), "+v"(R.d[3]), "+v"(R.d[4]), "+v"(R.d[5]), "+v"(R.d[6]), "+v"(R.d[7])
#define GSTORE_SYS(voff, val, sbase, off) asm volatile("global_store_dwordx4 %0, %1, %2 offset:" #off " sc0 sc1\n\ts_nop 1" :: "v"(voff), "v"(val), "s"(sbase) : "memory")

__device__ __forceinline__ void send_role(const Params& p, gu32* cnt, int id, int nw) {
    const int g = p.rank, u = g >> 2, v = g & 3, lane = threadIdx.x & 63;
    const int di = id % 7, nsub = nw / 7;
    if (id / 7 >= nsub) { const u32 c = *cnt; if (lane == 0) *cnt = c + 1; return; }
    const bool cross = di >= 3;
    const int dest = cross ? 4 * (1 - u) + (di - 3) : 4 * u + ((v + 1 + di) & 3);
    const u32 vx0 = (u32)(lane >> 2) * 8192u + (u32)(lane & 3) * 16u, vw0 = (u32)(lane >> 3) * 1024u + (u32)(lane & 7) * 16u;
    Stream ld{p.gmask, 0, id / 7, 0, 0, cross};
    int last_slot = 0;
    auto load = [&](Raw& R) -> int {
        const bool have = ld.locate();
        const char* sb; u32 v0, v1, js; int slot = -1;
        if (have && ld.isW()) { sb = (const char*)p.w + (size_t)(ld.kc * KC + 32 * (ld.t - B0 / 16)) * 1024 + (di - 3) * 256; v0 = vw0; v1 = vw0 + 128; js = 8192; }
        else { const int r0 = have ? (cross ? NA + (ld.G ? B0 : 0) : (ld.G ? A0 : 0)) + 16 * ld.t : 0; sb = (const char*)p.x + (size_t)r0 * 8192 + (have ? ld.kc : 0) * 512; v0 = vx0; v1 = vx0 + 64; js = 128; }
        if (have) { slot = ld.slot(v); ld.t += nsub; }
        GLOAD(R.d[0], v0, sb); GLOAD(R.d[1], v1, sb); sb += js; GLOAD(R.d[2], v0, sb); GLOAD(R.d[3], v1, sb); sb += js;
        GLOAD(R.d[4], v0, sb); GLOAD(R.d[5], v1, sb); sb += js; GLOAD(R.d[6], v0, sb); GLOAD(R.d[7], v1, sb);
        return slot;
    };
    Raw RA, RB;
    int slotA = load(RA), slotB = load(RB);
    const u32 call = (u32)uni((int)*cnt) + 1u;
    if (lane == 0) *cnt = call;
    asm volatile("s_waitcnt vmcnt(0)" : PIN8(RA) :: "memory");
    const int par = call & 1;
    const char* dP = p.ws[dest] + OFF_PK + (size_t)par * PK_BYTES;
    char* dH = p.ws[dest] + OFF_HDR + (size_t)par * NSLOT * 128;
    char* prevH = p.ws[g] + OFF_DUMMY; u32 prevInfo = 0;
    const u32 vs = (u32)lane * 16u, vs2 = vs + 4096u;
    stagger((id / 7) * p.stag);
    STAMP(0);
    int n = 0;
    auto iter = [&](Raw& R, int& slot, bool first) {
        if (!first) asm volatile("s_waitcnt vmcnt(15)" : PIN8(R) :: "memory");
        u32 m = 0;
#pragma unroll
        for (int e = 0; e < 4; ++e) { m = pkmax(m, R.d[0][e] & 0x7fff7fffu); m = pkmax(m, R.d[5][e] & 0x7fff7fffu); }
        const u32 base = wave_max(max(m & 0xffffu, m >> 16)) >> 8;
        const u32 low = base > 7u ? base - 7u : 0u, Lx = low * 0x01010101u;
        u32x4 lo[4], hi[4], nb[2]; u32 bad = 0;
#pragma unroll
        for (int j = 0; j < 4; ++j) {
            const u32x4 a = R.d[2 * j], b = R.d[2 * j + 1];
            lo[j] = (u32x4){vperm(a[1], a[0], 0x06040200u), vperm(a[3], a[2], 0x06040200u), vperm(b[1], b[0], 0x06040200u), vperm(b[3], b[2], 0x06040200u)};
            hi[j] = (u32x4){vperm(a[1], a[0], 0x07050301u), vperm(a[3], a[2], 0x07050301u), vperm(b[1], b[0], 0x07050301u), vperm(b[3], b[2], 0x07050301u)};
            const u32 t0 = hi[j][0] - Lx, t1 = hi[j][1] - Lx, t2 = hi[j][2] - Lx, t3 = hi[j][3] - Lx;
            bad |= t0 | t1 | t2 | t3;
            nb[j >> 1][2 * (j & 1)] = (0x0f0f0f0fu & (t0 | (t0 >> 4))) | (0xf0f0f0f0u & (t1 | (t1 << 4)));
            nb[j >> 1][2 * (j & 1) + 1] = (0x0f0f0f0fu & (t2 | (t2 >> 4))) | (0xf0f0f0f0u & (t3 | (t3 << 4)));
        }
        u32 info = low;
        bool rawmode = false;
        if (__ballot((bad & 0x78787878u) != 0u) != 0ull) {
            u32 cntl = 0, where = 0, val = 0;
#pragma unroll
            for (int j = 0; j < 4; ++j) {
                u32 ts[4];
#pragma unroll
                for (int e = 0; e < 4; ++e) {
                    const u32 h = hi[j][e], x = ((h & 0x7f7f7f7fu) | 0x80808080u) - Lx, y = ((x ^ 0x80808080u) & 0xf8f8f8f8u) >> 3;
                    const u32 z = (y + 0x1f1f1f1fu) & 0x20202020u;
                    if (z) { const u32 byte = (u32)__builtin_ctz(z) >> 3; where = (u32)(j * 16 + e * 4) + byte; val = (h >> (8 * byte)) & 0xffu; }
                    cntl += (u32)__builtin_popcount(z);
                    ts[e] = (x & 0x07070707u) | (h & 0x80808080u);
                }
                nb[j >> 1][2 * (j & 1)] = (0x0f0f0f0fu & (ts[0] | (ts[0] >> 4))) | (0xf0f0f0f0u & (ts[1] | (ts[1] << 4)));
                nb[j >> 1][2 * (j & 1) + 1] = (0x0f0f0f0fu & (ts[2] | (ts[2] >> 4))) | (0xf0f0f0f0u & (ts[3] | (ts[3] << 4)));
            }
            const u64 lm = __ballot(cntl != 0u);
            const int l0 = __builtin_ctzll(lm);
            if (__builtin_popcountll(lm) == 1 && __builtin_amdgcn_readlane((int)cntl, l0) == 1)
                info |= (1u << 7) | ((u32)l0 << 9) | ((u32)__builtin_amdgcn_readlane((int)where, l0) << 15) | ((u32)__builtin_amdgcn_readlane((int)val, l0) << 21);
            else { info |= 2u << 7; rawmode = true; }
        }
        const char* sp = dP + (size_t)slot * 8192;
        GSTORE_SYS(vs, lo[0], sp, 0); GSTORE_SYS(vs, lo[1], sp, 1024); GSTORE_SYS(vs, lo[2], sp, 2048); GSTORE_SYS(vs, lo[3], sp, 3072);
        if (!rawmode) { GSTORE_SYS(vs2, nb[0], sp, 0); GSTORE_SYS(vs2, nb[1], sp, 1024); }
        else { GSTORE_SYS(vs2, hi[0], sp, 0); GSTORE_SYS(vs2, hi[1], sp, 1024); GSTORE_SYS(vs2, hi[2], sp, 2048); GSTORE_SYS(vs2, hi[3], sp, 3072); }
        asm volatile("s_waitcnt vmcnt(15)" ::: "memory");
        { const u64 hv = ((u64)call << 32) | prevInfo;
          if (lane == 0) asm volatile("global_store_dwordx2 %0, %1, off sc0 sc1\n\ts_nop 1" :: "v"(prevH), "v"(hv) : "memory"); }
        prevH = dH + (size_t)slot * 128; prevInfo = info;
        slot = load(R);
        ++n; if (n < 30) STAMP(1 + n);
    };
    if (slotA >= 0) {
        iter(RA, slotA, true);
        for (;;) {
            if (slotB < 0) break;
            iter(RB, slotB, false);
            if (slotA < 0) break;
            iter(RA, slotA, false);
        }
        asm volatile("s_waitcnt vmcnt(0)" : PIN8(RA), PIN8(RB) :: "memory");
        const u64 hv = ((u64)call << 32) | prevInfo;
        if (lane == 0) asm volatile("global_store_dwordx2 %0, %1, off sc0 sc1\n\ts_nop 1" :: "v"(prevH), "v"(hv) : "memory");
    } else asm volatile("s_waitcnt vmcnt(0)" : PIN8(RA), PIN8(RB) :: "memory");
    (void)last_slot;
    STAMP(1);
}

__device__ __forceinline__ void unpack_role(const Params& p, u32 call, int id, int nw) {
    const int g = p.rank, u = g >> 2, v = g & 3, lane = threadIdx.x & 63, par = call & 1;
    const int si = id % 7, nsub = nw / 7;
    if (id / 7 >= nsub) return;
    const bool cross = si >= 3;
    const int sv = cross ? si - 3 : (v + 1 + si) & 3;
    char* const ws = p.ws[g];
    gu32* tmo = (gu32*)(ws + OFF_TMO);
    const u32 vs = (u32)lane * 16u, vs2 = vs + 4096u;
    const u32 vx0 = (u32)(lane >> 2) * 8192u + (u32)(lane & 3) * 16u, vw0 = (u32)(lane >> 3) * 1024u + (u32)(lane & 7) * 16u;
    Stream st{p.gmask, 0, id / 7, 0, 0, cross};
    STAMP(0);
    int n = 0;
    while (st.locate()) {
        const int slot = st.slot(sv);
        const GAS u64* hp = (const GAS u64*)(ws + OFF_HDR + ((size_t)par * NSLOT + slot) * 128);
        u32 info = 0; bool ok = true;
        { const u64 t0 = rtc();
          for (;;) {
              const u64 hv = __hip_atomic_load(hp, __ATOMIC_RELAXED, __HIP_MEMORY_SCOPE_SYSTEM);
              const u32 seq = (u32)uni((int)(u32)(hv >> 32)); info = (u32)uni((int)(u32)hv);
              if ((int)(seq - call) >= 0) break;
              if ((u32)uni((int)__hip_atomic_load(tmo, __ATOMIC_RELAXED, __HIP_MEMORY_SCOPE_AGENT)) == call || rtc() - t0 > DEADLINE) {
                  __hip_atomic_store(tmo, call, __ATOMIC_RELAXED, __HIP_MEMORY_SCOPE_AGENT); ok = false; break; }
              __builtin_amdgcn_s_sleep(1);
          } }
        if (!ok) break;
        const char* sp = ws + OFF_PK + (size_t)par * PK_BYTES + (size_t)slot * 8192;
        const u32 low = info & 127u, mode = (info >> 7) & 3u, Lx = low * 0x01010101u;
        u32x4 lo[4], hi[4];
        if (mode != 2u) {
            u32x4 nb0, nb1;
            asm volatile("global_load_dwordx4 %0, %6, %8 sc0 sc1\n\tglobal_load_dwordx4 %1, %6, %8 offset:1024 sc0 sc1\n\tglobal_load_dwordx4 %2, %6, %8 offset:2048 sc0 sc1\n\tglobal_load_dwordx4 %3, %6, %8 offset:3072 sc0 sc1\n\t"
                         "global_load_dwordx4 %4, %7, %8 sc0 sc1\n\tglobal_load_dwordx4 %5, %7, %8 offset:1024 sc0 sc1\n\ts_waitcnt vmcnt(0)"
                         : "=&v"(lo[0]), "=&v"(lo[1]), "=&v"(lo[2]), "=&v"(lo[3]), "=&v"(nb0), "=&v"(nb1) : "v"(vs), "v"(vs2), "s"(sp) : "memory");
#pragma unroll
            for (int j = 0; j < 4; ++j) {
                const u32 n0 = (j >> 1 ? nb1 : nb0)[2 * (j & 1)], n1 = (j >> 1 ? nb1 : nb0)[2 * (j & 1) + 1];
                hi[j][0] = ((n0 & 0x07070707u) | ((n0 << 4) & 0x80808080u)) + Lx; hi[j][1] = (((n0 >> 4) & 0x07070707u) | (n0 & 0x80808080u)) + Lx;
                hi[j][2] = ((n1 & 0x07070707u) | ((n1 << 4) & 0x80808080u)) + Lx; hi[j][3] = (((n1 >> 4) & 0x07070707u) | (n1 & 0x80808080u)) + Lx;
            }
            if (mode == 1u) {
                const u32 el = (info >> 9) & 63u, wh = (info >> 15) & 63u, val = (info >> 21) & 255u, sh = (wh & 3u) * 8u;
                const bool me = (u32)lane == el;
#pragma unroll
                for (int j = 0; j < 4; ++j)
#pragma unroll
                    for (int e = 0; e < 4; ++e) if ((wh >> 2) == (u32)(j * 4 + e)) hi[j][e] = me ? ((hi[j][e] & ~(0xffu << sh)) | (val << sh)) : hi[j][e];
            }
        } else {
            asm volatile("global_load_dwordx4 %0, %8, %10 sc0 sc1\n\tglobal_load_dwordx4 %1, %8, %10 offset:1024 sc0 sc1\n\tglobal_load_dwordx4 %2, %8, %10 offset:2048 sc0 sc1\n\tglobal_load_dwordx4 %3, %8, %10 offset:3072 sc0 sc1\n\t"
                         "global_load_dwordx4 %4, %9, %10 sc0 sc1\n\tglobal_load_dwordx4 %5, %9, %10 offset:1024 sc0 sc1\n\tglobal_load_dwordx4 %6, %9, %10 offset:2048 sc0 sc1\n\tglobal_load_dwordx4 %7, %9, %10 offset:3072 sc0 sc1\n\ts_waitcnt vmcnt(0)"
                         : "=&v"(lo[0]), "=&v"(lo[1]), "=&v"(lo[2]), "=&v"(lo[3]), "=&v"(hi[0]), "=&v"(hi[1]), "=&v"(hi[2]), "=&v"(hi[3]) : "v"(vs), "v"(vs2), "s"(sp) : "memory");
        }
        const bool isw = st.isW();
        const char* ob; u32 v0, v1, js; gu32* ctr;
        if (isw) { ob = ws + OFF_W + (size_t)(st.kc * KC + 32 * (st.t - B0 / 16)) * 1024 + sv * 256; v0 = vw0; v1 = vw0 + 128; js = 8192; ctr = (gu32*)(ws + OFF_WRDY + sv * 128) + st.kc; }
        else { const int lr = st.lrow(sv); ob = ws + OFF_A + (size_t)lr * 8192 + st.kc * 512; v0 = vx0; v1 = vx0 + 64; js = 128; ctr = (gu32*)(ws + OFF_XRDY + (lr >> 7) * 128) + st.kc; }
#pragma unroll
        for (int j = 0; j < 4; ++j) {
            const u32x4 a = (u32x4){vperm(hi[j][0], lo[j][0], 0x05010400u), vperm(hi[j][0], lo[j][0], 0x07030602u), vperm(hi[j][1], lo[j][1], 0x05010400u), vperm(hi[j][1], lo[j][1], 0x07030602u)};
            const u32x4 b = (u32x4){vperm(hi[j][2], lo[j][2], 0x05010400u), vperm(hi[j][2], lo[j][2], 0x07030602u), vperm(hi[j][3], lo[j][3], 0x05010400u), vperm(hi[j][3], lo[j][3], 0x07030602u)};
            GSTORE_SYS(v0, a, ob, 0); GSTORE_SYS(v1, b, ob, 0); ob += js;
        }
        asm volatile("s_waitcnt vmcnt(0)" ::: "memory");
        if (lane == 0) __hip_atomic_fetch_add(ctr, 1u, __ATOMIC_RELAXED, __HIP_MEMORY_SCOPE_AGENT);
        st.t += nsub;
        ++n; if (n < 30) STAMP(1 + n);
    }
    STAMP(1);
}

__global__ __launch_bounds__(NTHREADS, 1) void ag_gemm_kernel(Params p) {
    extern __shared__ __attribute__((aligned(16))) char smem[];
    LAS char* lds = (LAS char*)smem;
    const int b = blockIdx.x, xcd = b & 7, slot = b >> 3, wave = uni(threadIdx.x >> 6), lane = threadIdx.x & 63;
    gu32* cnt = (gu32*)(p.ws[p.rank] + OFF_CNT + (size_t)b * 128) + wave;
    const int c = (slot - 16) * 8 + xcd;
    if (slot >= 16 && c < p.nsw && wave < p.wpw && !(p.mode & 2)) { send_role(p, cnt, wave * p.nsw + c, p.nsw * p.wpw); return; }
    const u32 call = (u32)uni((int)*cnt) + 1u;
    if (lane == 0) *cnt = call;
    if (slot < 16) {
        if (p.mode & 1) return;
        gemm_role(p, lds, call, xcd, slot);
        return;
    }
    if (p.nuw == 0 && c < p.nsw) { if (wave >= p.wpw && wave < p.wpw + p.upw && !(p.mode & 2)) unpack_role(p, call, (wave - p.wpw) * p.nsw + c, p.nsw * p.upw); return; }
    if (c >= p.nsw && c < p.nsw + p.nuw) { if (wave < p.upw && !(p.mode & 2)) unpack_role(p, call, wave * p.nuw + (c - p.nsw), p.nuw * p.upw); return; }
    if (c == 127) {
        gu32* tmo = (gu32*)(p.ws[p.rank] + OFF_TMO);
        if (p.mode & 1) {
            if (wave < 8) {
            const int g = p.rank, u = g >> 2;
            STAMP(0);
            for (int step = wave; step < 32; step += 8) { const int G = (p.gmask >> step) & 1, c1 = __builtin_popcount(p.gmask & ((1u << step) - 1u)), kc = G ? c1 : step - c1;
                const int rt = G * 8 + (lane & 7); int nrem = 0;
                for (int i = 0; i < 8; ++i) { int rk, r; map_unit(rt * 8 + i, u, rk, r); nrem += (rk != g); }
                const bool wl = lane >= 8 && lane < 12 && G == 0;
                gu32* w = (gu32*)(p.ws[g] + (wl ? OFF_WRDY + (size_t)(lane - 8) * 128 : OFF_XRDY + (size_t)rt * 128)) + kc;
                const u32 want = call * (wl ? 8u : (u32)nrem);
                const u64 t0 = rtc();
                for (;;) { const u32 val = __hip_atomic_load(w, __ATOMIC_RELAXED, __HIP_MEMORY_SCOPE_SYSTEM); if (__all((int)(val - want) >= 0)) break; if (rtc() - t0 > DEADLINE) break; __builtin_amdgcn_s_sleep(2); }
                STAMP(1 + (step >> 3));
            } }
        } else if (!(p.mode & 2) && wave == 0) {
            STAMP(0);
            if (!wait_ge((gu32*)(p.ws[p.rank] + OFF_DONE), call * 64u, tmo, call)) {
                if (lane == 0) *(GAS uint16_t*)p.out[p.rank] = 0x7fc0;
            }
            STAMP(39);
        }
    }
}
}

struct AggKnobs { int nsw = 63, wpw = 2, nuw = 64, upw = 8, stag = 0, mode = 0, n1 = 12; };
static AggKnobs g_knobs;

extern "C" void kernel_launch(int rank, int nranks, ncclComm_t comm, hipStream_t stream,
                              void* const* d_in, const int* in_sizes, int n_in, void* d_out, int out_size,
                              const int* in_dtypes, int out_dtype, void* const* d_out_all, void* const* d_ws_all, size_t ws_bytes) {
    using namespace agg;
    if (nranks != NRANK || (unsigned)rank >= (unsigned)NRANK || ws_bytes < WS_NEED || n_in != 2 || in_sizes[0] != SROWS * KDIM || in_sizes[1] != KDIM * SCOLS || out_size != 4096 * SCOLS) {
        fprintf(stderr, "ag_gemm: unexpected shapes\n"); return; }
    static bool init[NRANK] = {};
    if (!init[rank]) { (void)hipFuncSetAttribute((const void*)ag_gemm_kernel, hipFuncAttributeMaxDynamicSharedMemorySize, LDS_BYTES); init[rank] = true; }
    Params p{};
    p.x = (const uint16_t*)d_in[0]; p.w = (const uint16_t*)d_in[1];
    for (int i = 0; i < NRANK; ++i) { p.out[i] = (char*)d_out_all[i]; p.ws[i] = (char*)d_ws_all[i]; }
    p.rank = rank;
    p.nsw = g_knobs.nsw; p.wpw = g_knobs.wpw; p.nuw = g_knobs.nuw; p.upw = g_knobs.upw; p.stag = g_knobs.stag; p.mode = g_knobs.mode;
    { const int n1 = g_knobs.n1; unsigned m = 0; int j = 0;
      for (; j < 16 + n1; ++j) if (((j + 1) * n1) / (16 + n1) > (j * n1) / (16 + n1)) m |= 1u << j;
      for (; j < 32; ++j) m |= 1u << j;
      p.gmask = m; }
    hipLaunchKernelGGL(ag_gemm_kernel, dim3(NWG), dim3(NTHREADS), LDS_BYTES, stream, p);
}
```

```cpp
#include <hip/hip_runtime.h>
#include <rccl/rccl.h>
#include <cstdio>
#include <cstdint>
#include <type_traits>
#define PACE0 2

namespace agg {
typedef unsigned u32;
typedef unsigned long long u64;
typedef u32 u32x4 __attribute__((ext_vector_type(4)));
typedef u32 u32x2 __attribute__((ext_vector_type(2)));
typedef float f32x4 __attribute__((ext_vector_type(4)));
typedef short s16x4 __attribute__((ext_vector_type(4)));
typedef short s16x8 __attribute__((ext_vector_type(8)));
typedef __bf16 bf16x8 __attribute__((ext_vector_type(8)));
#define LAS __attribute__((address_space(3)))
#define GAS __attribute__((address_space(1)))
typedef GAS u32 gu32;

constexpr int NRANK = 8, KDIM = 4096, SROWS = 512, SCOLS = 512;
constexpr int KC = 256, NKC = KDIM / KC;
#define P_R0 1024
#define P_A0 192
#define P_A1 144
constexpr int R0 = P_R0, R1 = 2048 - R0;
constexpr int A0 = P_A0, A1 = P_A1, B0 = R0 / 4 - A0, B1 = R1 / 4 - A1, NA = A0 + A1;
static_assert(A0 % 16 == 0 && A1 % 16 == 0 && B0 % 16 == 0 && B1 % 16 == 0 && B0 > 0 && B1 > 0 && B0 + B1 + NA == SROWS, "row split");
constexpr int NT0 = R0 / 128, NS1O = R1 / 256, NS1H = R1 / 128, NGS = NT0 + NS1O + NS1H;
static_assert(NT0 % 2 == 0 && R1 % 256 == 0 && NGS < 32, "tiles");
constexpr int NCOMM = (32 - NGS) * 8;
constexpr int NDONE = NT0 * 4 + (R1 / 64) * 4;
constexpr int BK = 64, NST = KDIM / BK;
constexpr int LDS_CTL = 147456, LDS_BYTES = LDS_CTL + 64;
constexpr int NTHREADS = 704, NWG = 256;

constexpr size_t OFF_CNT = 0;
constexpr size_t OFF_TMO = 32768;
constexpr size_t OFF_DONE = OFF_TMO + 128;
constexpr size_t OFF_DUMMY = OFF_TMO + 256;
constexpr size_t OFF_XRDY = 40960;
constexpr size_t OFF_WRDY = OFF_XRDY + 32 * 128;
constexpr size_t OFF_DBG = 65536;
constexpr size_t DBG_BYTES = 1048576 - OFF_DBG;
constexpr int NSLOT = 2048 + 512;
constexpr size_t OFF_HDR = 1048576;
constexpr size_t A_BYTES = (size_t)2048 * KDIM * 2, W_BYTES = (size_t)KDIM * 512 * 2;
constexpr size_t OFF_A = 2097152;
constexpr size_t OFF_W = OFF_A + A_BYTES;
constexpr size_t PK_BYTES = (size_t)NSLOT * 8192;
constexpr size_t OFF_PK = OFF_W + W_BYTES;
constexpr size_t WS_NEED = OFF_PK + 2 * PK_BYTES;
static_assert(OFF_WRDY + 4 * 128 <= OFF_DBG && OFF_HDR + 2 * NSLOT * 128 <= OFF_A, "ctl");
static_assert(WS_NEED <= (size_t)65 * 1048576, "workspace");
constexpr u64 DEADLINE = 30000000ull;

struct Params {
    const uint16_t* x; const uint16_t* w;
    char* out[NRANK]; char* ws[NRANK];
    int rank;
    int nsw, wpw;
    int nuw, upw;
    int nx;
    int stag;
    int mode;
    unsigned gmask;
};

__device__ __forceinline__ u64 rtc() { return __builtin_amdgcn_s_memrealtime(); }
__device__ __forceinline__ int uni(int v) { return __builtin_amdgcn_readfirstlane(v); }

__device__ __forceinline__ void map_unit(int unit, int u, int& rank, int& r) {
    const int lr = unit * 16;
    if (lr < R0) { int i = lr; if (i < 4 * A0) { const int j = i / A0; r = i - j * A0; rank = 4 * u + j; }
                   else { i -= 4 * A0; const int j = i / B0; r = NA + (i - j * B0); rank = 4 * (1 - u) + j; } }
    else { int i = lr - R0; if (i < 4 * A1) { const int j = i / A1; r = A0 + i - j * A1; rank = 4 * u + j; }
           else { i -= 4 * A1; const int j = i / B1; r = NA + B0 + (i - j * B1); rank = 4 * (1 - u) + j; } }
}

#define STAMP(idx) do {} while (0)
#define STAMPC(idx) do {} while (0)

__device__ __forceinline__ bool wait_ge(gu32* word, u32 want, gu32* tmo, u32 call) {
    const u64 t0 = rtc();
    for (;;) {
        const u32 v = (u32)uni((int)__hip_atomic_load(word, __ATOMIC_RELAXED, __HIP_MEMORY_SCOPE_SYSTEM));
        if ((int)(v - want) >= 0) return true;
        if ((u32)uni((int)__hip_atomic_load(tmo, __ATOMIC_RELAXED, __HIP_MEMORY_SCOPE_AGENT)) == call || rtc() - t0 > DEADLINE) {
            __hip_atomic_store(tmo, call, __ATOMIC_RELAXED, __HIP_MEMORY_SCOPE_AGENT);
            return false;
        }
        __builtin_amdgcn_s_sleep(2);
    }
}

__device__ __forceinline__ void st16_sys(char* ptr, u32x4 v) {
    asm volatile("global_store_dwordx4 %0, %1, off sc0 sc1\n\ts_nop 1" :: "v"(ptr), "v"(v) : "memory");
}
__device__ __forceinline__ void st16_sys_s(u32 voff, u32x4 v, const char* sbase) {
    asm volatile("global_store_dwordx4 %0, %1, %2 sc0 sc1\n\ts_nop 1" :: "v"(voff), "v"(v), "s"(sbase) : "memory");
}

__device__ __forceinline__ float gelu1(float y) {
    const float k1 = -2.0f * 0.7978845608028654f * 1.4426950408889634f, k2 = k1 * 0.044715f;
    const float t = y * y, e = __builtin_amdgcn_exp2f(y * __builtin_fmaf(t, k2, k1));
    return y * __builtin_amdgcn_rcpf(1.0f + e);
}
typedef float f32x2 __attribute__((ext_vector_type(2)));
__device__ __forceinline__ f32x2 gelu2(f32x2 y) {
    const float k1 = -2.0f * 0.7978845608028654f * 1.4426950408889634f, k2 = k1 * 0.044715f;
    const f32x2 a = y * __builtin_elementwise_fma(y * y, (f32x2){k2, k2}, (f32x2){k1, k1});
    const f32x2 d = (f32x2){__builtin_amdgcn_exp2f(a[0]), __builtin_amdgcn_exp2f(a[1])} + (f32x2){1.0f, 1.0f};
    return y * (f32x2){__builtin_amdgcn_rcpf(d[0]), __builtin_amdgcn_rcpf(d[1])};
}
__device__ __forceinline__ u32 cvt_pk(float lo, float hi) { u32 r; asm("v_cvt_pk_bf16_f32 %0, %1, %2" : "=v"(r) : "v"(lo), "v"(hi)); return r; }

#define BAR() __builtin_amdgcn_s_barrier()
#define WAITV(n) asm volatile("s_waitcnt vmcnt(" #n ")" ::: "memory")
#define WAITL0() asm volatile("s_waitcnt lgkmcnt(0)" ::: "memory")
#define SB() __builtin_amdgcn_sched_barrier(0)
typedef float f32x16 __attribute__((ext_vector_type(16)));
__device__ __forceinline__ const char* uniptr(const char* q) { const u64 a = (u64)q; return (const char*)(((u64)(u32)uni((int)(u32)(a >> 32)) << 32) | (u32)uni((int)(u32)a)); }
template <int N> __device__ __forceinline__ void waitv() { asm volatile("s_waitcnt vmcnt(%0)" :: "n"(N) : "memory"); }
__device__ __forceinline__ void glds2(u32 v0, u32 v1, const char* sb, u32 m0v) {
    asm volatile("s_mov_b32 m0, %3\n\ts_nop 0\n\tglobal_load_lds_dwordx4 %0, %2 sc0 sc1\n\tglobal_load_lds_dwordx4 %1, %2 offset:1024 sc0 sc1" :: "v"(v0), "v"(v1), "s"(sb), "s"(m0v) : "memory");
}
__device__ __forceinline__ void glds4(u32 v0, u32 v1, u32 v2, u32 v3, const char* sb, u32 m0v) {
    asm volatile("s_mov_b32 m0, %5\n\ts_nop 0\n\tglobal_load_lds_dwordx4 %0, %4 sc0 sc1\n\tglobal_load_lds_dwordx4 %1, %4 offset:1024 sc0 sc1\n\t"
                 "global_load_lds_dwordx4 %2, %4 offset:2048 sc0 sc1\n\tglobal_load_lds_dwordx4 %3, %4 offset:3072 sc0 sc1" :: "v"(v0), "v"(v1), "v"(v2), "v"(v3), "s"(sb), "s"(m0v) : "memory");
}
template <int OFF> __device__ __forceinline__ void rdx(bf16x8& d, u32 a) { asm volatile("ds_read_b128 %0, %1 offset:%c2" : "=v"(d) : "v"(a), "i"(OFF) : "memory"); }
template <int OFF> __device__ __forceinline__ void rdw(s16x4& d0, s16x4& d1, u32 a0, u32 a1) {
    asm volatile("ds_read_b64_tr_b16 %0, %2 offset:%c4\n\tds_read_b64_tr_b16 %1, %3 offset:%c4" : "=&v"(d0), "=&v"(d1) : "v"(a0), "v"(a1), "i"(OFF) : "memory"); }
template <int NF> struct Frags { s16x4 w[2][2]; bf16x8 x[2][NF]; };

template <int NF>
__device__ __forceinline__ void gemm_role(const Params& p, LAS char* lds, u32 call, int rt64, int helper, int ct, bool early) {
    constexpr int TM = 32 * NF, NBUF = NF == 4 ? 4 : 6, ASTG = TM * 128, LB = NBUF * ASTG;
    constexpr int NUN = TM / 16;
    constexpr int PPS = NF == 4 ? 16 : 12;
    constexpr int NPRE = NBUF - 1 < 4 ? NBUF - 1 : 4;
    const int g = p.rank, u = g >> 2, v = g & 3;
    const int wave = uni(threadIdx.x >> 6), lane = threadIdx.x & 63;
    char* const ws = p.ws[g];
    gu32* tmo = (gu32*)(ws + OFF_TMO);
    LAS u32* ctl = (LAS u32*)(lds + LDS_CTL);
    const u32 ldsb = (u32)(size_t)lds;
    if (wave == 10) {
        const int half = (NF == 4 && lane >= 32) ? 1 : 0;
        int nrem = 0;
        for (int i = 0; i < 4; ++i) { int rk, r; map_unit((rt64 + half) * 4 + i, u, rk, r); nrem += (rk != g); }
        const bool wl = helper && lane >= 16 && lane < 32;
        gu32* word = (gu32*)(ws + (wl ? OFF_WRDY + (size_t)ct * 128 : OFF_XRDY + (size_t)(rt64 + half) * 128)) + (lane & 15);
        const u32 want = call * (wl ? 8u : (u32)nrem);
        bool ok = true; int ready = (p.mode & 2) ? 16 : 0;
        auto need = [&](int c) {
            if (ready > c) return;
            const u64 t0 = rtc();
            for (;;) {
                u32 val = want;
                if (lane < 48) val = __hip_atomic_load(word, __ATOMIC_RELAXED, __HIP_MEMORY_SCOPE_SYSTEM);
                const u64 m = __ballot((int)(val - want) >= 0);
                const u32 mm = (u32)m & (u32)(m >> 16) & (u32)(m >> 32) & 0xffffu;
                ready = __builtin_ctz(~mm);
                if (ready > c) return;
                if ((u32)uni((int)__hip_atomic_load(tmo, __ATOMIC_RELAXED, __HIP_MEMORY_SCOPE_AGENT)) == call || rtc() - t0 > DEADLINE) {
                    __hip_atomic_store(tmo, call, __ATOMIC_RELAXED, __HIP_MEMORY_SCOPE_AGENT); ok = false; ready = 16; return; }
                __builtin_amdgcn_s_sleep(1);
            }
        };
        STAMP(0);
        need(0);
        STAMP(1);
        BAR();
        if (NBUF > 4) need(1);
        BAR();
        for (int s = 0; s < NST - 1; ++s) {
            if (((s + NBUF) & 3) == 0 && s + NBUF < NST) { need((s + NBUF) >> 2); STAMP(1 + ((s + NBUF) >> 2)); }
            if (s == NST - NBUF - 2) {
                const u32 t = (u32)uni((int)__hip_atomic_load(tmo, __ATOMIC_RELAXED, __HIP_MEMORY_SCOPE_AGENT));
                if (lane == 0) ctl[0] = (!ok || t == call) ? 1u : 0u;
                WAITL0();
            }
            BAR();
        }
        BAR();
        BAR();
        BAR();
        if (helper && lane == 0) {
            const int h = 4 * (1 - u) + ct;
            __hip_atomic_fetch_add((gu32*)(p.ws[h] + OFF_DONE), 1u, __ATOMIC_RELAXED, __HIP_MEMORY_SCOPE_SYSTEM);
        }
        STAMP(20);
        return;
    }
    if (wave >= 8) {
        const bool l0 = wave == 8;
        u32 va[2], vb[4]; const char* ub[NUN];
#pragma unroll
        for (int e = 0; e < 2; ++e) { const int ar = 8 * e + (lane >> 3); va[e] = (u32)ar * 8192u + (u32)(((lane & 7) ^ ((ar >> 1) & 7)) * 16) - (u32)e * 1024u; }
#pragma unroll
        for (int i = 0; i < NUN; ++i) { int rk, r; map_unit(rt64 * 4 + i, u, rk, r);
            ub[i] = uniptr((rk == g) ? (const char*)p.x + (size_t)r * 8192 : ws + OFF_A + (size_t)(rt64 * 64 + i * 16) * 8192); }
#pragma unroll
        for (int e = 0; e < 4; ++e) { const int kr = 4 * e + (lane >> 4), sw = ((kr & 3) << 2) | ((kr >> 2) & 3); vb[e] = (u32)kr * 1024u + (u32)(((lane & 15) ^ sw) * 16) - (u32)e * 1024u; }
        const char* b0 = uniptr(helper ? ws + OFF_W + ct * 256 : (const char*)p.w + ct * 256);
        auto issue = [&](int s, int buf) {
            const u32 la = ldsb + (u32)buf * ASTG, lb = ldsb + LB + (u32)buf * 16384u;
            const char* b = b0 + (size_t)s * 65536;
            if (l0) {
                const u32 v0 = va[0] + (u32)s * 128u, v1 = va[1] + (u32)s * 128u;
#pragma unroll
                for (int i = 0; i < NUN; ++i) glds2(v0, v1, ub[i], la + i * 2048);
                if (NF == 2) glds4(vb[0], vb[1], vb[2], vb[3], b, lb);
            } else {
#pragma unroll
                for (int q = (NF == 2 ? 1 : 0); q < 4; ++q) glds4(vb[0], vb[1], vb[2], vb[3], b + q * 16384, lb + q * 4096);
            }
        };
        BAR();
        for (int s = 0; s < NPRE; ++s) issue(s, s);
        waitv<(NPRE - 1) * PPS>(); BAR();
        for (int s = NPRE; s < NBUF; ++s) issue(s, s);
        int buf = 0;
        for (int s = 0; s < NST - NBUF; ++s) { waitv<(NBUF - 2) * PPS>(); BAR(); issue(s + NBUF, buf); buf = buf + 1 == NBUF ? 0 : buf + 1; }
        waitv<(NBUF - 2) * PPS>(); BAR();
        waitv<(NBUF - 3) * PPS>(); BAR();
        waitv<(NBUF - 4) * PPS>(); BAR();
        if (NBUF == 6) { waitv<PPS>(); BAR(); waitv<0>(); BAR(); }
        BAR(); BAR(); BAR();
        return;
    }
    const int kh = wave >> 2, wn = wave & 3;
    const int l31 = lane & 31, lh = lane >> 5;
    u32 xa[2], wa[2][2];
#pragma unroll
    for (int ks = 0; ks < 2; ++ks) xa[ks] = ldsb + (u32)(l31 * 128 + (((4 * kh + 2 * ks + lh) ^ ((l31 >> 1) & 7)) * 16));
    { const int blk = (lane >> 4) & 1, tq = (lane & 15) >> 2, tp = lane & 3;
#pragma unroll
      for (int t = 0; t < 2; ++t) {
          const int kr = 32 * kh + 8 * lh + 4 * t + tq;
          const int sw = ((kr & 3) << 2) | ((kr >> 2) & 3);
          const int ch = 4 * wn + 2 * blk + (tp >> 1);
          wa[0][t] = ldsb + (u32)(LB + kr * 256 + ((ch ^ sw) * 16) + 8 * (tp & 1));
          wa[1][t] = wa[0][t] + 3 * 16384;
      } }
    f32x16 acc[NF];
#pragma unroll
    for (int i = 0; i < NF; ++i)
#pragma unroll
        for (int e = 0; e < 16; ++e) acc[i][e] = 0.f;
#define RDW(f, BUF, ks) rdw<((BUF) % 3) * 16384 * (NBUF == 6) + (BUF) * 16384 * (NBUF == 4) + (ks) * 4096>(f.w[ks][0], f.w[ks][1], wa[NBUF == 6 ? (BUF) / 3 : 0][0], wa[NBUF == 6 ? (BUF) / 3 : 0][1])
#define RDX(f, BUF, ks, i) rdx<(BUF) * ASTG + (i) * 4096>(f.x[ks][i], xa[ks])
#define MF(f, ks, i) do { acc[i] = __builtin_amdgcn_mfma_f32_32x32x16_bf16(__builtin_bit_cast(bf16x8, __builtin_shufflevector(f.w[ks][0], f.w[ks][1], 0, 1, 2, 3, 4, 5, 6, 7)), f.x[ks][i], acc[i], 0, 0, 0); SB(); } while (0)
#define LANDED(f) do { if constexpr (NF == 4) asm volatile("s_waitcnt lgkmcnt(0)" : "+v"(f.w[0][0]), "+v"(f.w[0][1]), "+v"(f.w[1][0]), "+v"(f.w[1][1]), "+v"(f.x[0][0]), "+v"(f.x[0][1]), "+v"(f.x[0][NF - 2]), "+v"(f.x[0][NF - 1]), \
                     "+v"(f.x[1][0]), "+v"(f.x[1][1]), "+v"(f.x[1][NF - 2]), "+v"(f.x[1][NF - 1]) :: "memory"); \
        else asm volatile("s_waitcnt lgkmcnt(0)" : "+v"(f.w[0][0]), "+v"(f.w[0][1]), "+v"(f.w[1][0]), "+v"(f.w[1][1]), "+v"(f.x[0][0]), "+v"(f.x[0][1]), "+v"(f.x[1][0]), "+v"(f.x[1][1]) :: "memory"); } while (0)
#define RDALL(f, NB) do { RDW(f, NB, 0); RDX(f, NB, 0, 0); RDX(f, NB, 0, 1); if constexpr (NF == 4) { RDX(f, NB, 0, NF - 2); RDX(f, NB, 0, NF - 1); } \
        RDW(f, NB, 1); RDX(f, NB, 1, 0); RDX(f, NB, 1, 1); if constexpr (NF == 4) { RDX(f, NB, 1, NF - 2); RDX(f, NB, 1, NF - 1); } } while (0)
#define MFALL(f) do { MF(f, 0, 0); MF(f, 0, 1); if constexpr (NF == 4) { MF(f, 0, NF - 2); MF(f, 0, NF - 1); } MF(f, 1, 0); MF(f, 1, 1); if constexpr (NF == 4) { MF(f, 1, NF - 2); MF(f, 1, NF - 1); } } while (0)
#define STEP(cur, nxt, NB) do { BAR(); SB(); \
        if constexpr (NF == 4) { \
            MF(cur, 0, 0); RDW(nxt, NB, 0); RDX(nxt, NB, 0, 0); SB(); \
            MF(cur, 0, 1); RDX(nxt, NB, 0, 1); RDX(nxt, NB, 0, NF - 2); SB(); \
            MF(cur, 0, NF - 2); RDX(nxt, NB, 0, NF - 1); RDW(nxt, NB, 1); SB(); \
            MF(cur, 0, NF - 1); RDX(nxt, NB, 1, 0); RDX(nxt, NB, 1, 1); SB(); \
            MF(cur, 1, 0); RDX(nxt, NB, 1, NF - 2); RDX(nxt, NB, 1, NF - 1); SB(); \
            MF(cur, 1, 1); MF(cur, 1, NF - 2); MF(cur, 1, NF - 1); \
        } else { \
            MF(cur, 0, 0); RDW(nxt, NB, 0); RDX(nxt, NB, 0, 0); SB(); \
            MF(cur, 0, 1); RDX(nxt, NB, 0, 1); RDW(nxt, NB, 1); SB(); \
            MF(cur, 1, 0); RDX(nxt, NB, 1, 0); RDX(nxt, NB, 1, 1); SB(); \
            MF(cur, 1, 1); \
        } \
        LANDED(nxt); } while (0)
    Frags<NF> P, Q;
    BAR();
    STAMP(0); STAMPC(30);
    BAR();
    RDALL(P, 0); LANDED(P);
    for (int s = 0; s < 60; s += NBUF) {
        STEP(P, Q, 1); STEP(Q, P, 2); STEP(P, Q, 3);
        if constexpr (NBUF == 4) { STEP(Q, P, 0); }
        else { STEP(Q, P, 4 % NBUF); STEP(P, Q, 5 % NBUF); STEP(Q, P, 0); }
    }
    STEP(P, Q, 1); STEP(Q, P, 2); STEP(P, Q, 3);
    MFALL(Q);
    STAMP(1); STAMPC(31);
    constexpr int NH = NF / 2, XB = NH * 4096;
    auto xslot = [&](int w) -> LAS char* { return NF == 4 ? lds + (w < 6 ? w * XB : LB + (w - 6) * XB) : lds + LB + w * XB; };
    LAS char* otile = lds + LB + (NF == 4 ? 2 * 16384 : 4 * 16384);
    auto epi = [&](auto khc) {
        constexpr int KH = decltype(khc)::value;
        { LAS f32x4* mine = (LAS f32x4*)xslot(wave) + lane;
#pragma unroll
          for (int i = 0; i < NH; ++i) { const f32x16& a = acc[(1 - KH) * NH + i];
#pragma unroll
              for (int e = 0; e < 4; ++e) mine[(i * 4 + e) * 64] = (f32x4){a[4 * e], a[4 * e + 1], a[4 * e + 2], a[4 * e + 3]}; } }
        WAITL0(); BAR();
        const LAS f32x4* theirs = (const LAS f32x4*)xslot(wave ^ 4) + lane;
        f32x4 o[NH][4]; u32x2 pk[NH][4];
#pragma unroll
        for (int i = 0; i < NH; ++i)
#pragma unroll
            for (int e = 0; e < 4; ++e) o[i][e] = theirs[(i * 4 + e) * 64];
#pragma unroll
        for (int i = 0; i < NH; ++i) { const f32x16& a = acc[KH * NH + i];
#pragma unroll
            for (int e = 0; e < 4; ++e) {
                const f32x2 y0 = gelu2((f32x2){a[4 * e], a[4 * e + 1]} + (f32x2){o[i][e][0], o[i][e][1]}), y1 = gelu2((f32x2){a[4 * e + 2], a[4 * e + 3]} + (f32x2){o[i][e][2], o[i][e][3]});
                pk[i][e] = (u32x2){cvt_pk(y0[0], y0[1]), cvt_pk(y1[0], y1[1])}; } }
#pragma unroll
        for (int i = 0; i < NH; ++i) { const int row = 32 * (KH * NH + i) + l31;
#pragma unroll
            for (int e = 0; e < 4; ++e) *(LAS u32x2*)(otile + row * 256 + (((4 * wn + e) ^ (row & 15)) * 16) + lh * 8) = pk[i][e]; }
        WAITL0(); BAR();
    };
    if (kh) epi(std::integral_constant<int, 1>{}); else epi(std::integral_constant<int, 0>{});
    { constexpr int RPW = TM / 8, NI = RPW / 4;
      const int row0 = RPW * wave;
      int urk, urow; map_unit(rt64 * 4 + (row0 >> 4), u, urk, urow);
      char* obase = (helper ? p.out[4 * (1 - u) + ct] + v * 256 : p.out[g] + ct * 256) + (size_t)(urk * SROWS + urow + (row0 & 15)) * 1024;
      const bool bad = ctl[0] != 0u;
      u32x4 ov[NI];
#pragma unroll
      for (int it = 0; it < NI; ++it) {
          const int row = row0 + 4 * it + (lane >> 4), ch = lane & 15;
          ov[it] = *(LAS u32x4*)(otile + row * 256 + ((ch ^ (row & 15)) * 16));
          if (bad) ov[it] = (u32x4){0x7fc07fc0u, 0x7fc07fc0u, 0x7fc07fc0u, 0x7fc07fc0u};
      }
      const bool paced = helper && early;
#pragma unroll
      for (int it = 0; it < NI; ++it) {
          st16_sys(obase + (size_t)(4 * it + (lane >> 4)) * 1024 + (lane & 15) * 16, ov[it]);
          if (paced && (it + 1) % PACE0 == 0) WAITV(0);
      } }
    if (helper) WAITV(0);
    STAMP(2);
    BAR();
}

struct Stream {
    u32 gmask; int step, t, G, kc; bool cross; int c1;
    __device__ __forceinline__ bool locate() {
#pragma unroll 1
        for (;;) { if (step >= 32) return false;
            G = (gmask >> step) & 1; kc = G ? c1 : step - c1;
            const int cn = cross ? (G ? B1 / 16 : B0 / 16 + 8) : (G ? A1 / 16 : A0 / 16); if (t < cn) return true; t -= cn; c1 += G; ++step; } }
    __device__ __forceinline__ bool isW() const { return cross && G == 0 && t >= B0 / 16; }
    __device__ __forceinline__ int lrow(int sv) const { return (G ? R0 : 0) + (cross ? 4 * (G ? A1 : A0) + sv * (G ? B1 : B0) : sv * (G ? A1 : A0)) + 16 * t; }
    __device__ __forceinline__ int slot(int sv) const { return isW() ? 2048 + sv * 128 + kc * 8 + (t - B0 / 16) : (lrow(sv) >> 4) * 16 + kc; }
};

__device__ __forceinline__ void stagger(int ticks16) { if (ticks16 > 0) { const u64 t = rtc() + (u64)(ticks16 >> 4); while (rtc() < t) __builtin_amdgcn_s_sleep(1); } }
__device__ __forceinline__ u32 vperm(u32 s0, u32 s1, u32 sel) { return __builtin_amdgcn_perm(s0, s1, sel); }
__device__ __forceinline__ u32 pkmax(u32 a, u32 b) { u32 r; asm("v_pk_max_u16 %0, %1, %2" : "=v"(r) : "v"(a), "v"(b)); return r; }
__device__ __forceinline__ u32 pkadd(u32 a, u32 b) { u32 r; asm("v_pk_add_u16 %0, %1, %2" : "=v"(r) : "v"(a), "s"(b)); return r; }
__device__ __forceinline__ u32 pksub(u32 a, u32 b) { u32 r; asm("v_pk_sub_u16 %0, %1, %2" : "=v"(r) : "v"(a), "s"(b)); return r; }
__device__ __forceinline__ u32 wave_scan_incl(u32 x) {
    x += (u32)__builtin_amdgcn_update_dpp(0, (int)x, 0x111, 0xf, 0xf, true);
    x += (u32)__builtin_amdgcn_update_dpp(0, (int)x, 0x112, 0xf, 0xf, true);
    x += (u32)__builtin_amdgcn_update_dpp(0, (int)x, 0x114, 0xf, 0xf, true);
    x += (u32)__builtin_amdgcn_update_dpp(0, (int)x, 0x118, 0xf, 0xf, true);
    x += (u32)__builtin_amdgcn_update_dpp(0, (int)x, 0x142, 0xa, 0xf, true);
    x += (u32)__builtin_amdgcn_update_dpp(0, (int)x, 0x143, 0xc, 0xf, true);
    return x;
}
__device__ __forceinline__ u32 wave_max(u32 x) {
    x = max(x, (u32)__builtin_amdgcn_update_dpp(0, (int)x, 0x111, 0xf, 0xf, true));
    x = max(x, (u32)__builtin_amdgcn_update_dpp(0, (int)x, 0x112, 0xf, 0xf, true));
    x = max(x, (u32)__builtin_amdgcn_update_dpp(0, (int)x, 0x114, 0xf, 0xf, true));
    x = max(x, (u32)__builtin_amdgcn_update_dpp(0, (int)x, 0x118, 0xf, 0xf, true));
    const u32 a = (u32)__builtin_amdgcn_readlane((int)x, 15), b = (u32)__builtin_amdgcn_readlane((int)x, 31), c = (u32)__builtin_amdgcn_readlane((int)x, 47), d = (u32)__builtin_amdgcn_readlane((int)x, 63);
    return max(max(a, b), max(c, d));
}
struct Raw { u32x4 d[8]; };
#define GLOAD(dst, voff, sbase) asm volatile("global_load_dwordx4 %0, %1, %2" : "=v"(dst) : "v"(voff), "s"(sbase) : "memory")
#define GLOAD_SYS(dst, voff, sbase, off) asm volatile("global_load_dwordx4 %0, %1, %2 offset:" #off " sc0 sc1" : "=v"(dst) : "v"(voff), "s"(sbase) : "memory")
#define PIN8(R) "+v"(R.d[0]), "+v"(R.d[1]), "+v"(R.d[2]), "+v"(R.d[3]), "+v"(R.d[4]), "+v"(R.d[5]), "+v"(R.d[6]), "+v"(R.d[7])
#define GSTORE_SYS(voff, val, sbase, off) asm volatile("global_store_dwordx4 %0, %1, %2 offset:" #off " sc0 sc1\n\ts_nop 1" :: "v"(voff), "v"(val), "s"(sbase) : "memory")

__device__ __forceinline__ void send_role(const Params& p, LAS char* wlds, gu32* cnt, int id, int nw) {
    const int g = p.rank, u = g >> 2, v = g & 3, lane = threadIdx.x & 63;
    const int nx = p.nx, ni = (nw - 4 * nx) / 3 < nx ? nx : (nw - 4 * nx) / 3;
    int di, sub;
    if (id < 7 * nx) { di = id % 7; sub = id / 7; } else { di = (id - 7 * nx) % 3; sub = nx + (id - 7 * nx) / 3; }
    const int nsub = di >= 3 ? nx : ni;
    if (sub >= nsub) { const u32 c = *cnt; if (lane == 0) *cnt = c + 1; return; }
    const bool cross = di >= 3;
    const int dest = cross ? 4 * (1 - u) + (di - 3) : 4 * u + ((v + 1 + di) & 3);
    const u32 vx0 = (u32)(lane >> 2) * 8192u + (u32)(lane & 3) * 16u, vw0 = (u32)(lane >> 3) * 1024u + (u32)(lane & 7) * 16u;
    Stream ld{p.gmask, 0, sub, 0, 0, cross, 0};
    int last_slot = 0;
    auto load = [&](Raw& R) -> int {
        const bool have = ld.locate();
        const char* sb; u32 v0, v1, js; int slot = -1;
        if (have && ld.isW()) { sb = (const char*)p.w + (size_t)(ld.kc * KC + 32 * (ld.t - B0 / 16)) * 1024 + (di - 3) * 256; v0 = vw0; v1 = vw0 + 128; js = 8192; }
        else { const int r0 = have ? (cross ? NA + (ld.G ? B0 : 0) : (ld.G ? A0 : 0)) + 16 * ld.t : 0; sb = (const char*)p.x + (size_t)r0 * 8192 + (have ? ld.kc : 0) * 512; v0 = vx0; v1 = vx0 + 64; js = 128; }
        if (have) { slot = ld.slot(v); ld.t += nsub; }
        GLOAD(R.d[0], v0, sb); GLOAD(R.d[1], v1, sb); sb += js; GLOAD(R.d[2], v0, sb); GLOAD(R.d[3], v1, sb); sb += js;
        GLOAD(R.d[4], v0, sb); GLOAD(R.d[5], v1, sb); sb += js; GLOAD(R.d[6], v0, sb); GLOAD(R.d[7], v1, sb);
        return slot;
    };
    Raw RA, RB;
    int slotA = load(RA), slotB = load(RB);
    const u32 call = (u32)uni((int)*cnt) + 1u;
    if (lane == 0) *cnt = call;
    asm volatile("s_waitcnt vmcnt(0)" : PIN8(RA) :: "memory");
    const int par = call & 1;
    const char* dP = p.ws[dest] + OFF_PK + (size_t)par * PK_BYTES;
    char* dH = p.ws[dest] + OFF_HDR + (size_t)par * NSLOT * 128;
    char* prevH = p.ws[g] + OFF_DUMMY; u32 prevInfo = 0;
    const u32 vs = (u32)lane * 16u, vs2 = vs + 4096u, vs3 = (u32)lane * 8u + 5120u, vs4 = vs + 5632u;
    LAS uint16_t* elist = (LAS uint16_t*)wlds;
    stagger(sub * p.stag);
    STAMP(0);
    int n = 0;
    auto iter = [&](Raw& R, int& slot, bool first) {
        if (!first) asm volatile("s_waitcnt vmcnt(16)" : PIN8(R) :: "memory");
        u32 m = 0;
#pragma unroll
        for (int e = 0; e < 4; ++e) { m = pkmax(m, R.d[0][e] & 0x7fff7fffu); m = pkmax(m, R.d[5][e] & 0x7fff7fffu); }
        const u32 emax = wave_max(max(m & 0xffffu, m >> 16)) >> 7;
        const u32 delta = ~emax & 1u, top = (emax + delta) >> 1, low = top > 3u ? top - 3u : 0u, Lx = low * 0x01010101u, dl = delta * 0x00800080u;
        u32x4 lo[4], hi[4]; u32 cd[16]; u32 mlo = 0, mhi = 0;
        LAS u32x4* xs = (LAS u32x4*)(wlds + 1024) + lane;
#pragma unroll
        for (int j = 0; j < 4; ++j) {
            u32x4 a = R.d[2 * j], b = R.d[2 * j + 1], xv;
#pragma unroll
            for (int e = 0; e < 4; ++e) { a[e] = pkadd(a[e], dl); b[e] = pkadd(b[e], dl); }
            lo[j] = (u32x4){vperm(a[1], a[0], 0x06040200u), vperm(a[3], a[2], 0x06040200u), vperm(b[1], b[0], 0x06040200u), vperm(b[3], b[2], 0x06040200u)};
            hi[j] = (u32x4){vperm(a[1], a[0], 0x07050301u), vperm(a[3], a[2], 0x07050301u), vperm(b[1], b[0], 0x07050301u), vperm(b[3], b[2], 0x07050301u)};
            u32 z[4];
#pragma unroll
            for (int e = 0; e < 4; ++e) {
                const u32 h = hi[j][e], x = ((h & 0x7f7f7f7fu) | 0x80808080u) - Lx;
                xv[e] = x;
                cd[4 * j + e] = (x & 0x03030303u) | ((h >> 5) & 0x04040404u);
                z[e] = (((((x ^ 0x80808080u) & 0xfcfcfcfcu) >> 2) + 0x3f3f3f3fu) >> 6) & 0x01010101u;
            }
            xs[j * 64] = xv;
            const u32 m0 = __builtin_amdgcn_udot4(z[1], 0x80402010u, __builtin_amdgcn_udot4(z[0], 0x08040201u, 0u, false), false);
            const u32 m1 = __builtin_amdgcn_udot4(z[3], 0x80402010u, __builtin_amdgcn_udot4(z[2], 0x08040201u, 0u, false), false);
            const u32 m16 = m0 | (m1 << 8);
            if (j < 2) mlo |= m16 << (16 * j); else mhi |= m16 << (16 * (j - 2));
        }
        u32 idx = wave_scan_incl((u32)__builtin_popcount(mlo) + (u32)__builtin_popcount(mhi));
        const u32 total = (u32)__builtin_amdgcn_readlane((int)idx, 63);
        idx -= (u32)__builtin_popcount(mlo) + (u32)__builtin_popcount(mhi);
        bool bad = false;
        u64 mm = ((u64)mhi << 32) | mlo;
        while (__ballot(mm != 0ull) != 0ull) {
            u32 ps[4], xb[4]; bool act[4];
#pragma unroll
            for (int k = 0; k < 4; ++k) {
                act[k] = mm != 0ull; ps[k] = act[k] ? (u32)__builtin_ctzll(mm) : 0u; mm &= mm - 1ull;
                xb[k] = *(LAS uint8_t*)(wlds + 1024 + (ps[k] >> 4) * 1024u + (u32)lane * 16u + (ps[k] & 15u));
            }
#pragma unroll
            for (int k = 0; k < 4; ++k) {
                const int d4 = 32 - (int)(xb[k] >> 2);
                bad |= act[k] && (d4 > 15 || d4 < -1);
                elist[(act[k] && idx < 256u) ? idx : 256u + (u32)lane] = (uint16_t)((u32)lane * 64u + ps[k] + ((u32)max(d4, 0) << 12));
                idx += act[k] ? 1u : 0u;
            }
        }
        const bool rawmode = __ballot(bad) != 0ull || total > 256u;
        const u32 info = low | (rawmode ? 128u : 0u) | (delta << 8) | ((rawmode ? 0u : total) << 9);
        const char* sp = dP + (size_t)slot * 8192;
        GSTORE_SYS(vs, lo[0], sp, 0); GSTORE_SYS(vs, lo[1], sp, 1024); GSTORE_SYS(vs, lo[2], sp, 2048); GSTORE_SYS(vs, lo[3], sp, 3072);
        if (!rawmode) {
            u32 o[6];
#pragma unroll
            for (int gq = 0; gq < 2; ++gq) { const u32* t = &cd[8 * gq];
                o[3 * gq] = t[0] | (t[1] << 3) | ((t[2] & 0x03030303u) << 6);
                o[3 * gq + 1] = ((t[2] >> 2) & 0x01010101u) | (t[3] << 1) | (t[4] << 4) | ((t[5] & 0x01010101u) << 7);
                o[3 * gq + 2] = ((t[5] >> 1) & 0x03030303u) | (t[6] << 2) | (t[7] << 5); }
            const u32x4 c1 = (u32x4){o[0], o[1], o[2], o[3]}; const u32x2 c2 = (u32x2){o[4], o[5]};
            GSTORE_SYS(vs2, c1, sp, 0);
            asm volatile("global_store_dwordx2 %0, %1, %2 sc0 sc1\n\ts_nop 1" :: "v"(vs3), "v"(c2), "s"(sp) : "memory");
            if ((u32)lane * 8u < max(total, 1u)) { const u32x4 le = *(LAS u32x4*)(wlds + lane * 16); GSTORE_SYS(vs4, le, sp, 0); }
        } else { GSTORE_SYS(vs2, hi[0], sp, 0); GSTORE_SYS(vs2, hi[1], sp, 1024); GSTORE_SYS(vs2, hi[2], sp, 2048); GSTORE_SYS(vs2, hi[3], sp, 3072); }
        asm volatile("s_waitcnt vmcnt(16)" ::: "memory");
        { const u64 hv = ((u64)call << 32) | prevInfo;
          if (lane == 0) asm volatile("global_store_dwordx2 %0, %1, off sc0 sc1\n\ts_nop 1" :: "v"(prevH), "v"(hv) : "memory"); }
        prevH = dH + (size_t)slot * 128; prevInfo = info;
        slot = load(R);
        ++n; if (n < 30) STAMP(1 + n);
    };
    if (slotA >= 0) {
        iter(RA, slotA, true);
        for (;;) {
            if (slotB < 0) break;
            iter(RB, slotB, false);
            if (slotA < 0) break;
            iter(RA, slotA, false);
        }
        asm volatile("s_waitcnt vmcnt(0)" : PIN8(RA), PIN8(RB) :: "memory");
        const u64 hv = ((u64)call << 32) | prevInfo;
        if (lane == 0) asm volatile("global_store_dwordx2 %0, %1, off sc0 sc1\n\ts_nop 1" :: "v"(prevH), "v"(hv) : "memory");
    } else asm volatile("s_waitcnt vmcnt(0)" : PIN8(RA), PIN8(RB) :: "memory");
    (void)last_slot;
    STAMP(1);
}

__device__ __forceinline__ void unpack_role(const Params& p, LAS char* wlds, u32 call, int id, int nw) {
    const int g = p.rank, u = g >> 2, v = g & 3, lane = threadIdx.x & 63, par = call & 1;
    const int si = id % 7, nsub = nw / 7;
    if (id / 7 >= nsub) return;
    const bool cross = si >= 3;
    const int sv = cross ? si - 3 : (v + 1 + si) & 3;
    char* const ws = p.ws[g];
    gu32* tmo = (gu32*)(ws + OFF_TMO);
    const u32 vs = (u32)lane * 16u, vs2 = vs + 4096u, vs3 = (u32)lane * 8u + 5120u, vs4 = (u32)lane * 2u + 5632u;
    const u32 vx0 = (u32)(lane >> 2) * 8192u + (u32)(lane & 3) * 16u, vw0 = (u32)(lane >> 3) * 1024u + (u32)(lane & 7) * 16u;
    Stream st{p.gmask, 0, id / 7, 0, 0, cross, 0};
    STAMP(0);
    int n = 0;
    bool have = st.locate();
    u64 hv = 0;
    while (have) {
        const int slot = st.slot(sv);
        const GAS u64* hp = (const GAS u64*)(ws + OFF_HDR + ((size_t)par * NSLOT + slot) * 128);
        u32 info = 0; bool ok = true;
        { const u64 t0 = rtc();
          for (;;) {
              const u32 seq = (u32)uni((int)(u32)(hv >> 32)); info = (u32)uni((int)(u32)hv);
              if (seq != 0u && (int)(seq - call) >= 0) break;
              hv = __hip_atomic_load(hp, __ATOMIC_RELAXED, __HIP_MEMORY_SCOPE_SYSTEM);
              const u32 seq2 = (u32)uni((int)(u32)(hv >> 32)); info = (u32)uni((int)(u32)hv);
              if ((int)(seq2 - call) >= 0) break;
              if ((u32)uni((int)__hip_atomic_load(tmo, __ATOMIC_RELAXED, __HIP_MEMORY_SCOPE_AGENT)) == call || rtc() - t0 > DEADLINE) {
                  __hip_atomic_store(tmo, call, __ATOMIC_RELAXED, __HIP_MEMORY_SCOPE_AGENT); ok = false; break; }
              __builtin_amdgcn_s_sleep(1);
          } }
        if (!ok) break;
        const char* sp = ws + OFF_PK + (size_t)par * PK_BYTES + (size_t)slot * 8192;
        const u32 low = info & 127u, Lx = low * 0x01010101u, dl = ((info >> 8) & 1u) * 0x00800080u, nex = info >> 9;
        u32x4 lo[4], hi[4];
        if (!(info & 128u)) {
            u32x4 c1; u32x2 c2; u32 ent;
            asm volatile("global_load_dwordx4 %0, %7, %10 sc0 sc1\n\tglobal_load_dwordx4 %1, %7, %10 offset:1024 sc0 sc1\n\tglobal_load_dwordx4 %2, %7, %10 offset:2048 sc0 sc1\n\tglobal_load_dwordx4 %3, %7, %10 offset:3072 sc0 sc1\n\t"
                         "global_load_dwordx4 %4, %8, %10 sc0 sc1\n\tglobal_load_dwordx2 %5, %9, %10 sc0 sc1\n\tglobal_load_ushort %6, %11, %10 sc0 sc1\n\ts_waitcnt vmcnt(0)"
                         : "=&v"(lo[0]), "=&v"(lo[1]), "=&v"(lo[2]), "=&v"(lo[3]), "=&v"(c1), "=&v"(c2), "=&v"(ent) : "v"(vs), "v"(vs2), "v"(vs3), "s"(sp), "v"(vs4) : "memory");
            const u32 o[6] = {c1[0], c1[1], c1[2], c1[3], c2[0], c2[1]};
            LAS u32x4* hl = (LAS u32x4*)wlds + lane;
#pragma unroll
            for (int gq = 0; gq < 2; ++gq) { const u32* w = &o[3 * gq]; u32 t[8];
                t[0] = w[0] & 0x07070707u; t[1] = (w[0] >> 3) & 0x07070707u; t[2] = ((w[0] >> 6) & 0x03030303u) | ((w[1] & 0x01010101u) << 2);
                t[3] = (w[1] >> 1) & 0x07070707u; t[4] = (w[1] >> 4) & 0x07070707u; t[5] = ((w[1] >> 7) & 0x01010101u) | ((w[2] & 0x03030303u) << 1);
                t[6] = (w[2] >> 2) & 0x07070707u; t[7] = (w[2] >> 5) & 0x07070707u;
#pragma unroll
                for (int k = 0; k < 8; ++k) hi[2 * gq + (k >> 2)][k & 3] = ((t[k] & 0x03030303u) + Lx) | ((t[k] & 0x04040404u) << 5);
                hl[(2 * gq) * 64] = hi[2 * gq]; hl[(2 * gq + 1) * 64] = hi[2 * gq + 1]; }
            for (u32 k0 = 0; k0 < nex; k0 += 64u) {
                if (k0 > 0u) asm volatile("global_load_ushort %0, %1, %2 sc0 sc1\n\ts_waitcnt vmcnt(0)" : "=&v"(ent) : "v"(vs4 + k0 * 2u), "s"(sp) : "memory");
                if (k0 + (u32)lane < nex) {
                    const u32 f = (ent >> 12) & 15u, q = (ent >> 2) & 15u;
                    LAS u32* w = (LAS u32*)(wlds + (q >> 2) * 1024u + ((ent >> 6) & 63u) * 16u + (q & 3u) * 4u);
                    __hip_atomic_fetch_sub(w, (f ? f * 4u : (u32)-4) << ((ent & 3u) * 8u), __ATOMIC_RELAXED, __HIP_MEMORY_SCOPE_WORKGROUP);
                }
            }
#pragma unroll
            for (int j = 0; j < 4; ++j) hi[j] = hl[j * 64];
        } else {
            asm volatile("global_load_dwordx4 %0, %8, %10 sc0 sc1\n\tglobal_load_dwordx4 %1, %8, %10 offset:1024 sc0 sc1\n\tglobal_load_dwordx4 %2, %8, %10 offset:2048 sc0 sc1\n\tglobal_load_dwordx4 %3, %8, %10 offset:3072 sc0 sc1\n\t"
                         "global_load_dwordx4 %4, %9, %10 sc0 sc1\n\tglobal_load_dwordx4 %5, %9, %10 offset:1024 sc0 sc1\n\tglobal_load_dwordx4 %6, %9, %10 offset:2048 sc0 sc1\n\tglobal_load_dwordx4 %7, %9, %10 offset:3072 sc0 sc1\n\ts_waitcnt vmcnt(0)"
                         : "=&v"(lo[0]), "=&v"(lo[1]), "=&v"(lo[2]), "=&v"(lo[3]), "=&v"(hi[0]), "=&v"(hi[1]), "=&v"(hi[2]), "=&v"(hi[3]) : "v"(vs), "v"(vs2), "s"(sp) : "memory");
        }
        const bool isw = st.isW();
        const char* ob; u32 v0, v1, js; gu32* ctr;
        if (isw) { ob = ws + OFF_W + (size_t)(st.kc * KC + 32 * (st.t - B0 / 16)) * 1024 + sv * 256; v0 = vw0; v1 = vw0 + 128; js = 8192; ctr = (gu32*)(ws + OFF_WRDY + sv * 128) + st.kc; }
        else { const int lr = st.lrow(sv); ob = ws + OFF_A + (size_t)lr * 8192 + st.kc * 512; v0 = vx0; v1 = vx0 + 64; js = 128; ctr = (gu32*)(ws + OFF_XRDY + (lr >> 6) * 128) + st.kc; }
#pragma unroll
        for (int j = 0; j < 4; ++j) {
            const u32x4 a = (u32x4){pksub(vperm(hi[j][0], lo[j][0], 0x05010400u), dl), pksub(vperm(hi[j][0], lo[j][0], 0x07030602u), dl), pksub(vperm(hi[j][1], lo[j][1], 0x05010400u), dl), pksub(vperm(hi[j][1], lo[j][1], 0x07030602u), dl)};
            const u32x4 b = (u32x4){pksub(vperm(hi[j][2], lo[j][2], 0x05010400u), dl), pksub(vperm(hi[j][2], lo[j][2], 0x07030602u), dl), pksub(vperm(hi[j][3], lo[j][3], 0x05010400u), dl), pksub(vperm(hi[j][3], lo[j][3], 0x07030602u), dl)};
            GSTORE_SYS(v0, a, ob, 0); GSTORE_SYS(v1, b, ob, 0); ob += js;
        }
        st.t += nsub;
        have = st.locate();
        { const GAS char* hn = (const GAS char*)(ws + OFF_HDR + ((size_t)par * NSLOT + (have ? st.slot(sv) : slot)) * 128);
          asm volatile("global_load_dwordx2 %0, %1, off sc0 sc1\n\ts_waitcnt vmcnt(0)" : "=&v"(hv) : "v"(hn) : "memory"); }
        if (lane == 0) __hip_atomic_fetch_add(ctr, 1u, __ATOMIC_RELAXED, __HIP_MEMORY_SCOPE_AGENT);
        ++n; if (n < 30) STAMP(1 + n);
    }
    STAMP(1);
}

__global__ __launch_bounds__(NTHREADS, 1) void ag_gemm_kernel(Params p) {
    extern __shared__ __attribute__((aligned(16))) char smem[];
    LAS char* lds = (LAS char*)smem;
    const int b = blockIdx.x, xcd = b & 7, slot = b >> 3, wave = uni(threadIdx.x >> 6), lane = threadIdx.x & 63;
    gu32* cnt = (gu32*)(p.ws[p.rank] + OFF_CNT + (size_t)b * 128) + wave;
    const int c = (slot - NGS) * 8 + xcd;
    if (c >= 0 && c < p.nsw && wave < p.wpw && !(p.mode & 2)) { send_role(p, lds + wave * 8192, cnt, wave * p.nsw + c, p.nsw * p.wpw); return; }
    const u32 call = (u32)uni((int)*cnt) + 1u;
    if (lane == 0) *cnt = call;
    if (c < 0) {
        if (p.mode & 1) return;
        const int ct = xcd & 3, hb = xcd >> 2;
        if (slot < NT0) gemm_role<4>(p, lds, call, 2 * (hb * (NT0 / 2) + (slot >> 1)), slot & 1, ct, true);
        else if (slot < NT0 + NS1O) gemm_role<4>(p, lds, call, R0 / 64 + 2 * (hb * NS1O + slot - NT0), 0, ct, false);
        else gemm_role<2>(p, lds, call, R0 / 64 + hb * NS1H + slot - NT0 - NS1O, 1, ct, false);
        return;
    }
    if (c >= p.nsw && c < p.nsw + p.nuw) { if (wave < p.upw && !(p.mode & 2)) unpack_role(p, lds + wave * 4096, call, wave * p.nuw + (c - p.nsw), p.nuw * p.upw); return; }
    if (c == NCOMM - 1 && wave == 0 && !(p.mode & 3)) {
        gu32* tmo = (gu32*)(p.ws[p.rank] + OFF_TMO);
        STAMP(0);
        if (!wait_ge((gu32*)(p.ws[p.rank] + OFF_DONE), call * (u32)NDONE, tmo, call)) {
            if (lane == 0) *(GAS uint16_t*)p.out[p.rank] = 0x7fc0;
        }
        STAMP(39);
    }
}
}

struct AggKnobs { int nsw = 63, wpw = 2, nuw = 32, upw = 8, nx = 17, stag = 0, mode = 0, n1 = 8; };
static AggKnobs g_knobs;

extern "C" void kernel_launch(int rank, int nranks, ncclComm_t comm, hipStream_t stream,
                              void* const* d_in, const int* in_sizes, int n_in, void* d_out, int out_size,
                              const int* in_dtypes, int out_dtype, void* const* d_out_all, void* const* d_ws_all, size_t ws_bytes) {
    using namespace agg;
    if (nranks != NRANK || (unsigned)rank >= (unsigned)NRANK || ws_bytes < WS_NEED || n_in != 2 || in_sizes[0] != SROWS * KDIM || in_sizes[1] != KDIM * SCOLS || out_size != 4096 * SCOLS) {
        fprintf(stderr, "ag_gemm: unexpected shapes\n"); return; }
    static bool init[NRANK] = {};
    if (!init[rank]) { (void)hipFuncSetAttribute((const void*)ag_gemm_kernel, hipFuncAttributeMaxDynamicSharedMemorySize, LDS_BYTES); init[rank] = true; }
    Params p{};
    p.x = (const uint16_t*)d_in[0]; p.w = (const uint16_t*)d_in[1];
    for (int i = 0; i < NRANK; ++i) { p.out[i] = (char*)d_out_all[i]; p.ws[i] = (char*)d_ws_all[i]; }
    p.rank = rank;
    p.nsw = g_knobs.nsw; p.wpw = g_knobs.wpw; p.nuw = g_knobs.nuw; p.upw = g_knobs.upw; p.nx = g_knobs.nx; p.stag = g_knobs.stag; p.mode = g_knobs.mode;
    { const int n1 = g_knobs.n1; unsigned m = 0; int j = 0;
      for (; j < 16 + n1; ++j) if (((j + 1) * n1) / (16 + n1) > (j * n1) / (16 + n1)) m |= 1u << j;
      for (; j < 32; ++j) m |= 1u << j;
      p.gmask = m; }
    hipLaunchKernelGGL(ag_gemm_kernel, dim3(NWG), dim3(NTHREADS), LDS_BYTES, stream, p);
}
```

```cpp
#include <hip/hip_runtime.h>
#include <rccl/rccl.h>
#include <cstdio>
#include <cstdint>
#include <type_traits>
#define DIRECT_HELPER 0
#define PACE0 2

namespace agg {
typedef unsigned u32;
typedef unsigned long long u64;
typedef u32 u32x4 __attribute__((ext_vector_type(4)));
typedef u32 u32x2 __attribute__((ext_vector_type(2)));
typedef float f32x4 __attribute__((ext_vector_type(4)));
typedef short s16x4 __attribute__((ext_vector_type(4)));
typedef short s16x8 __attribute__((ext_vector_type(8)));
typedef __bf16 bf16x8 __attribute__((ext_vector_type(8)));
#define LAS __attribute__((address_space(3)))
#define GAS __attribute__((address_space(1)))
typedef GAS u32 gu32;

constexpr int NRANK = 8, KDIM = 4096, SROWS = 512, SCOLS = 512;
constexpr int KC = 256, NKC = KDIM / KC;
#define P_R0 1024
#define P_A0 208
#define P_A1 128
constexpr int R0 = P_R0, R1 = 2048 - R0;
constexpr int A0 = P_A0, A1 = P_A1, B0 = R0 / 4 - A0, B1 = R1 / 4 - A1, NA = A0 + A1;
static_assert(A0 % 16 == 0 && A1 % 16 == 0 && B0 % 16 == 0 && B1 % 16 == 0 && B0 > 0 && B1 > 0 && B0 + B1 + NA == SROWS, "row split");
constexpr int NT0 = R0 / 128, NS1O = R1 / 256, NS1H = R1 / 128, NGS = NT0 + NS1O + NS1H;
static_assert(NT0 % 2 == 0 && R1 % 256 == 0 && NGS < 32, "tiles");
constexpr int NCOMM = (32 - NGS) * 8;
constexpr int NDONE = NT0 + R1 / 64;
constexpr int BK = 64, NST = KDIM / BK;
constexpr int LDS_CTL = 147456, LDS_BYTES = LDS_CTL + 64;
constexpr int NTHREADS = 704, NWG = 256;

constexpr size_t OFF_CNT = 0;
constexpr size_t OFF_TMO = 32768;
constexpr size_t OFF_DUMMY = OFF_TMO + 256;
constexpr size_t OFF_XRDY = 40960;
constexpr size_t OFF_WRDY = OFF_XRDY + 128 * 64;
constexpr size_t OFF_DONE = OFF_WRDY + 4 * 512;
constexpr size_t OFF_DBG = 65536;
constexpr size_t DBG_BYTES = 1048576 - OFF_DBG;
constexpr int NSLOT = 2048 + 512;
constexpr size_t OFF_HDR = 1048576;
constexpr size_t A_BYTES = (size_t)2048 * KDIM * 2, W_BYTES = (size_t)KDIM * 512 * 2;
constexpr size_t OFF_A = 2097152;
constexpr size_t OFF_W = OFF_A + A_BYTES;
constexpr size_t PK_BYTES = (size_t)NSLOT * 8192;
constexpr size_t OFF_PK = OFF_W + W_BYTES;
constexpr size_t WS_NEED = OFF_PK + 2 * PK_BYTES;
static_assert(OFF_DONE + 4 * NDONE * 32 <= OFF_DBG && (4 * NDONE * 8) % 64 == 0 && OFF_HDR + 2 * NSLOT * 128 <= OFF_A, "ctl");
static_assert(WS_NEED <= (size_t)65 * 1048576, "workspace");
constexpr u64 DEADLINE = 150000000ull;

struct Params {
    const uint16_t* x; const uint16_t* w;
    char* out[NRANK]; char* ws[NRANK];
    int rank;
    int nsw, wpw;
    int nuw, upw;
    int nx;
    int stag;
    int mode;
    unsigned gmask;
};

__device__ __forceinline__ u64 rtc() { return __builtin_amdgcn_s_memrealtime(); }
__device__ __forceinline__ int uni(int v) { return __builtin_amdgcn_readfirstlane(v); }

__device__ __forceinline__ void map_unit(int unit, int u, int& rank, int& r) {
    const int lr = unit * 16;
    if (lr < R0) { int i = lr; if (i < 4 * A0) { const int j = i / A0; r = i - j * A0; rank = 4 * u + j; }
                   else { i -= 4 * A0; const int j = i / B0; r = NA + (i - j * B0); rank = 4 * (1 - u) + j; } }
    else { int i = lr - R0; if (i < 4 * A1) { const int j = i / A1; r = A0 + i - j * A1; rank = 4 * u + j; }
           else { i -= 4 * A1; const int j = i / B1; r = NA + B0 + (i - j * B1); rank = 4 * (1 - u) + j; } }
}

#define STAMP(idx) do {} while (0)
#define STAMPC(idx) do {} while (0)

template <int NW> __device__ __forceinline__ bool wait_all(gu32* words, gu32* tmo, u32 call) {
    const u64 t0 = rtc();
    const int lane = threadIdx.x & 63;
    for (;;) {
        u32 v[NW];
#pragma unroll
        for (int j = 0; j < NW; ++j) v[j] = __hip_atomic_load(words + j * 64 + lane, __ATOMIC_RELAXED, __HIP_MEMORY_SCOPE_SYSTEM);
        const u32 t = __hip_atomic_load(tmo, __ATOMIC_RELAXED, __HIP_MEMORY_SCOPE_AGENT);
        bool all = true;
#pragma unroll
        for (int j = 0; j < NW; ++j) all = all && (int)(v[j] - call) >= 0;
        if (__all(all)) return true;
        if ((u32)uni((int)t) == call || rtc() - t0 > DEADLINE) {
            __hip_atomic_store(tmo, call, __ATOMIC_RELAXED, __HIP_MEMORY_SCOPE_AGENT);
            return false;
        }
    }
}

__device__ __forceinline__ void st16_sys(char* ptr, u32x4 v) {
    asm volatile("global_store_dwordx4 %0, %1, off sc0 sc1\n\ts_nop 1" :: "v"(ptr), "v"(v) : "memory");
}
__device__ __forceinline__ void st16_sys_s(u32 voff, u32x4 v, const char* sbase) {
    asm volatile("global_store_dwordx4 %0, %1, %2 sc0 sc1\n\ts_nop 1" :: "v"(voff), "v"(v), "s"(sbase) : "memory");
}

__device__ __forceinline__ float gelu1(float y) {
    const float k1 = -2.0f * 0.7978845608028654f * 1.4426950408889634f, k2 = k1 * 0.044715f;
    const float t = y * y, e = __builtin_amdgcn_exp2f(y * __builtin_fmaf(t, k2, k1));
    return y * __builtin_amdgcn_rcpf(1.0f + e);
}
typedef float f32x2 __attribute__((ext_vector_type(2)));
__device__ __forceinline__ f32x2 gelu2(f32x2 y) {
    const float k1 = -2.0f * 0.7978845608028654f * 1.4426950408889634f, k2 = k1 * 0.044715f;
    const f32x2 a = y * __builtin_elementwise_fma(y * y, (f32x2){k2, k2}, (f32x2){k1, k1});
    const f32x2 d = (f32x2){__builtin_amdgcn_exp2f(a[0]), __builtin_amdgcn_exp2f(a[1])} + (f32x2){1.0f, 1.0f};
    return y * (f32x2){__builtin_amdgcn_rcpf(d[0]), __builtin_amdgcn_rcpf(d[1])};
}
__device__ __forceinline__ u32 cvt_pk(float lo, float hi) { u32 r; asm("v_cvt_pk_bf16_f32 %0, %1, %2" : "=v"(r) : "v"(lo), "v"(hi)); return r; }

#define BAR() __builtin_amdgcn_s_barrier()
#define WAITV(n) asm volatile("s_waitcnt vmcnt(" #n ")" ::: "memory")
#define WAITL0() asm volatile("s_waitcnt lgkmcnt(0)" ::: "memory")
#define SB() __builtin_amdgcn_sched_barrier(0)
typedef float f32x16 __attribute__((ext_vector_type(16)));
__device__ __forceinline__ const char* uniptr(const char* q) { const u64 a = (u64)q; return (const char*)(((u64)(u32)uni((int)(u32)(a >> 32)) << 32) | (u32)uni((int)(u32)a)); }
template <int N> __device__ __forceinline__ void waitv() { asm volatile("s_waitcnt vmcnt(%0)" :: "n"(N) : "memory"); }
__device__ __forceinline__ void glds2(u32 v0, u32 v1, const char* sb, u32 m0v) {
    asm volatile("s_mov_b32 m0, %3\n\ts_nop 0\n\tglobal_load_lds_dwordx4 %0, %2 sc0 sc1\n\tglobal_load_lds_dwordx4 %1, %2 offset:1024 sc0 sc1" :: "v"(v0), "v"(v1), "s"(sb), "s"(m0v) : "memory");
}
__device__ __forceinline__ void glds4(u32 v0, u32 v1, u32 v2, u32 v3, const char* sb, u32 m0v) {
    asm volatile("s_mov_b32 m0, %5\n\ts_nop 0\n\tglobal_load_lds_dwordx4 %0, %4 sc0 sc1\n\tglobal_load_lds_dwordx4 %1, %4 offset:1024 sc0 sc1\n\t"
                 "global_load_lds_dwordx4 %2, %4 offset:2048 sc0 sc1\n\tglobal_load_lds_dwordx4 %3, %4 offset:3072 sc0 sc1" :: "v"(v0), "v"(v1), "v"(v2), "v"(v3), "s"(sb), "s"(m0v) : "memory");
}
template <int OFF> __device__ __forceinline__ void rdx(bf16x8& d, u32 a) { asm volatile("ds_read_b128 %0, %1 offset:%c2" : "=v"(d) : "v"(a), "i"(OFF) : "memory"); }
template <int OFF> __device__ __forceinline__ void rdw(s16x4& d0, s16x4& d1, u32 a0, u32 a1) {
    asm volatile("ds_read_b64_tr_b16 %0, %2 offset:%c4\n\tds_read_b64_tr_b16 %1, %3 offset:%c4" : "=&v"(d0), "=&v"(d1) : "v"(a0), "v"(a1), "i"(OFF) : "memory"); }
template <int NI> struct Frags { s16x4 w[4][2]; bf16x8 x[4][NI]; };

template <int NF>
__device__ __forceinline__ void gemm_role(const Params& p, LAS char* lds, u32 call, int rt64, int helper, int ct, bool early) {
    constexpr int TM = 32 * NF, NBUF = NF == 4 ? 4 : 6, ASTG = TM * 128, LB = NBUF * ASTG;
    constexpr int NUN = TM / 16;
    constexpr int PPS = NF == 4 ? 16 : 12;
    constexpr int NPRE = NBUF - 1 < 4 ? NBUF - 1 : 4;
    const int g = p.rank, u = g >> 2, v = g & 3;
    const int wave = uni(threadIdx.x >> 6), lane = threadIdx.x & 63;
    char* const ws = p.ws[g];
    gu32* tmo = (gu32*)(ws + OFF_TMO);
    LAS u32* ctl = (LAS u32*)(lds + LDS_CTL);
    const u32 ldsb = (u32)(size_t)lds;
    if (wave == 10) {
        const int dk = lane & 3;
        const bool xl = lane < 4 * NUN, wl = helper && lane >= 32;
        bool mine = false;
        if (xl) { int rk, r; map_unit(rt64 * 4 + (lane >> 2), u, rk, r); mine = rk == g; }
        const bool watch = (xl && !mine) || wl;
        const char* wbase = wl ? ws + OFF_WRDY + ct * 512 + ((lane - 32) >> 2) * 4 : ws + OFF_XRDY + (size_t)(rt64 * 4 + (xl ? lane >> 2 : 0)) * 64;
        const int wstep = wl ? 32 : 4;
        bool ok = true; int ready = (p.mode & 2) ? 16 : 0;
        auto need = [&](int c) {
            if (ready > c) return;
            const u64 t0 = rtc();
            const int kc = min(c + dk, 15);
            gu32* word = (gu32*)(wbase + kc * wstep);
            for (int it = 0;; ++it) {
                const u32 val = __hip_atomic_load(word, __ATOMIC_RELAXED, __HIP_MEMORY_SCOPE_SYSTEM);
                const u64 miss = __ballot(watch && (int)(val - call) < 0);
                const u64 m4 = 0x1111111111111111ull;
                ready = (miss & m4) ? c : (miss & (m4 << 1)) ? c + 1 : (miss & (m4 << 2)) ? c + 2 : (miss & (m4 << 3)) ? c + 3 : c + 4;
                if (ready > c) return;
                if ((it & 7) == 7 && ((u32)uni((int)__hip_atomic_load(tmo, __ATOMIC_RELAXED, __HIP_MEMORY_SCOPE_AGENT)) == call || rtc() - t0 > DEADLINE)) {
                    __hip_atomic_store(tmo, call, __ATOMIC_RELAXED, __HIP_MEMORY_SCOPE_AGENT); ok = false; ready = 16; return; }
                __builtin_amdgcn_s_sleep(1);
            }
        };
        STAMP(0);
        need(0);
        STAMP(1);
        BAR();
        if (NBUF > 4) need(1);
        BAR();
        for (int s = 0; s < NST - 1; ++s) {
            if (((s + NBUF) & 3) == 0 && s + NBUF < NST) { STAMP(20 + ((s + NBUF) >> 2)); need((s + NBUF) >> 2); STAMP(1 + ((s + NBUF) >> 2)); }
            if (s == NST - NBUF - 2) {
                const u32 t = (u32)uni((int)__hip_atomic_load(tmo, __ATOMIC_RELAXED, __HIP_MEMORY_SCOPE_AGENT));
                if (lane == 0) ctl[0] = (!ok || t == call) ? 1u : 0u;
                WAITL0();
            }
            BAR();
        }
        if (helper && !DIRECT_HELPER) BAR();
        return;
    }
    if (wave >= 8) {
        const bool l0 = wave == 8;
        u32 va[2], vb[4]; const char* ub[NUN];
#pragma unroll
        for (int e = 0; e < 2; ++e) { const int ar = 8 * e + (lane >> 3); va[e] = (u32)ar * 8192u + (u32)(((lane & 7) ^ ((ar >> 1) & 7)) * 16) - (u32)e * 1024u; }
#pragma unroll
        for (int i = 0; i < NUN; ++i) { int rk, r; map_unit(rt64 * 4 + i, u, rk, r);
            ub[i] = uniptr((rk == g) ? (const char*)p.x + (size_t)r * 8192 : ws + OFF_A + (size_t)(rt64 * 64 + i * 16) * 8192); }
#pragma unroll
        for (int e = 0; e < 4; ++e) { const int kr = 4 * e + (lane >> 4), sw = ((kr & 3) << 2) | ((kr >> 2) & 3); vb[e] = (u32)kr * 1024u + (u32)(((lane & 15) ^ sw) * 16) - (u32)e * 1024u; }
        const char* b0 = uniptr(helper ? ws + OFF_W + ct * 256 : (const char*)p.w + ct * 256);
        auto issue = [&](int s, int buf) {
            const u32 la = ldsb + (u32)buf * ASTG, lb = ldsb + LB + (u32)buf * 16384u;
            const char* b = b0 + (size_t)s * 65536;
            if (l0) {
                const u32 v0 = va[0] + (u32)s * 128u, v1 = va[1] + (u32)s * 128u;
#pragma unroll
                for (int i = 0; i < NUN; ++i) glds2(v0, v1, ub[i], la + i * 2048);
                if (NF == 2) glds4(vb[0], vb[1], vb[2], vb[3], b, lb);
            } else {
#pragma unroll
                for (int q = (NF == 2 ? 1 : 0); q < 4; ++q) glds4(vb[0], vb[1], vb[2], vb[3], b + q * 16384, lb + q * 4096);
            }
        };
        BAR();
        for (int s = 0; s < NPRE; ++s) issue(s, s);
        waitv<(NPRE - 1) * PPS>(); BAR();
        for (int s = NPRE; s < NBUF; ++s) issue(s, s);
        int buf = 0;
        for (int s = 0; s < NST - NBUF; ++s) { waitv<(NBUF - 2) * PPS>(); BAR(); issue(s + NBUF, buf); buf = buf + 1 == NBUF ? 0 : buf + 1; }
        waitv<(NBUF - 2) * PPS>(); BAR();
        waitv<(NBUF - 3) * PPS>(); BAR();
        waitv<(NBUF - 4) * PPS>(); BAR();
        if (NBUF == 6) { waitv<PPS>(); BAR(); waitv<0>(); BAR(); }
        if (helper && !DIRECT_HELPER) BAR();
        return;
    }
    constexpr int NI = NF / 2;
    const int wm = wave >> 2, wn = wave & 3;
    const int l31 = lane & 31, lh = lane >> 5;
    u32 xa[4], wa[2][2];
#pragma unroll
    for (int ks = 0; ks < 4; ++ks) xa[ks] = ldsb + (u32)((wm * (TM / 2) + l31) * 128 + (((2 * ks + lh) ^ ((l31 >> 1) & 7)) * 16));
    { const int blk = (lane >> 4) & 1, tq = (lane & 15) >> 2, tp = lane & 3;
#pragma unroll
      for (int t = 0; t < 2; ++t) {
          const int kr = 8 * lh + 4 * t + tq;
          const int sw = ((kr & 3) << 2) | ((kr >> 2) & 3);
          const int ch = 4 * wn + 2 * blk + (tp >> 1);
          wa[0][t] = ldsb + (u32)(LB + kr * 256 + ((ch ^ sw) * 16) + 8 * (tp & 1));
          wa[1][t] = wa[0][t] + 3 * 16384;
      } }
    f32x16 acc[NI];
#pragma unroll
    for (int i = 0; i < NI; ++i)
#pragma unroll
        for (int e = 0; e < 16; ++e) acc[i][e] = 0.f;
#define RDW(f, BUF, ks) rdw<((BUF) % 3) * 16384 * (NBUF == 6) + (BUF) * 16384 * (NBUF == 4) + (ks) * 4096>(f.w[ks][0], f.w[ks][1], wa[NBUF == 6 ? (BUF) / 3 : 0][0], wa[NBUF == 6 ? (BUF) / 3 : 0][1])
#define RDX(f, BUF, ks, i) rdx<(BUF) * ASTG + (i) * 4096>(f.x[ks][i], xa[ks])
#define MF(f, ks, i) do { acc[i] = __builtin_amdgcn_mfma_f32_32x32x16_bf16(__builtin_bit_cast(bf16x8, __builtin_shufflevector(f.w[ks][0], f.w[ks][1], 0, 1, 2, 3, 4, 5, 6, 7)), f.x[ks][i], acc[i], 0, 0, 0); SB(); } while (0)
#define LANDED(f) asm volatile("s_waitcnt lgkmcnt(0)" : "+v"(f.w[0][0]), "+v"(f.w[0][1]), "+v"(f.w[1][0]), "+v"(f.w[1][1]), "+v"(f.w[2][0]), "+v"(f.w[2][1]), "+v"(f.w[3][0]), "+v"(f.w[3][1]), \
                     "+v"(f.x[0][0]), "+v"(f.x[1][0]), "+v"(f.x[2][0]), "+v"(f.x[3][0]), "+v"(f.x[0][NI - 1]), "+v"(f.x[1][NI - 1]), "+v"(f.x[2][NI - 1]), "+v"(f.x[3][NI - 1]) :: "memory")
#define LANDED1(f) asm volatile("s_waitcnt lgkmcnt(0)" : "+v"(f.w[0][0]), "+v"(f.w[0][1]), "+v"(f.w[1][0]), "+v"(f.w[1][1]), "+v"(f.w[2][0]), "+v"(f.w[2][1]), "+v"(f.w[3][0]), "+v"(f.w[3][1]), \
                     "+v"(f.x[0][0]), "+v"(f.x[1][0]), "+v"(f.x[2][0]), "+v"(f.x[3][0]) :: "memory")
#define WAITF(f) do { if constexpr (NI == 2) { LANDED(f); } else { LANDED1(f); } } while (0)
#define RDK(f, NB, ks) do { RDW(f, NB, ks); RDX(f, NB, ks, 0); if constexpr (NI == 2) { RDX(f, NB, ks, NI - 1); } } while (0)
#define MFK(f, ks) do { MF(f, ks, 0); if constexpr (NI == 2) { MF(f, ks, NI - 1); } } while (0)
#define STEP(cur, nxt, NB) do { BAR(); SB(); \
        if constexpr (NI == 2) { \
            MF(cur, 0, 0); RDW(nxt, NB, 0); RDX(nxt, NB, 0, 0); SB(); \
            MF(cur, 0, NI - 1); RDX(nxt, NB, 0, NI - 1); RDW(nxt, NB, 1); SB(); \
            MF(cur, 1, 0); RDX(nxt, NB, 1, 0); RDX(nxt, NB, 1, NI - 1); SB(); \
            MF(cur, 1, NI - 1); RDW(nxt, NB, 2); RDX(nxt, NB, 2, 0); SB(); \
            MF(cur, 2, 0); RDX(nxt, NB, 2, NI - 1); RDW(nxt, NB, 3); SB(); \
            MF(cur, 2, NI - 1); RDX(nxt, NB, 3, 0); RDX(nxt, NB, 3, NI - 1); SB(); \
            MF(cur, 3, 0); MF(cur, 3, NI - 1); \
        } else { \
            MF(cur, 0, 0); RDK(nxt, NB, 0); RDW(nxt, NB, 1); SB(); \
            MF(cur, 1, 0); RDX(nxt, NB, 1, 0); RDK(nxt, NB, 2); SB(); \
            MF(cur, 2, 0); RDK(nxt, NB, 3); SB(); \
            MF(cur, 3, 0); \
        } \
        WAITF(nxt); } while (0)
    Frags<NI> P, Q;
    BAR();
    STAMP(0); STAMPC(30);
    BAR();
    RDK(P, 0, 0); RDK(P, 0, 1); RDK(P, 0, 2); RDK(P, 0, 3); WAITF(P);
    for (int s = 0; s < 60; s += NBUF) {
        STEP(P, Q, 1); STEP(Q, P, 2); STEP(P, Q, 3);
        if constexpr (NBUF == 4) { STEP(Q, P, 0); }
        else { STEP(Q, P, 4 % NBUF); STEP(P, Q, 5 % NBUF); STEP(Q, P, 0); }
    }
    STEP(P, Q, 1); STEP(Q, P, 2); STEP(P, Q, 3);
    MFK(Q, 0); MFK(Q, 1); MFK(Q, 2); MFK(Q, 3);
    STAMP(1); STAMPC(31);
    const bool bad = ctl[0] != 0u;
    u32x2 pk[NI][4];
#pragma unroll
    for (int i = 0; i < NI; ++i) { const f32x16& a = acc[i];
#pragma unroll
        for (int e = 0; e < 4; ++e) { const f32x2 y0 = gelu2((f32x2){a[4 * e], a[4 * e + 1]}), y1 = gelu2((f32x2){a[4 * e + 2], a[4 * e + 3]});
            pk[i][e] = (u32x2){cvt_pk(y0[0], y0[1]), cvt_pk(y1[0], y1[1])}; } }
    char* const ocol = helper ? p.out[4 * (1 - u) + ct] + v * 256 : p.out[g] + ct * 256;
    if (!helper || DIRECT_HELPER) {
#pragma unroll
        for (int i = 0; i < NI; ++i) {
            const int row0 = wm * (TM / 2) + 32 * i;
            int rk0, r0, rk1, r1; map_unit(rt64 * 4 + (row0 >> 4), u, rk0, r0); map_unit(rt64 * 4 + (row0 >> 4) + 1, u, rk1, r1);
            const int grow = (l31 < 16 ? rk0 * SROWS + r0 : rk1 * SROWS + r1) + (l31 & 15);
            char* dst = ocol + (size_t)grow * 1024 + wn * 64 + lh * 16;
#pragma unroll
            for (int e = 0; e < 4; e += 2) {
                auto s0 = __builtin_amdgcn_permlane32_swap(pk[i][e][0], pk[i][e + 1][0], false, false);
                auto s1 = __builtin_amdgcn_permlane32_swap(pk[i][e][1], pk[i][e + 1][1], false, false);
                u32x4 ov = (u32x4){s0[0], s1[0], s0[1], s1[1]};
                if (bad) ov = (u32x4){0x7fc07fc0u, 0x7fc07fc0u, 0x7fc07fc0u, 0x7fc07fc0u};
                if (helper) { st16_sys(dst + e * 16, ov); if (early && ((i * 2 + (e >> 1) + 1) % PACE0 == 0)) WAITV(0); }
                else *(GAS u32x4*)(dst + e * 16) = ov;
            }
        }
    } else {
        LAS char* otile = lds + LB + (NF == 4 ? 0 : 4 * 16384);
#pragma unroll
        for (int i = 0; i < NI; ++i) { const int row = wm * (TM / 2) + 32 * i + l31;
#pragma unroll
            for (int e = 0; e < 4; ++e) *(LAS u32x2*)(otile + row * 256 + (((4 * wn + e) ^ (row & 15)) * 16) + lh * 8) = pk[i][e]; }
        WAITL0(); BAR();
        constexpr int RPW = TM / 8, NS = RPW / 4;
        const int row0 = RPW * wave;
        int urk, urow; map_unit(rt64 * 4 + (row0 >> 4), u, urk, urow);
        char* obase = ocol + (size_t)(urk * SROWS + urow + (row0 & 15)) * 1024;
        u32x4 ov[NS];
#pragma unroll
        for (int it = 0; it < NS; ++it) {
            const int row = row0 + 4 * it + (lane >> 4), ch = lane & 15;
            ov[it] = *(LAS u32x4*)(otile + row * 256 + ((ch ^ (row & 15)) * 16));
            if (bad) ov[it] = (u32x4){0x7fc07fc0u, 0x7fc07fc0u, 0x7fc07fc0u, 0x7fc07fc0u};
        }
#pragma unroll
        for (int it = 0; it < NS; ++it) {
            st16_sys(obase + (size_t)(4 * it + (lane >> 4)) * 1024 + (lane & 15) * 16, ov[it]);
            if (early && (it + 1) % PACE0 == 0) WAITV(0);
        }
    }
    if (helper) {
        WAITV(0);
        const int tile = early ? rt64 / 2 : NT0 + rt64 - R0 / 64;
        if (lane == 0) __hip_atomic_store((gu32*)(p.ws[4 * (1 - u) + ct] + OFF_DONE) + (v * NDONE + tile) * 8 + wave, call, __ATOMIC_RELAXED, __HIP_MEMORY_SCOPE_SYSTEM);
    }
    STAMP(2);
}

struct Stream {
    u32 gmask; int step, t, G, kc; bool cross; int c1;
    __device__ __forceinline__ bool locate() {
#pragma unroll 1
        for (;;) { if (step >= 32) return false;
            G = (gmask >> step) & 1; kc = G ? c1 : step - c1;
            const int cn = cross ? (G ? B1 / 16 : B0 / 16 + 8) : (G ? A1 / 16 : A0 / 16); if (t < cn) return true; t -= cn; c1 += G; ++step; } }
    __device__ __forceinline__ bool isW() const { return cross && G == 0 && t >= B0 / 16; }
    __device__ __forceinline__ int lrow(int sv) const { return (G ? R0 : 0) + (cross ? 4 * (G ? A1 : A0) + sv * (G ? B1 : B0) : sv * (G ? A1 : A0)) + 16 * t; }
    __device__ __forceinline__ int slot(int sv) const { return isW() ? 2048 + sv * 128 + kc * 8 + (t - B0 / 16) : (lrow(sv) >> 4) * 16 + kc; }
};

__device__ __forceinline__ void stagger(int ticks16) { if (ticks16 > 0) { const u64 t = rtc() + (u64)(ticks16 >> 4); while (rtc() < t) __builtin_amdgcn_s_sleep(1); } }
__device__ __forceinline__ u32 vperm(u32 s0, u32 s1, u32 sel) { return __builtin_amdgcn_perm(s0, s1, sel); }
__device__ __forceinline__ u32 pkmax(u32 a, u32 b) { u32 r; asm("v_pk_max_u16 %0, %1, %2" : "=v"(r) : "v"(a), "v"(b)); return r; }
__device__ __forceinline__ u32 pkadd(u32 a, u32 b) { u32 r; asm("v_pk_add_u16 %0, %1, %2" : "=v"(r) : "v"(a), "s"(b)); return r; }
__device__ __forceinline__ u32 pksub(u32 a, u32 b) { u32 r; asm("v_pk_sub_u16 %0, %1, %2" : "=v"(r) : "v"(a), "s"(b)); return r; }
__device__ __forceinline__ u32 wave_scan_incl(u32 x) {
    x += (u32)__builtin_amdgcn_update_dpp(0, (int)x, 0x111, 0xf, 0xf, true);
    x += (u32)__builtin_amdgcn_update_dpp(0, (int)x, 0x112, 0xf, 0xf, true);
    x += (u32)__builtin_amdgcn_update_dpp(0, (int)x, 0x114, 0xf, 0xf, true);
    x += (u32)__builtin_amdgcn_update_dpp(0, (int)x, 0x118, 0xf, 0xf, true);
    x += (u32)__builtin_amdgcn_update_dpp(0, (int)x, 0x142, 0xa, 0xf, true);
    x += (u32)__builtin_amdgcn_update_dpp(0, (int)x, 0x143, 0xc, 0xf, true);
    return x;
}
__device__ __forceinline__ u32 wave_max(u32 x) {
    x = max(x, (u32)__builtin_amdgcn_update_dpp(0, (int)x, 0x111, 0xf, 0xf, true));
    x = max(x, (u32)__builtin_amdgcn_update_dpp(0, (int)x, 0x112, 0xf, 0xf, true));
    x = max(x, (u32)__builtin_amdgcn_update_dpp(0, (int)x, 0x114, 0xf, 0xf, true));
    x = max(x, (u32)__builtin_amdgcn_update_dpp(0, (int)x, 0x118, 0xf, 0xf, true));
    const u32 a = (u32)__builtin_amdgcn_readlane((int)x, 15), b = (u32)__builtin_amdgcn_readlane((int)x, 31), c = (u32)__builtin_amdgcn_readlane((int)x, 47), d = (u32)__builtin_amdgcn_readlane((int)x, 63);
    return max(max(a, b), max(c, d));
}
struct Raw { u32x4 d[8]; };
#define GLOAD(dst, voff, sbase) asm volatile("global_load_dwordx4 %0, %1, %2" : "=v"(dst) : "v"(voff), "s"(sbase) : "memory")
#define GLOAD_SYS(dst, voff, sbase, off) asm volatile("global_load_dwordx4 %0, %1, %2 offset:" #off " sc0 sc1" : "=v"(dst) : "v"(voff), "s"(sbase) : "memory")
#define PIN8(R) "+v"(R.d[0]), "+v"(R.d[1]), "+v"(R.d[2]), "+v"(R.d[3]), "+v"(R.d[4]), "+v"(R.d[5]), "+v"(R.d[6]), "+v"(R.d[7])
#define GSTORE_SYS(voff, val, sbase, off) asm volatile("global_store_dwordx4 %0, %1, %2 offset:" #off " sc0 sc1\n\ts_nop 1" :: "v"(voff), "v"(val), "s"(sbase) : "memory")

__device__ __forceinline__ void send_role(const Params& p, LAS char* wlds, gu32* cnt, int id, int nw) {
    const int g = p.rank, u = g >> 2, v = g & 3, lane = threadIdx.x & 63;
    const int nx = p.nx, ni = (nw - 4 * nx) / 3 < nx ? nx : (nw - 4 * nx) / 3;
    int di, sub;
    if (id < 7 * nx) { di = id % 7; sub = id / 7; } else { di = (id - 7 * nx) % 3; sub = nx + (id - 7 * nx) / 3; }
    const int nsub = di >= 3 ? nx : ni;
    if (sub >= nsub) { const u32 c = *cnt; if (lane == 0) *cnt = c + 1; return; }
    const bool cross = di >= 3;
    const int dest = cross ? 4 * (1 - u) + (di - 3) : 4 * u + ((v + 1 + di) & 3);
    const u32 vx0 = (u32)(lane >> 2) * 8192u + (u32)(lane & 3) * 16u, vw0 = (u32)(lane >> 3) * 1024u + (u32)(lane & 7) * 16u;
    Stream ld{p.gmask, 0, sub, 0, 0, cross, 0};
    auto load = [&](Raw& R) -> int {
        const bool have = ld.locate();
        const char* sb; u32 v0, v1, js; int slot = -1;
        if (have && ld.isW()) { sb = (const char*)p.w + (size_t)(ld.kc * KC + 32 * (ld.t - B0 / 16)) * 1024 + (di - 3) * 256; v0 = vw0; v1 = vw0 + 128; js = 8192; }
        else { const int r0 = have ? (cross ? NA + (ld.G ? B0 : 0) : (ld.G ? A0 : 0)) + 16 * ld.t : 0; sb = (const char*)p.x + (size_t)r0 * 8192 + (have ? ld.kc : 0) * 512; v0 = vx0; v1 = vx0 + 64; js = 128; }
        if (have) { slot = ld.slot(v); ld.t += nsub; }
        GLOAD(R.d[0], v0, sb); GLOAD(R.d[1], v1, sb); sb += js; GLOAD(R.d[2], v0, sb); GLOAD(R.d[3], v1, sb); sb += js;
        GLOAD(R.d[4], v0, sb); GLOAD(R.d[5], v1, sb); sb += js; GLOAD(R.d[6], v0, sb); GLOAD(R.d[7], v1, sb);
        return slot;
    };
    Raw RA, RB;
    int slotA = load(RA), slotB = load(RB);
    const u32 call = (u32)uni((int)*cnt) + 1u;
    if (lane == 0) *cnt = call;
    asm volatile("s_waitcnt vmcnt(0)" : PIN8(RA) :: "memory");
    const int par = call & 1;
    const char* dP = p.ws[dest] + OFF_PK + (size_t)par * PK_BYTES;
    char* dH = p.ws[dest] + OFF_HDR + (size_t)par * NSLOT * 128;
    char* prevH = p.ws[g] + OFF_DUMMY; u32 prevInfo = 0;
    const u32 vs = (u32)lane * 16u, vs2 = vs + 4096u, vs3 = (u32)lane * 8u + 5120u, vs4 = vs + 5632u;
    LAS uint16_t* elist = (LAS uint16_t*)wlds;
    stagger(sub * p.stag);
    STAMP(0);
    int n = 0;
    auto iter = [&](Raw& R, int& slot, bool first) {
        if (!first) asm volatile("s_waitcnt vmcnt(16)" : PIN8(R) :: "memory");
        u32 m = 0;
#pragma unroll
        for (int e = 0; e < 4; ++e) m = pkmax(m, R.d[0][e] & 0x7fff7fffu);
        const bool quick = first && (sub & 1);
        const u32 emax = quick ? 257u : wave_max(max(m & 0xffffu, m >> 16)) >> 7;
        const u32 delta = ~emax & 1u, top = (emax + delta) >> 1, low = top > 3u ? top - 3u : 0u, Lx = low * 0x01010101u, dl = delta * 0x00800080u;
        u32x4 lo[4], hi[4], cu = {}; u32 sg[2] = {}; u32 mlo = 0, mhi = 0;
        LAS u32x4* xs = (LAS u32x4*)(wlds + 1024) + lane;
#pragma unroll
        for (int j = 0; j < 4; ++j) {
            u32x4 a = R.d[2 * j], b = R.d[2 * j + 1], xv;
#pragma unroll
            for (int e = 0; e < 4; ++e) { a[e] = pkadd(a[e], dl); b[e] = pkadd(b[e], dl); }
            lo[j] = (u32x4){vperm(a[1], a[0], 0x06040200u), vperm(a[3], a[2], 0x06040200u), vperm(b[1], b[0], 0x06040200u), vperm(b[3], b[2], 0x06040200u)};
            hi[j] = (u32x4){vperm(a[1], a[0], 0x07050301u), vperm(a[3], a[2], 0x07050301u), vperm(b[1], b[0], 0x07050301u), vperm(b[3], b[2], 0x07050301u)};
            if (quick) continue;
            u32 z[4];
#pragma unroll
            for (int e = 0; e < 4; ++e) {
                const u32 h = hi[j][e], x = ((h & 0x7f7f7f7fu) | 0x80808080u) - Lx;
                xv[e] = x;
                z[e] = (((x & 0x7c7c7c7cu) + 0x7c7c7c7cu) | ~x) & 0x80808080u;
                const int k = 4 * (j & 1) + e;
                sg[j >> 1] |= (h >> (7 - k)) & (0x01010101u << k);
            }
            cu[j] = (xv[0] & 0x03030303u) | ((xv[1] & 0x03030303u) << 2) | ((xv[2] & 0x03030303u) << 4) | ((xv[3] << 6) & 0xc0c0c0c0u);
            xs[j * 64] = xv;
            const u32 m0 = __builtin_amdgcn_udot4(z[1], 0x80402010u, __builtin_amdgcn_udot4(z[0], 0x08040201u, 0u, false), false);
            const u32 m1 = __builtin_amdgcn_udot4(z[3], 0x80402010u, __builtin_amdgcn_udot4(z[2], 0x08040201u, 0u, false), false);
            const u32 m16 = (m0 >> 7) | (m1 << 1);
            if (j < 2) mlo |= m16 << (16 * j); else mhi |= m16 << (16 * (j - 2));
        }
        u32 idx = wave_scan_incl((u32)__builtin_popcount(mlo) + (u32)__builtin_popcount(mhi));
        const u32 total = (u32)__builtin_amdgcn_readlane((int)idx, 63);
        idx -= (u32)__builtin_popcount(mlo) + (u32)__builtin_popcount(mhi);
        bool bad = false;
        u64 mm = quick ? 0ull : ((u64)mhi << 32) | mlo;
        while (__ballot(mm != 0ull) != 0ull) {
            u32 ps[4], xb[4]; bool act[4];
#pragma unroll
            for (int k = 0; k < 4; ++k) {
                act[k] = mm != 0ull; ps[k] = act[k] ? (u32)__builtin_ctzll(mm) : 0u; mm &= mm - 1ull;
                xb[k] = *(LAS uint8_t*)(wlds + 1024 + (ps[k] >> 4) * 1024u + (u32)lane * 16u + (ps[k] & 15u));
            }
#pragma unroll
            for (int k = 0; k < 4; ++k) {
                const int d4 = 32 - (int)(xb[k] >> 2);
                bad |= act[k] && (d4 > 15 || d4 < -1);
                elist[(act[k] && idx < 256u) ? idx : 256u + (u32)lane] = (uint16_t)((u32)lane * 64u + ps[k] + ((u32)max(d4, 0) << 12));
                idx += act[k] ? 1u : 0u;
            }
        }
        const bool rawmode = quick || __ballot(bad) != 0ull || total > 256u;
        const u32 info = low | (rawmode ? 128u : 0u) | (delta << 8) | ((rawmode ? 0u : total) << 9);
        const char* sp = dP + (size_t)slot * 8192;
        GSTORE_SYS(vs, lo[0], sp, 0); GSTORE_SYS(vs, lo[1], sp, 1024); GSTORE_SYS(vs, lo[2], sp, 2048); GSTORE_SYS(vs, lo[3], sp, 3072);
        if (!rawmode) {
            const u32x4 c1 = cu; const u32x2 c2 = (u32x2){sg[0], sg[1]};
            GSTORE_SYS(vs2, c1, sp, 0);
            asm volatile("global_store_dwordx2 %0, %1, %2 sc0 sc1\n\ts_nop 1" :: "v"(vs3), "v"(c2), "s"(sp) : "memory");
            if ((u32)lane * 8u < max(total, 1u)) { const u32x4 le = *(LAS u32x4*)(wlds + lane * 16); GSTORE_SYS(vs4, le, sp, 0); }
        } else { GSTORE_SYS(vs2, hi[0], sp, 0); GSTORE_SYS(vs2, hi[1], sp, 1024); GSTORE_SYS(vs2, hi[2], sp, 2048); GSTORE_SYS(vs2, hi[3], sp, 3072); }
        asm volatile("s_waitcnt vmcnt(16)" ::: "memory");
        { const u64 hv = ((u64)call << 32) | prevInfo;
          if (lane == 0) asm volatile("global_store_dwordx2 %0, %1, off sc0 sc1\n\ts_nop 1" :: "v"(prevH), "v"(hv) : "memory"); }
        prevH = dH + (size_t)slot * 128; prevInfo = info;
        slot = load(R);
        ++n; if (n < 30) STAMP(1 + n);
    };
    if (slotA >= 0) {
        iter(RA, slotA, true);
        for (;;) {
            if (slotB < 0) break;
            iter(RB, slotB, false);
            if (slotA < 0) break;
            iter(RA, slotA, false);
        }
        asm volatile("s_waitcnt vmcnt(0)" : PIN8(RA), PIN8(RB) :: "memory");
        const u64 hv = ((u64)call << 32) | prevInfo;
        if (lane == 0) asm volatile("global_store_dwordx2 %0, %1, off sc0 sc1\n\ts_nop 1" :: "v"(prevH), "v"(hv) : "memory");
    } else asm volatile("s_waitcnt vmcnt(0)" : PIN8(RA), PIN8(RB) :: "memory");
    STAMP(1);
}

__device__ __forceinline__ void unpack_role(const Params& p, LAS char* wlds, u32 call, int id, int nw) {
    const int g = p.rank, u = g >> 2, v = g & 3, lane = threadIdx.x & 63, par = call & 1;
    const int si = id % 7, nsub = nw / 7;
    if (id / 7 >= nsub) return;
    const bool cross = si >= 3;
    const int sv = cross ? si - 3 : (v + 1 + si) & 3;
    char* const ws = p.ws[g];
    gu32* tmo = (gu32*)(ws + OFF_TMO);
    const u32 vs = (u32)lane * 16u, vs2 = vs + 4096u, vs3 = (u32)lane * 8u + 5120u, vs4 = (u32)lane * 2u + 5632u;
    const u32 vx0 = (u32)(lane >> 2) * 8192u + (u32)(lane & 3) * 16u, vw0 = (u32)(lane >> 3) * 1024u + (u32)(lane & 7) * 16u;
    Stream st{p.gmask, 0, id / 7, 0, 0, cross, 0};
    STAMP(0);
    int n = 0;
    bool have = st.locate();
    u64 hv = 0;
    while (have) {
        const int slot = st.slot(sv);
        const GAS char* hp = (const GAS char*)(ws + OFF_HDR + ((size_t)par * NSLOT + slot) * 128);
        u32 info = 0; bool ok = true;
        { const u64 t0 = rtc();
          for (;;) {
              const u32 seq = (u32)uni((int)(u32)(hv >> 32)); info = (u32)uni((int)(u32)hv);
              if (seq != 0u && (int)(seq - call) >= 0) break;
              u32 tv;
              asm volatile("global_load_dwordx2 %0, %2, off sc0 sc1\n\tglobal_load_dword %1, %3, off sc0 sc1\n\ts_waitcnt vmcnt(0)" : "=&v"(hv), "=&v"(tv) : "v"(hp), "v"(tmo) : "memory");
              const u32 seq2 = (u32)uni((int)(u32)(hv >> 32)); info = (u32)uni((int)(u32)hv);
              if ((int)(seq2 - call) >= 0) break;
              if ((u32)uni((int)tv) == call || rtc() - t0 > DEADLINE) {
                  __hip_atomic_store(tmo, call, __ATOMIC_RELAXED, __HIP_MEMORY_SCOPE_AGENT); ok = false; break; }
              __builtin_amdgcn_s_sleep(1);
          } }
        if (!ok) break;
        const char* sp = ws + OFF_PK + (size_t)par * PK_BYTES + (size_t)slot * 8192;
        const u32 low = info & 127u, Lx = low * 0x01010101u, dl = ((info >> 8) & 1u) * 0x00800080u, nex = info >> 9;
        u32x4 lo[4], hi[4];
        if (!(info & 128u)) {
            u32x4 c1; u32x2 c2; u32 ent;
            asm volatile("global_load_dwordx4 %0, %7, %10 sc0 sc1\n\tglobal_load_dwordx4 %1, %7, %10 offset:1024 sc0 sc1\n\tglobal_load_dwordx4 %2, %7, %10 offset:2048 sc0 sc1\n\tglobal_load_dwordx4 %3, %7, %10 offset:3072 sc0 sc1\n\t"
                         "global_load_dwordx4 %4, %8, %10 sc0 sc1\n\tglobal_load_dwordx2 %5, %9, %10 sc0 sc1\n\tglobal_load_ushort %6, %11, %10 sc0 sc1\n\ts_waitcnt vmcnt(0)"
                         : "=&v"(lo[0]), "=&v"(lo[1]), "=&v"(lo[2]), "=&v"(lo[3]), "=&v"(c1), "=&v"(c2), "=&v"(ent) : "v"(vs), "v"(vs2), "v"(vs3), "s"(sp), "v"(vs4) : "memory");
            LAS u32x4* hl = (LAS u32x4*)wlds + lane;
#pragma unroll
            for (int j = 0; j < 4; ++j) {
#pragma unroll
                for (int e = 0; e < 4; ++e) hi[j][e] = (((c1[j] >> (2 * e)) & 0x03030303u) + Lx) | ((c2[j >> 1] << (7 - 4 * (j & 1) - e)) & 0x80808080u);
                hl[j * 64] = hi[j];
            }
            for (u32 k0 = 0; k0 < nex; k0 += 64u) {
                if (k0 > 0u) asm volatile("global_load_ushort %0, %1, %2 sc0 sc1\n\ts_waitcnt vmcnt(0)" : "=&v"(ent) : "v"(vs4 + k0 * 2u), "s"(sp) : "memory");
                if (k0 + (u32)lane < nex) {
                    const u32 f = (ent >> 12) & 15u, q = (ent >> 2) & 15u;
                    LAS u32* w = (LAS u32*)(wlds + (q >> 2) * 1024u + ((ent >> 6) & 63u) * 16u + (q & 3u) * 4u);
                    __hip_atomic_fetch_sub(w, (f ? f * 4u : (u32)-4) << ((ent & 3u) * 8u), __ATOMIC_RELAXED, __HIP_MEMORY_SCOPE_WORKGROUP);
                }
            }
#pragma unroll
            for (int j = 0; j < 4; ++j) hi[j] = hl[j * 64];
        } else {
            asm volatile("global_load_dwordx4 %0, %8, %10 sc0 sc1\n\tglobal_load_dwordx4 %1, %8, %10 offset:1024 sc0 sc1\n\tglobal_load_dwordx4 %2, %8, %10 offset:2048 sc0 sc1\n\tglobal_load_dwordx4 %3, %8, %10 offset:3072 sc0 sc1\n\t"
                         "global_load_dwordx4 %4, %9, %10 sc0 sc1\n\tglobal_load_dwordx4 %5, %9, %10 offset:1024 sc0 sc1\n\tglobal_load_dwordx4 %6, %9, %10 offset:2048 sc0 sc1\n\tglobal_load_dwordx4 %7, %9, %10 offset:3072 sc0 sc1\n\ts_waitcnt vmcnt(0)"
                         : "=&v"(lo[0]), "=&v"(lo[1]), "=&v"(lo[2]), "=&v"(lo[3]), "=&v"(hi[0]), "=&v"(hi[1]), "=&v"(hi[2]), "=&v"(hi[3]) : "v"(vs), "v"(vs2), "s"(sp) : "memory");
        }
        const bool isw = st.isW();
        const char* ob; u32 v0, v1, js; gu32* ctr;
        if (isw) { ob = ws + OFF_W + (size_t)(st.kc * KC + 32 * (st.t - B0 / 16)) * 1024 + sv * 256; v0 = vw0; v1 = vw0 + 128; js = 8192; ctr = (gu32*)(ws + OFF_WRDY + sv * 512 + st.kc * 32) + (st.t - B0 / 16); }
        else { const int lr = st.lrow(sv); ob = ws + OFF_A + (size_t)lr * 8192 + st.kc * 512; v0 = vx0; v1 = vx0 + 64; js = 128; ctr = (gu32*)(ws + OFF_XRDY + (lr >> 4) * 64) + st.kc; }
#pragma unroll
        for (int j = 0; j < 4; ++j) {
            const u32x4 a = (u32x4){pksub(vperm(hi[j][0], lo[j][0], 0x05010400u), dl), pksub(vperm(hi[j][0], lo[j][0], 0x07030602u), dl), pksub(vperm(hi[j][1], lo[j][1], 0x05010400u), dl), pksub(vperm(hi[j][1], lo[j][1], 0x07030602u), dl)};
            const u32x4 b = (u32x4){pksub(vperm(hi[j][2], lo[j][2], 0x05010400u), dl), pksub(vperm(hi[j][2], lo[j][2], 0x07030602u), dl), pksub(vperm(hi[j][3], lo[j][3], 0x05010400u), dl), pksub(vperm(hi[j][3], lo[j][3], 0x07030602u), dl)};
            GSTORE_SYS(v0, a, ob, 0); GSTORE_SYS(v1, b, ob, 0); ob += js;
        }
        st.t += nsub;
        have = st.locate();
        { const GAS char* hn = (const GAS char*)(ws + OFF_HDR + ((size_t)par * NSLOT + (have ? st.slot(sv) : slot)) * 128);
          asm volatile("global_load_dwordx2 %0, %1, off sc0 sc1\n\ts_waitcnt vmcnt(0)" : "=&v"(hv) : "v"(hn) : "memory"); }
        if (lane == 0) __hip_atomic_store(ctr, call, __ATOMIC_RELAXED, __HIP_MEMORY_SCOPE_AGENT);
        ++n; if (n < 30) STAMP(1 + n);
    }
    STAMP(1);
}

__global__ __launch_bounds__(NTHREADS, 1) void ag_gemm_kernel(Params p) {
    extern __shared__ __attribute__((aligned(16))) char smem[];
    LAS char* lds = (LAS char*)smem;
    const int b = blockIdx.x, xcd = b & 7, slot = b >> 3, wave = uni(threadIdx.x >> 6), lane = threadIdx.x & 63;
    gu32* cnt = (gu32*)(p.ws[p.rank] + OFF_CNT + (size_t)b * 128) + wave;
    const int c = slot < 32 - NGS ? slot * 8 + xcd : -1;
    const int gs = slot - (32 - NGS);
    if (c >= 0 && c < p.nsw && wave < p.wpw && !(p.mode & 2)) { send_role(p, lds + wave * 8192, cnt, wave * p.nsw + c, p.nsw * p.wpw); return; }
    const u32 call = (u32)uni((int)*cnt) + 1u;
    if (lane == 0) *cnt = call;
    if (c < 0) {
        if (p.mode & 1) return;
        const int ct = xcd & 3, hb = xcd >> 2;
        if (gs < NT0 + NS1O) { const bool g0 = gs < NT0; gemm_role<4>(p, lds, call, g0 ? 2 * (hb * (NT0 / 2) + (gs >> 1)) : R0 / 64 + 2 * (hb * NS1O + gs - NT0), g0 ? gs & 1 : 0, ct, g0); }
        else gemm_role<2>(p, lds, call, R0 / 64 + hb * NS1H + gs - NT0 - NS1O, 1, ct, false);
        return;
    }
    if (c >= p.nsw && c < p.nsw + p.nuw) { if (wave < p.upw && !(p.mode & 2)) unpack_role(p, lds + wave * 4096, call, wave * p.nuw + (c - p.nsw), p.nuw * p.upw); return; }
    if (c == NCOMM - 1 && wave == 0 && !(p.mode & 3)) {
        gu32* tmo = (gu32*)(p.ws[p.rank] + OFF_TMO);
        STAMP(0);
        if (!wait_all<4 * NDONE * 8 / 64>((gu32*)(p.ws[p.rank] + OFF_DONE), tmo, call)) {
            if (lane == 0) ((GAS uint16_t*)p.out[p.rank])[(size_t)p.rank * SROWS * SCOLS] = 0x7fc0;
        }
        STAMP(39);
    }
}
}

struct AggKnobs { int nsw = 63, wpw = 2, nuw = 32, upw = 8, nx = 17, stag = 0, mode = 0, n1 = 8; };
static AggKnobs g_knobs;

extern "C" void kernel_launch(int rank, int nranks, ncclComm_t comm, hipStream_t stream,
                              void* const* d_in, const int* in_sizes, int n_in, void* d_out, int out_size,
                              const int* in_dtypes, int out_dtype, void* const* d_out_all, void* const* d_ws_all, size_t ws_bytes) {
    using namespace agg;
    if (nranks != NRANK || (unsigned)rank >= (unsigned)NRANK || ws_bytes < WS_NEED || n_in != 2 || in_sizes[0] != SROWS * KDIM || in_sizes[1] != KDIM * SCOLS || out_size != 4096 * SCOLS) {
        fprintf(stderr, "ag_gemm: unexpected shapes\n"); return; }
    static bool init[NRANK] = {};
    if (!init[rank]) { (void)hipFuncSetAttribute((const void*)ag_gemm_kernel, hipFuncAttributeMaxDynamicSharedMemorySize, LDS_BYTES); init[rank] = true; }
    Params p{};
    p.x = (const uint16_t*)d_in[0]; p.w = (const uint16_t*)d_in[1];
    for (int i = 0; i < NRANK; ++i) { p.out[i] = (char*)d_out_all[i]; p.ws[i] = (char*)d_ws_all[i]; }
    p.rank = rank;
    p.nsw = g_knobs.nsw; p.wpw = g_knobs.wpw; p.nuw = g_knobs.nuw; p.upw = g_knobs.upw; p.nx = g_knobs.nx; p.stag = g_knobs.stag; p.mode = g_knobs.mode;
    { const int n1 = g_knobs.n1; unsigned m = 0; int j = 0;
      for (; j < 16 + n1; ++j) if (((j + 1) * n1) / (16 + n1) > (j * n1) / (16 + n1)) m |= 1u << j;
      for (; j < 32; ++j) m |= 1u << j;
      p.gmask = m; }
    hipLaunchKernelGGL(ag_gemm_kernel, dim3(NWG), dim3(NTHREADS), LDS_BYTES, stream, p);
}
```

```cpp
#include <hip/hip_runtime.h>
#include <rccl/rccl.h>
#include <cstdio>
#include <cstdint>

namespace agg {
typedef unsigned u32;
typedef unsigned long long u64;
typedef u32 u32x4 __attribute__((ext_vector_type(4)));
typedef u32 u32x2 __attribute__((ext_vector_type(2)));
typedef float f32x4 __attribute__((ext_vector_type(4)));
typedef short s16x4 __attribute__((ext_vector_type(4)));
typedef __bf16 bf16x8 __attribute__((ext_vector_type(8)));
#define LAS __attribute__((address_space(3)))
#define GAS __attribute__((address_space(1)))
typedef GAS u32 gu32;

constexpr int NRANK = 8, KDIM = 4096, SROWS = 512, SCOLS = 512;
constexpr int KC = 256, NKC = KDIM / KC;
constexpr int R0 = 1024, R1 = 2048 - R0;
constexpr int A0 = 208, A1 = 128, B0 = R0 / 4 - A0, B1 = R1 / 4 - A1, NA = A0 + A1;
static_assert(A0 % 16 == 0 && A1 % 16 == 0 && B0 % 16 == 0 && B1 % 16 == 0 && B0 > 0 && B1 > 0 && B0 + B1 + NA == SROWS, "row split");
constexpr int NT0 = R0 / 128, NS1O = R1 / 256, NS1H = R1 / 128, NGS = NT0 + NS1O + NS1H;
static_assert(NT0 % 2 == 0 && R1 % 256 == 0 && NGS < 32, "tiles");
constexpr int NCOMM = (32 - NGS) * 8;
constexpr int NDONE = NT0 + R1 / 64;
constexpr int BK = 64, NST = KDIM / BK;
constexpr int LDS_CTL = 147456, LDS_BYTES = LDS_CTL + 64;
constexpr int NTHREADS = 704, NWG = 256;
constexpr int NSW = 63, WPW = 2, NX = 17, NUW = 32, UPW = 8;
constexpr int N1 = 8;
constexpr int PACE = 2;
static_assert(NSW + NUW + 1 <= NCOMM && 7 * NX <= NSW * WPW, "roles");

constexpr size_t OFF_CNT = 0;
constexpr size_t OFF_TMO = 32768;
constexpr size_t OFF_DUMMY = OFF_TMO + 256;
constexpr size_t OFF_XRDY = 40960;
constexpr size_t OFF_WRDY = OFF_XRDY + 128 * 64;
constexpr size_t OFF_DONE = OFF_WRDY + 4 * 512;
constexpr int NSLOT = 2048 + 512;
constexpr size_t OFF_HDR = 1048576;
constexpr size_t A_BYTES = (size_t)2048 * KDIM * 2, W_BYTES = (size_t)KDIM * 512 * 2;
constexpr size_t OFF_A = 2097152;
constexpr size_t OFF_W = OFF_A + A_BYTES;
constexpr size_t PK_BYTES = (size_t)NSLOT * 8192;
constexpr size_t OFF_PK = OFF_W + W_BYTES;
constexpr size_t WS_NEED = OFF_PK + 2 * PK_BYTES;
static_assert(OFF_DONE + 4 * NDONE * 32 <= OFF_HDR && (4 * NDONE * 8) % 64 == 0 && OFF_HDR + 2 * NSLOT * 128 <= OFF_A, "ctl");
static_assert(WS_NEED <= (size_t)65 * 1048576, "workspace");
constexpr u64 DEADLINE = 150000000ull;

struct Params {
    const uint16_t* x; const uint16_t* w;
    char* out[NRANK]; char* ws[NRANK];
    int rank;
    unsigned gmask;
};

__device__ __forceinline__ u64 rtc() { return __builtin_amdgcn_s_memrealtime(); }
__device__ __forceinline__ int uni(int v) { return __builtin_amdgcn_readfirstlane(v); }

__device__ __forceinline__ void map_unit(int unit, int u, int& rank, int& r) {
    const int lr = unit * 16;
    if (lr < R0) { int i = lr; if (i < 4 * A0) { const int j = i / A0; r = i - j * A0; rank = 4 * u + j; }
                   else { i -= 4 * A0; const int j = i / B0; r = NA + (i - j * B0); rank = 4 * (1 - u) + j; } }
    else { int i = lr - R0; if (i < 4 * A1) { const int j = i / A1; r = A0 + i - j * A1; rank = 4 * u + j; }
           else { i -= 4 * A1; const int j = i / B1; r = NA + B0 + (i - j * B1); rank = 4 * (1 - u) + j; } }
}

template <int NW> __device__ __forceinline__ bool wait_all(gu32* words, gu32* tmo, u32 call) {
    const u64 t0 = rtc();
    const int lane = threadIdx.x & 63;
    for (;;) {
        u32 v[NW];
#pragma unroll
        for (int j = 0; j < NW; ++j) v[j] = __hip_atomic_load(words + j * 64 + lane, __ATOMIC_RELAXED, __HIP_MEMORY_SCOPE_SYSTEM);
        const u32 t = __hip_atomic_load(tmo, __ATOMIC_RELAXED, __HIP_MEMORY_SCOPE_AGENT);
        bool all = true;
#pragma unroll
        for (int j = 0; j < NW; ++j) all = all && (int)(v[j] - call) >= 0;
        if (__all(all)) return true;
        if ((u32)uni((int)t) == call || rtc() - t0 > DEADLINE) {
            __hip_atomic_store(tmo, call, __ATOMIC_RELAXED, __HIP_MEMORY_SCOPE_AGENT);
            return false;
        }
    }
}

__device__ __forceinline__ void st16_sys(char* ptr, u32x4 v) {
    asm volatile("global_store_dwordx4 %0, %1, off sc0 sc1\n\ts_nop 1" :: "v"(ptr), "v"(v) : "memory");
}
typedef float f32x2 __attribute__((ext_vector_type(2)));
__device__ __forceinline__ f32x2 gelu2(f32x2 y) {
    const float k1 = -2.0f * 0.7978845608028654f * 1.4426950408889634f, k2 = k1 * 0.044715f;
    const f32x2 a = y * __builtin_elementwise_fma(y * y, (f32x2){k2, k2}, (f32x2){k1, k1});
    const f32x2 d = (f32x2){__builtin_amdgcn_exp2f(a[0]), __builtin_amdgcn_exp2f(a[1])} + (f32x2){1.0f, 1.0f};
    return y * (f32x2){__builtin_amdgcn_rcpf(d[0]), __builtin_amdgcn_rcpf(d[1])};
}
__device__ __forceinline__ u32 cvt_pk(float lo, float hi) { u32 r; asm("v_cvt_pk_bf16_f32 %0, %1, %2" : "=v"(r) : "v"(lo), "v"(hi)); return r; }

#define BAR() __builtin_amdgcn_s_barrier()
#define WAITV(n) asm volatile("s_waitcnt vmcnt(" #n ")" ::: "memory")
#define WAITL0() asm volatile("s_waitcnt lgkmcnt(0)" ::: "memory")
#define SB() __builtin_amdgcn_sched_barrier(0)
typedef float f32x16 __attribute__((ext_vector_type(16)));
__device__ __forceinline__ const char* uniptr(const char* q) { const u64 a = (u64)q; return (const char*)(((u64)(u32)uni((int)(u32)(a >> 32)) << 32) | (u32)uni((int)(u32)a)); }
template <int N> __device__ __forceinline__ void waitv() { asm volatile("s_waitcnt vmcnt(%0)" :: "n"(N) : "memory"); }
__device__ __forceinline__ void glds2(u32 v0, u32 v1, const char* sb, u32 m0v) {
    asm volatile("s_mov_b32 m0, %3\n\ts_nop 0\n\tglobal_load_lds_dwordx4 %0, %2 sc0 sc1\n\tglobal_load_lds_dwordx4 %1, %2 offset:1024 sc0 sc1" :: "v"(v0), "v"(v1), "s"(sb), "s"(m0v) : "memory");
}
__device__ __forceinline__ void glds4(u32 v0, u32 v1, u32 v2, u32 v3, const char* sb, u32 m0v) {
    asm volatile("s_mov_b32 m0, %5\n\ts_nop 0\n\tglobal_load_lds_dwordx4 %0, %4 sc0 sc1\n\tglobal_load_lds_dwordx4 %1, %4 offset:1024 sc0 sc1\n\t"
                 "global_load_lds_dwordx4 %2, %4 offset:2048 sc0 sc1\n\tglobal_load_lds_dwordx4 %3, %4 offset:3072 sc0 sc1" :: "v"(v0), "v"(v1), "v"(v2), "v"(v3), "s"(sb), "s"(m0v) : "memory");
}
template <int OFF> __device__ __forceinline__ void rdx(bf16x8& d, u32 a) { asm volatile("ds_read_b128 %0, %1 offset:%c2" : "=v"(d) : "v"(a), "i"(OFF) : "memory"); }
template <int OFF> __device__ __forceinline__ void rdw(s16x4& d0, s16x4& d1, u32 a0, u32 a1) {
    asm volatile("ds_read_b64_tr_b16 %0, %2 offset:%c4\n\tds_read_b64_tr_b16 %1, %3 offset:%c4" : "=&v"(d0), "=&v"(d1) : "v"(a0), "v"(a1), "i"(OFF) : "memory"); }
template <int NI> struct Frags { s16x4 w[4][2]; bf16x8 x[4][NI]; };

template <int NF>
__device__ __forceinline__ void gemm_role(const Params& p, LAS char* lds, u32 call, int rt64, int helper, int ct, bool early) {
    constexpr int TM = 32 * NF, NBUF = NF == 4 ? 4 : 6, ASTG = TM * 128, LB = NBUF * ASTG;
    constexpr int NUN = TM / 16;
    constexpr int PPS = NF == 4 ? 16 : 12;
    constexpr int NPRE = NBUF - 1 < 4 ? NBUF - 1 : 4;
    const int g = p.rank, u = g >> 2, v = g & 3;
    const int wave = uni(threadIdx.x >> 6), lane = threadIdx.x & 63;
    char* const ws = p.ws[g];
    gu32* tmo = (gu32*)(ws + OFF_TMO);
    LAS u32* ctl = (LAS u32*)(lds + LDS_CTL);
    const u32 ldsb = (u32)(size_t)lds;
    if (wave == 10) {
        const int dk = lane & 3;
        const bool xl = lane < 4 * NUN, wl = helper && lane >= 32;
        bool mine = false;
        if (xl) { int rk, r; map_unit(rt64 * 4 + (lane >> 2), u, rk, r); mine = rk == g; }
        const bool watch = (xl && !mine) || wl;
        const char* wbase = wl ? ws + OFF_WRDY + ct * 512 + ((lane - 32) >> 2) * 4 : ws + OFF_XRDY + (size_t)(rt64 * 4 + (xl ? lane >> 2 : 0)) * 64;
        const int wstep = wl ? 32 : 4;
        bool ok = true; int ready = 0;
        auto need = [&](int c) {
            if (ready > c) return;
            const u64 t0 = rtc();
            const int kc = min(c + dk, 15);
            gu32* word = (gu32*)(wbase + kc * wstep);
            for (int it = 0;; ++it) {
                const u32 val = __hip_atomic_load(word, __ATOMIC_RELAXED, __HIP_MEMORY_SCOPE_SYSTEM);
                const u64 miss = __ballot(watch && (int)(val - call) < 0);
                const u64 m4 = 0x1111111111111111ull;
                ready = (miss & m4) ? c : (miss & (m4 << 1)) ? c + 1 : (miss & (m4 << 2)) ? c + 2 : (miss & (m4 << 3)) ? c + 3 : c + 4;
                if (ready > c) return;
                if ((it & 7) == 7 && ((u32)uni((int)__hip_atomic_load(tmo, __ATOMIC_RELAXED, __HIP_MEMORY_SCOPE_AGENT)) == call || rtc() - t0 > DEADLINE)) {
                    __hip_atomic_store(tmo, call, __ATOMIC_RELAXED, __HIP_MEMORY_SCOPE_AGENT); ok = false; ready = 16; return; }
                __builtin_amdgcn_s_sleep(1);
            }
        };
        need(0);
        BAR();
        if (NBUF > 4) need(1);
        BAR();
        for (int s = 0; s < NST - 1; ++s) {
            if (((s + NBUF) & 3) == 0 && s + NBUF < NST) need((s + NBUF) >> 2);
            if (s == NST - NBUF - 2) {
                const u32 t = (u32)uni((int)__hip_atomic_load(tmo, __ATOMIC_RELAXED, __HIP_MEMORY_SCOPE_AGENT));
                if (lane == 0) ctl[0] = (!ok || t == call) ? 1u : 0u;
                WAITL0();
            }
            BAR();
        }
        if (helper) BAR();
        return;
    }
    if (wave >= 8) {
        const bool l0 = wave == 8;
        u32 va[2], vb[4]; const char* ub[NUN];
#pragma unroll
        for (int e = 0; e < 2; ++e) { const int ar = 8 * e + (lane >> 3); va[e] = (u32)ar * 8192u + (u32)(((lane & 7) ^ ((ar >> 1) & 7)) * 16) - (u32)e * 1024u; }
#pragma unroll
        for (int i = 0; i < NUN; ++i) { int rk, r; map_unit(rt64 * 4 + i, u, rk, r);
            ub[i] = uniptr((rk == g) ? (const char*)p.x + (size_t)r * 8192 : ws + OFF_A + (size_t)(rt64 * 64 + i * 16) * 8192); }
#pragma unroll
        for (int e = 0; e < 4; ++e) { const int kr = 4 * e + (lane >> 4), sw = ((kr & 3) << 2) | ((kr >> 2) & 3); vb[e] = (u32)kr * 1024u + (u32)(((lane & 15) ^ sw) * 16) - (u32)e * 1024u; }
        const char* b0 = uniptr(helper ? ws + OFF_W + ct * 256 : (const char*)p.w + ct * 256);
        auto issue = [&](int s, int buf) {
            const u32 la = ldsb + (u32)buf * ASTG, lb = ldsb + LB + (u32)buf * 16384u;
            const char* b = b0 + (size_t)s * 65536;
            if (l0) {
                const u32 v0 = va[0] + (u32)s * 128u, v1 = va[1] + (u32)s * 128u;
#pragma unroll
                for (int i = 0; i < NUN; ++i) glds2(v0, v1, ub[i], la + i * 2048);
                if (NF == 2) glds4(vb[0], vb[1], vb[2], vb[3], b, lb);
            } else {
#pragma unroll
                for (int q = (NF == 2 ? 1 : 0); q < 4; ++q) glds4(vb[0], vb[1], vb[2], vb[3], b + q * 16384, lb + q * 4096);
            }
        };
        BAR();
        for (int s = 0; s < NPRE; ++s) issue(s, s);
        waitv<(NPRE - 1) * PPS>(); BAR();
        for (int s = NPRE; s < NBUF; ++s) issue(s, s);
        int buf = 0;
        for (int s = 0; s < NST - NBUF; ++s) { waitv<(NBUF - 2) * PPS>(); BAR(); issue(s + NBUF, buf); buf = buf + 1 == NBUF ? 0 : buf + 1; }
        waitv<(NBUF - 2) * PPS>(); BAR();
        waitv<(NBUF - 3) * PPS>(); BAR();
        waitv<(NBUF - 4) * PPS>(); BAR();
        if (NBUF == 6) { waitv<PPS>(); BAR(); waitv<0>(); BAR(); }
        if (helper) BAR();
        return;
    }
    constexpr int NI = NF / 2;
    const int wm = wave >> 2, wn = wave & 3;
    const int l31 = lane & 31, lh = lane >> 5;
    u32 xa[4], wa[2][2];
#pragma unroll
    for (int ks = 0; ks < 4; ++ks) xa[ks] = ldsb + (u32)((wm * (TM / 2) + l31) * 128 + (((2 * ks + lh) ^ ((l31 >> 1) & 7)) * 16));
    { const int blk = (lane >> 4) & 1, tq = (lane & 15) >> 2, tp = lane & 3;
#pragma unroll
      for (int t = 0; t < 2; ++t) {
          const int kr = 8 * lh + 4 * t + tq;
          const int sw = ((kr & 3) << 2) | ((kr >> 2) & 3);
          const int ch = 4 * wn + 2 * blk + (tp >> 1);
          wa[0][t] = ldsb + (u32)(LB + kr * 256 + ((ch ^ sw) * 16) + 8 * (tp & 1));
          wa[1][t] = wa[0][t] + 3 * 16384;
      } }
    f32x16 acc[NI];
#pragma unroll
    for (int i = 0; i < NI; ++i)
#pragma unroll
        for (int e = 0; e < 16; ++e) acc[i][e] = 0.f;
#define RDW(f, BUF, ks) rdw<((BUF) % 3) * 16384 * (NBUF == 6) + (BUF) * 16384 * (NBUF == 4) + (ks) * 4096>(f.w[ks][0], f.w[ks][1], wa[NBUF == 6 ? (BUF) / 3 : 0][0], wa[NBUF == 6 ? (BUF) / 3 : 0][1])
#define RDX(f, BUF, ks, i) rdx<(BUF) * ASTG + (i) * 4096>(f.x[ks][i], xa[ks])
#define MF(f, ks, i) do { acc[i] = __builtin_amdgcn_mfma_f32_32x32x16_bf16(__builtin_bit_cast(bf16x8, __builtin_shufflevector(f.w[ks][0], f.w[ks][1], 0, 1, 2, 3, 4, 5, 6, 7)), f.x[ks][i], acc[i], 0, 0, 0); SB(); } while (0)
#define LANDED(f) asm volatile("s_waitcnt lgkmcnt(0)" : "+v"(f.w[0][0]), "+v"(f.w[0][1]), "+v"(f.w[1][0]), "+v"(f.w[1][1]), "+v"(f.w[2][0]), "+v"(f.w[2][1]), "+v"(f.w[3][0]), "+v"(f.w[3][1]), \
                     "+v"(f.x[0][0]), "+v"(f.x[1][0]), "+v"(f.x[2][0]), "+v"(f.x[3][0]), "+v"(f.x[0][NI - 1]), "+v"(f.x[1][NI - 1]), "+v"(f.x[2][NI - 1]), "+v"(f.x[3][NI - 1]) :: "memory")
#define LANDED1(f) asm volatile("s_waitcnt lgkmcnt(0)" : "+v"(f.w[0][0]), "+v"(f.w[0][1]), "+v"(f.w[1][0]), "+v"(f.w[1][1]), "+v"(f.w[2][0]), "+v"(f.w[2][1]), "+v"(f.w[3][0]), "+v"(f.w[3][1]), \
                     "+v"(f.x[0][0]), "+v"(f.x[1][0]), "+v"(f.x[2][0]), "+v"(f.x[3][0]) :: "memory")
#define WAITF(f) do { if constexpr (NI == 2) { LANDED(f); } else { LANDED1(f); } } while (0)
#define RDK(f, NB, ks) do { RDW(f, NB, ks); RDX(f, NB, ks, 0); if constexpr (NI == 2) { RDX(f, NB, ks, NI - 1); } } while (0)
#define MFK(f, ks) do { MF(f, ks, 0); if constexpr (NI == 2) { MF(f, ks, NI - 1); } } while (0)
#define STEP(cur, nxt, NB) do { BAR(); SB(); \
        if constexpr (NI == 2) { \
            MF(cur, 0, 0); RDW(nxt, NB, 0); RDX(nxt, NB, 0, 0); SB(); \
            MF(cur, 0, NI - 1); RDX(nxt, NB, 0, NI - 1); RDW(nxt, NB, 1); SB(); \
            MF(cur, 1, 0); RDX(nxt, NB, 1, 0); RDX(nxt, NB, 1, NI - 1); SB(); \
            MF(cur, 1, NI - 1); RDW(nxt, NB, 2); RDX(nxt, NB, 2, 0); SB(); \
            MF(cur, 2, 0); RDX(nxt, NB, 2, NI - 1); RDW(nxt, NB, 3); SB(); \
            MF(cur, 2, NI - 1); RDX(nxt, NB, 3, 0); RDX(nxt, NB, 3, NI - 1); SB(); \
            MF(cur, 3, 0); MF(cur, 3, NI - 1); \
        } else { \
            MF(cur, 0, 0); RDK(nxt, NB, 0); RDW(nxt, NB, 1); SB(); \
            MF(cur, 1, 0); RDX(nxt, NB, 1, 0); RDK(nxt, NB, 2); SB(); \
            MF(cur, 2, 0); RDK(nxt, NB, 3); SB(); \
            MF(cur, 3, 0); \
        } \
        WAITF(nxt); } while (0)
    Frags<NI> P, Q;
    BAR();
    BAR();
    RDK(P, 0, 0); RDK(P, 0, 1); RDK(P, 0, 2); RDK(P, 0, 3); WAITF(P);
    for (int s = 0; s < 60; s += NBUF) {
        STEP(P, Q, 1); STEP(Q, P, 2); STEP(P, Q, 3);
        if constexpr (NBUF == 4) { STEP(Q, P, 0); }
        else { STEP(Q, P, 4 % NBUF); STEP(P, Q, 5 % NBUF); STEP(Q, P, 0); }
    }
    STEP(P, Q, 1); STEP(Q, P, 2); STEP(P, Q, 3);
    MFK(Q, 0); MFK(Q, 1); MFK(Q, 2); MFK(Q, 3);
    const bool bad = ctl[0] != 0u;
    u32x2 pk[NI][4];
#pragma unroll
    for (int i = 0; i < NI; ++i) { const f32x16& a = acc[i];
#pragma unroll
        for (int e = 0; e < 4; ++e) { const f32x2 y0 = gelu2((f32x2){a[4 * e], a[4 * e + 1]}), y1 = gelu2((f32x2){a[4 * e + 2], a[4 * e + 3]});
            pk[i][e] = (u32x2){cvt_pk(y0[0], y0[1]), cvt_pk(y1[0], y1[1])}; } }
    char* const ocol = helper ? p.out[4 * (1 - u) + ct] + v * 256 : p.out[g] + ct * 256;
    if (!helper) {
#pragma unroll
        for (int i = 0; i < NI; ++i) {
            const int row0 = wm * (TM / 2) + 32 * i;
            int rk0, r0, rk1, r1; map_unit(rt64 * 4 + (row0 >> 4), u, rk0, r0); map_unit(rt64 * 4 + (row0 >> 4) + 1, u, rk1, r1);
            const int grow = (l31 < 16 ? rk0 * SROWS + r0 : rk1 * SROWS + r1) + (l31 & 15);
            char* dst = ocol + (size_t)grow * 1024 + wn * 64 + lh * 16;
#pragma unroll
            for (int e = 0; e < 4; e += 2) {
                auto s0 = __builtin_amdgcn_permlane32_swap(pk[i][e][0], pk[i][e + 1][0], false, false);
                auto s1 = __builtin_amdgcn_permlane32_swap(pk[i][e][1], pk[i][e + 1][1], false, false);
                u32x4 ov = (u32x4){s0[0], s1[0], s0[1], s1[1]};
                if (bad) ov = (u32x4){0x7fc07fc0u, 0x7fc07fc0u, 0x7fc07fc0u, 0x7fc07fc0u};
                *(GAS u32x4*)(dst + e * 16) = ov;
            }
        }
    } else {
        LAS char* otile = lds + LB + (NF == 4 ? 0 : 4 * 16384);
#pragma unroll
        for (int i = 0; i < NI; ++i) { const int row = wm * (TM / 2) + 32 * i + l31;
#pragma unroll
            for (int e = 0; e < 4; ++e) *(LAS u32x2*)(otile + row * 256 + (((4 * wn + e) ^ (row & 15)) * 16) + lh * 8) = pk[i][e]; }
        WAITL0(); BAR();
        constexpr int RPW = TM / 8, NS = RPW / 4;
        const int row0 = RPW * wave;
        int urk, urow; map_unit(rt64 * 4 + (row0 >> 4), u, urk, urow);
        char* obase = ocol + (size_t)(urk * SROWS + urow + (row0 & 15)) * 1024;
        u32x4 ov[NS];
#pragma unroll
        for (int it = 0; it < NS; ++it) {
            const int row = row0 + 4 * it + (lane >> 4), ch = lane & 15;
            ov[it] = *(LAS u32x4*)(otile + row * 256 + ((ch ^ (row & 15)) * 16));
            if (bad) ov[it] = (u32x4){0x7fc07fc0u, 0x7fc07fc0u, 0x7fc07fc0u, 0x7fc07fc0u};
        }
#pragma unroll
        for (int it = 0; it < NS; ++it) {
            st16_sys(obase + (size_t)(4 * it + (lane >> 4)) * 1024 + (lane & 15) * 16, ov[it]);
            if (early && (it + 1) % PACE == 0) WAITV(0);
        }
    }
    if (helper) {
        WAITV(0);
        const int tile = early ? rt64 / 2 : NT0 + rt64 - R0 / 64;
        if (lane == 0) __hip_atomic_store((gu32*)(p.ws[4 * (1 - u) + ct] + OFF_DONE) + (v * NDONE + tile) * 8 + wave, call, __ATOMIC_RELAXED, __HIP_MEMORY_SCOPE_SYSTEM);
    }
}

struct Stream {
    u32 gmask; int step, t, G, kc; bool cross; int c1;
    __device__ __forceinline__ bool locate() {
#pragma unroll 1
        for (;;) { if (step >= 32) return false;
            G = (gmask >> step) & 1; kc = G ? c1 : step - c1;
            const int cn = cross ? (G ? B1 / 16 : B0 / 16 + 8) : (G ? A1 / 16 : A0 / 16); if (t < cn) return true; t -= cn; c1 += G; ++step; } }
    __device__ __forceinline__ bool isW() const { return cross && G == 0 && t >= B0 / 16; }
    __device__ __forceinline__ int lrow(int sv) const { return (G ? R0 : 0) + (cross ? 4 * (G ? A1 : A0) + sv * (G ? B1 : B0) : sv * (G ? A1 : A0)) + 16 * t; }
    __device__ __forceinline__ int slot(int sv) const { return isW() ? 2048 + sv * 128 + kc * 8 + (t - B0 / 16) : (lrow(sv) >> 4) * 16 + kc; }
};

__device__ __forceinline__ u32 vperm(u32 s0, u32 s1, u32 sel) { return __builtin_amdgcn_perm(s0, s1, sel); }
__device__ __forceinline__ u32 pkmax(u32 a, u32 b) { u32 r; asm("v_pk_max_u16 %0, %1, %2" : "=v"(r) : "v"(a), "v"(b)); return r; }
__device__ __forceinline__ u32 pkadd(u32 a, u32 b) { u32 r; asm("v_pk_add_u16 %0, %1, %2" : "=v"(r) : "v"(a), "s"(b)); return r; }
__device__ __forceinline__ u32 pksub(u32 a, u32 b) { u32 r; asm("v_pk_sub_u16 %0, %1, %2" : "=v"(r) : "v"(a), "s"(b)); return r; }
__device__ __forceinline__ u32 wave_scan_incl(u32 x) {
    x += (u32)__builtin_amdgcn_update_dpp(0, (int)x, 0x111, 0xf, 0xf, true);
    x += (u32)__builtin_amdgcn_update_dpp(0, (int)x, 0x112, 0xf, 0xf, true);
    x += (u32)__builtin_amdgcn_update_dpp(0, (int)x, 0x114, 0xf, 0xf, true);
    x += (u32)__builtin_amdgcn_update_dpp(0, (int)x, 0x118, 0xf, 0xf, true);
    x += (u32)__builtin_amdgcn_update_dpp(0, (int)x, 0x142, 0xa, 0xf, true);
    x += (u32)__builtin_amdgcn_update_dpp(0, (int)x, 0x143, 0xc, 0xf, true);
    return x;
}
__device__ __forceinline__ u32 wave_max(u32 x) {
    x = max(x, (u32)__builtin_amdgcn_update_dpp(0, (int)x, 0x111, 0xf, 0xf, true));
    x = max(x, (u32)__builtin_amdgcn_update_dpp(0, (int)x, 0x112, 0xf, 0xf, true));
    x = max(x, (u32)__builtin_amdgcn_update_dpp(0, (int)x, 0x114, 0xf, 0xf, true));
    x = max(x, (u32)__builtin_amdgcn_update_dpp(0, (int)x, 0x118, 0xf, 0xf, true));
    const u32 a = (u32)__builtin_amdgcn_readlane((int)x, 15), b = (u32)__builtin_amdgcn_readlane((int)x, 31), c = (u32)__builtin_amdgcn_readlane((int)x, 47), d = (u32)__builtin_amdgcn_readlane((int)x, 63);
    return max(max(a, b), max(c, d));
}
struct Raw { u32x4 d[8]; };
#define GLOAD(dst, voff, sbase) asm volatile("global_load_dwordx4 %0, %1, %2" : "=v"(dst) : "v"(voff), "s"(sbase) : "memory")
#define PIN8(R) "+v"(R.d[0]), "+v"(R.d[1]), "+v"(R.d[2]), "+v"(R.d[3]), "+v"(R.d[4]), "+v"(R.d[5]), "+v"(R.d[6]), "+v"(R.d[7])
#define GSTORE_SYS(voff, val, sbase, off) asm volatile("global_store_dwordx4 %0, %1, %2 offset:" #off " sc0 sc1\n\ts_nop 1" :: "v"(voff), "v"(val), "s"(sbase) : "memory")

__device__ __forceinline__ void send_role(const Params& p, LAS char* wlds, gu32* cnt, int id, int nw) {
    const int g = p.rank, u = g >> 2, v = g & 3, lane = threadIdx.x & 63;
    const int nx = NX, ni = (nw - 4 * nx) / 3 < nx ? nx : (nw - 4 * nx) / 3;
    int di, sub;
    if (id < 7 * nx) { di = id % 7; sub = id / 7; } else { di = (id - 7 * nx) % 3; sub = nx + (id - 7 * nx) / 3; }
    const int nsub = di >= 3 ? nx : ni;
    if (sub >= nsub) { const u32 c = *cnt; if (lane == 0) *cnt = c + 1; return; }
    const bool cross = di >= 3;
    const int dest = cross ? 4 * (1 - u) + (di - 3) : 4 * u + ((v + 1 + di) & 3);
    const u32 vx0 = (u32)(lane >> 2) * 8192u + (u32)(lane & 3) * 16u, vw0 = (u32)(lane >> 3) * 1024u + (u32)(lane & 7) * 16u;
    Stream ld{p.gmask, 0, sub, 0, 0, cross, 0};
    auto load = [&](Raw& R) -> int {
        const bool have = ld.locate();
        const char* sb; u32 v0, v1, js; int slot = -1;
        if (have && ld.isW()) { sb = (const char*)p.w + (size_t)(ld.kc * KC + 32 * (ld.t - B0 / 16)) * 1024 + (di - 3) * 256; v0 = vw0; v1 = vw0 + 128; js = 8192; }
        else { const int r0 = have ? (cross ? NA + (ld.G ? B0 : 0) : (ld.G ? A0 : 0)) + 16 * ld.t : 0; sb = (const char*)p.x + (size_t)r0 * 8192 + (have ? ld.kc : 0) * 512; v0 = vx0; v1 = vx0 + 64; js = 128; }
        if (have) { slot = ld.slot(v); ld.t += nsub; }
        GLOAD(R.d[0], v0, sb); GLOAD(R.d[1], v1, sb); sb += js; GLOAD(R.d[2], v0, sb); GLOAD(R.d[3], v1, sb); sb += js;
        GLOAD(R.d[4], v0, sb); GLOAD(R.d[5], v1, sb); sb += js; GLOAD(R.d[6], v0, sb); GLOAD(R.d[7], v1, sb);
        return slot;
    };
    Raw RA, RB;
    int slotA = load(RA), slotB = load(RB);
    const u32 call = (u32)uni((int)*cnt) + 1u;
    if (lane == 0) *cnt = call;
    asm volatile("s_waitcnt vmcnt(0)" : PIN8(RA) :: "memory");
    const int par = call & 1;
    const char* dP = p.ws[dest] + OFF_PK + (size_t)par * PK_BYTES;
    char* dH = p.ws[dest] + OFF_HDR + (size_t)par * NSLOT * 128;
    char* prevH = p.ws[g] + OFF_DUMMY; u32 prevInfo = 0;
    const u32 vs = (u32)lane * 16u, vs2 = vs + 4096u, vs3 = (u32)lane * 8u + 5120u, vs4 = vs + 5632u;
    LAS uint16_t* elist = (LAS uint16_t*)wlds;
    auto iter = [&](Raw& R, int& slot, bool first) {
        if (!first) asm volatile("s_waitcnt vmcnt(16)" : PIN8(R) :: "memory");
        u32 m = 0;
#pragma unroll
        for (int e = 0; e < 4; ++e) m = pkmax(m, R.d[0][e] & 0x7fff7fffu);
        const bool quick = first && (sub & 1);
        const u32 emax = quick ? 257u : wave_max(max(m & 0xffffu, m >> 16)) >> 7;
        const u32 delta = ~emax & 1u, top = (emax + delta) >> 1, low = top > 3u ? top - 3u : 0u, Lx = low * 0x01010101u, dl = delta * 0x00800080u;
        u32x4 lo[4], hi[4], cu = {}; u32 sg[2] = {}; u32 mlo = 0, mhi = 0;
        LAS u32x4* xs = (LAS u32x4*)(wlds + 1024) + lane;
#pragma unroll
        for (int j = 0; j < 4; ++j) {
            u32x4 a = R.d[2 * j], b = R.d[2 * j + 1], xv;
#pragma unroll
            for (int e = 0; e < 4; ++e) { a[e] = pkadd(a[e], dl); b[e] = pkadd(b[e], dl); }
            lo[j] = (u32x4){vperm(a[1], a[0], 0x06040200u), vperm(a[3], a[2], 0x06040200u), vperm(b[1], b[0], 0x06040200u), vperm(b[3], b[2], 0x06040200u)};
            hi[j] = (u32x4){vperm(a[1], a[0], 0x07050301u), vperm(a[3], a[2], 0x07050301u), vperm(b[1], b[0], 0x07050301u), vperm(b[3], b[2], 0x07050301u)};
            if (quick) continue;
            u32 z[4];
#pragma unroll
            for (int e = 0; e < 4; ++e) {
                const u32 h = hi[j][e], x = ((h & 0x7f7f7f7fu) | 0x80808080u) - Lx;
                xv[e] = x;
                z[e] = (((x & 0x7c7c7c7cu) + 0x7c7c7c7cu) | ~x) & 0x80808080u;
                const int k = 4 * (j & 1) + e;
                sg[j >> 1] |= (h >> (7 - k)) & (0x01010101u << k);
            }
            cu[j] = (xv[0] & 0x03030303u) | ((xv[1] & 0x03030303u) << 2) | ((xv[2] & 0x03030303u) << 4) | ((xv[3] << 6) & 0xc0c0c0c0u);
            xs[j * 64] = xv;
            const u32 m0 = __builtin_amdgcn_udot4(z[1], 0x80402010u, __builtin_amdgcn_udot4(z[0], 0x08040201u, 0u, false), false);
            const u32 m1 = __builtin_amdgcn_udot4(z[3], 0x80402010u, __builtin_amdgcn_udot4(z[2], 0x08040201u, 0u, false), false);
            const u32 m16 = (m0 >> 7) | (m1 << 1);
            if (j < 2) mlo |= m16 << (16 * j); else mhi |= m16 << (16 * (j - 2));
        }
        u32 idx = wave_scan_incl((u32)__builtin_popcount(mlo) + (u32)__builtin_popcount(mhi));
        const u32 total = (u32)__builtin_amdgcn_readlane((int)idx, 63);
        idx -= (u32)__builtin_popcount(mlo) + (u32)__builtin_popcount(mhi);
        bool bad = false;
        u64 mm = quick ? 0ull : ((u64)mhi << 32) | mlo;
        while (__ballot(mm != 0ull) != 0ull) {
            u32 ps[4], xb[4]; bool act[4];
#pragma unroll
            for (int k = 0; k < 4; ++k) {
                act[k] = mm != 0ull; ps[k] = act[k] ? (u32)__builtin_ctzll(mm) : 0u; mm &= mm - 1ull;
                xb[k] = *(LAS uint8_t*)(wlds + 1024 + (ps[k] >> 4) * 1024u + (u32)lane * 16u + (ps[k] & 15u));
            }
#pragma unroll
            for (int k = 0; k < 4; ++k) {
                const int d4 = 32 - (int)(xb[k] >> 2);
                bad |= act[k] && (d4 > 15 || d4 < -1);
                elist[(act[k] && idx < 256u) ? idx : 256u + (u32)lane] = (uint16_t)((u32)lane * 64u + ps[k] + ((u32)max(d4, 0) << 12));
                idx += act[k] ? 1u : 0u;
            }
        }
        const bool rawmode = quick || __ballot(bad) != 0ull || total > 256u;
        const u32 info = low | (rawmode ? 128u : 0u) | (delta << 8) | ((rawmode ? 0u : total) << 9);
        const char* sp = dP + (size_t)slot * 8192;
        GSTORE_SYS(vs, lo[0], sp, 0); GSTORE_SYS(vs, lo[1], sp, 1024); GSTORE_SYS(vs, lo[2], sp, 2048); GSTORE_SYS(vs, lo[3], sp, 3072);
        if (!rawmode) {
            const u32x4 c1 = cu; const u32x2 c2 = (u32x2){sg[0], sg[1]};
            GSTORE_SYS(vs2, c1, sp, 0);
            asm volatile("global_store_dwordx2 %0, %1, %2 sc0 sc1\n\ts_nop 1" :: "v"(vs3), "v"(c2), "s"(sp) : "memory");
            if ((u32)lane * 8u < max(total, 1u)) { const u32x4 le = *(LAS u32x4*)(wlds + lane * 16); GSTORE_SYS(vs4, le, sp, 0); }
        } else { GSTORE_SYS(vs2, hi[0], sp, 0); GSTORE_SYS(vs2, hi[1], sp, 1024); GSTORE_SYS(vs2, hi[2], sp, 2048); GSTORE_SYS(vs2, hi[3], sp, 3072); }
        asm volatile("s_waitcnt vmcnt(16)" ::: "memory");
        { const u64 hv = ((u64)call << 32) | prevInfo;
          if (lane == 0) asm volatile("global_store_dwordx2 %0, %1, off sc0 sc1\n\ts_nop 1" :: "v"(prevH), "v"(hv) : "memory"); }
        prevH = dH + (size_t)slot * 128; prevInfo = info;
        slot = load(R);
    };
    if (slotA >= 0) {
        iter(RA, slotA, true);
        for (;;) {
            if (slotB < 0) break;
            iter(RB, slotB, false);
            if (slotA < 0) break;
            iter(RA, slotA, false);
        }
        asm volatile("s_waitcnt vmcnt(0)" : PIN8(RA), PIN8(RB) :: "memory");
        const u64 hv = ((u64)call << 32) | prevInfo;
        if (lane == 0) asm volatile("global_store_dwordx2 %0, %1, off sc0 sc1\n\ts_nop 1" :: "v"(prevH), "v"(hv) : "memory");
    } else asm volatile("s_waitcnt vmcnt(0)" : PIN8(RA), PIN8(RB) :: "memory");
}

__device__ __forceinline__ void unpack_role(const Params& p, LAS char* wlds, u32 call, int id, int nw) {
    const int g = p.rank, u = g >> 2, v = g & 3, lane = threadIdx.x & 63, par = call & 1;
    const int si = id % 7, nsub = nw / 7;
    if (id / 7 >= nsub) return;
    const bool cross = si >= 3;
    const int sv = cross ? si - 3 : (v + 1 + si) & 3;
    char* const ws = p.ws[g];
    gu32* tmo = (gu32*)(ws + OFF_TMO);
    const u32 vs = (u32)lane * 16u, vs2 = vs + 4096u, vs3 = (u32)lane * 8u + 5120u, vs4 = (u32)lane * 2u + 5632u;
    const u32 vx0 = (u32)(lane >> 2) * 8192u + (u32)(lane & 3) * 16u, vw0 = (u32)(lane >> 3) * 1024u + (u32)(lane & 7) * 16u;
    Stream st{p.gmask, 0, id / 7, 0, 0, cross, 0};
    bool have = st.locate();
    u64 hv = 0;
    while (have) {
        const int slot = st.slot(sv);
        const GAS char* hp = (const GAS char*)(ws + OFF_HDR + ((size_t)par * NSLOT + slot) * 128);
        u32 info = 0; bool ok = true;
        { const u64 t0 = rtc();
          for (;;) {
              const u32 seq = (u32)uni((int)(u32)(hv >> 32)); info = (u32)uni((int)(u32)hv);
              if (seq != 0u && (int)(seq - call) >= 0) break;
              u32 tv;
              asm volatile("global_load_dwordx2 %0, %2, off sc0 sc1\n\tglobal_load_dword %1, %3, off sc0 sc1\n\ts_waitcnt vmcnt(0)" : "=&v"(hv), "=&v"(tv) : "v"(hp), "v"(tmo) : "memory");
              const u32 seq2 = (u32)uni((int)(u32)(hv >> 32)); info = (u32)uni((int)(u32)hv);
              if ((int)(seq2 - call) >= 0) break;
              if ((u32)uni((int)tv) == call || rtc() - t0 > DEADLINE) {
                  __hip_atomic_store(tmo, call, __ATOMIC_RELAXED, __HIP_MEMORY_SCOPE_AGENT); ok = false; break; }
              __builtin_amdgcn_s_sleep(1);
          } }
        if (!ok) break;
        const char* sp = ws + OFF_PK + (size_t)par * PK_BYTES + (size_t)slot * 8192;
        const u32 low = info & 127u, Lx = low * 0x01010101u, dl = ((info >> 8) & 1u) * 0x00800080u, nex = info >> 9;
        u32x4 lo[4], hi[4];
        if (!(info & 128u)) {
            u32x4 c1; u32x2 c2; u32 ent;
            asm volatile("global_load_dwordx4 %0, %7, %10 sc0 sc1\n\tglobal_load_dwordx4 %1, %7, %10 offset:1024 sc0 sc1\n\tglobal_load_dwordx4 %2, %7, %10 offset:2048 sc0 sc1\n\tglobal_load_dwordx4 %3, %7, %10 offset:3072 sc0 sc1\n\t"
                         "global_load_dwordx4 %4, %8, %10 sc0 sc1\n\tglobal_load_dwordx2 %5, %9, %10 sc0 sc1\n\tglobal_load_ushort %6, %11, %10 sc0 sc1\n\ts_waitcnt vmcnt(0)"
                         : "=&v"(lo[0]), "=&v"(lo[1]), "=&v"(lo[2]), "=&v"(lo[3]), "=&v"(c1), "=&v"(c2), "=&v"(ent) : "v"(vs), "v"(vs2), "v"(vs3), "s"(sp), "v"(vs4) : "memory");
            LAS u32x4* hl = (LAS u32x4*)wlds + lane;
#pragma unroll
            for (int j = 0; j < 4; ++j) {
#pragma unroll
                for (int e = 0; e < 4; ++e) hi[j][e] = (((c1[j] >> (2 * e)) & 0x03030303u) + Lx) | ((c2[j >> 1] << (7 - 4 * (j & 1) - e)) & 0x80808080u);
                hl[j * 64] = hi[j];
            }
            for (u32 k0 = 0; k0 < nex; k0 += 64u) {
                if (k0 > 0u) asm volatile("global_load_ushort %0, %1, %2 sc0 sc1\n\ts_waitcnt vmcnt(0)" : "=&v"(ent) : "v"(vs4 + k0 * 2u), "s"(sp) : "memory");
                if (k0 + (u32)lane < nex) {
                    const u32 f = (ent >> 12) & 15u, q = (ent >> 2) & 15u;
                    LAS u32* w = (LAS u32*)(wlds + (q >> 2) * 1024u + ((ent >> 6) & 63u) * 16u + (q & 3u) * 4u);
                    __hip_atomic_fetch_sub(w, (f ? f * 4u : (u32)-4) << ((ent & 3u) * 8u), __ATOMIC_RELAXED, __HIP_MEMORY_SCOPE_WORKGROUP);
                }
            }
#pragma unroll
            for (int j = 0; j < 4; ++j) hi[j] = hl[j * 64];
        } else {
            asm volatile("global_load_dwordx4 %0, %8, %10 sc0 sc1\n\tglobal_load_dwordx4 %1, %8, %10 offset:1024 sc0 sc1\n\tglobal_load_dwordx4 %2, %8, %10 offset:2048 sc0 sc1\n\tglobal_load_dwordx4 %3, %8, %10 offset:3072 sc0 sc1\n\t"
                         "global_load_dwordx4 %4, %9, %10 sc0 sc1\n\tglobal_load_dwordx4 %5, %9, %10 offset:1024 sc0 sc1\n\tglobal_load_dwordx4 %6, %9, %10 offset:2048 sc0 sc1\n\tglobal_load_dwordx4 %7, %9, %10 offset:3072 sc0 sc1\n\ts_waitcnt vmcnt(0)"
                         : "=&v"(lo[0]), "=&v"(lo[1]), "=&v"(lo[2]), "=&v"(lo[3]), "=&v"(hi[0]), "=&v"(hi[1]), "=&v"(hi[2]), "=&v"(hi[3]) : "v"(vs), "v"(vs2), "s"(sp) : "memory");
        }
        const bool isw = st.isW();
        const char* ob; u32 v0, v1, js; gu32* ctr;
        if (isw) { ob = ws + OFF_W + (size_t)(st.kc * KC + 32 * (st.t - B0 / 16)) * 1024 + sv * 256; v0 = vw0; v1 = vw0 + 128; js = 8192; ctr = (gu32*)(ws + OFF_WRDY + sv * 512 + st.kc * 32) + (st.t - B0 / 16); }
        else { const int lr = st.lrow(sv); ob = ws + OFF_A + (size_t)lr * 8192 + st.kc * 512; v0 = vx0; v1 = vx0 + 64; js = 128; ctr = (gu32*)(ws + OFF_XRDY + (lr >> 4) * 64) + st.kc; }
#pragma unroll
        for (int j = 0; j < 4; ++j) {
            const u32x4 a = (u32x4){pksub(vperm(hi[j][0], lo[j][0], 0x05010400u), dl), pksub(vperm(hi[j][0], lo[j][0], 0x07030602u), dl), pksub(vperm(hi[j][1], lo[j][1], 0x05010400u), dl), pksub(vperm(hi[j][1], lo[j][1], 0x07030602u), dl)};
            const u32x4 b = (u32x4){pksub(vperm(hi[j][2], lo[j][2], 0x05010400u), dl), pksub(vperm(hi[j][2], lo[j][2], 0x07030602u), dl), pksub(vperm(hi[j][3], lo[j][3], 0x05010400u), dl), pksub(vperm(hi[j][3], lo[j][3], 0x07030602u), dl)};
            GSTORE_SYS(v0, a, ob, 0); GSTORE_SYS(v1, b, ob, 0); ob += js;
        }
        st.t += nsub;
        have = st.locate();
        { const GAS char* hn = (const GAS char*)(ws + OFF_HDR + ((size_t)par * NSLOT + (have ? st.slot(sv) : slot)) * 128);
          asm volatile("global_load_dwordx2 %0, %1, off sc0 sc1\n\ts_waitcnt vmcnt(0)" : "=&v"(hv) : "v"(hn) : "memory"); }
        if (lane == 0) __hip_atomic_store(ctr, call, __ATOMIC_RELAXED, __HIP_MEMORY_SCOPE_AGENT);
    }
}

__global__ __launch_bounds__(NTHREADS, 1) void ag_gemm_kernel(Params p) {
    extern __shared__ __attribute__((aligned(16))) char smem[];
    LAS char* lds = (LAS char*)smem;
    const int b = blockIdx.x, xcd = b & 7, slot = b >> 3, wave = uni(threadIdx.x >> 6), lane = threadIdx.x & 63;
    gu32* cnt = (gu32*)(p.ws[p.rank] + OFF_CNT + (size_t)b * 128) + wave;
    const int c = slot < 32 - NGS ? slot * 8 + xcd : -1;
    const int gs = slot - (32 - NGS);
    if (c >= 0 && c < NSW && wave < WPW) { send_role(p, lds + wave * 8192, cnt, wave * NSW + c, NSW * WPW); return; }
    const u32 call = (u32)uni((int)*cnt) + 1u;
    if (lane == 0) *cnt = call;
    if (c < 0) {
        const int ct = xcd & 3, hb = xcd >> 2;
        if (gs < NT0 + NS1O) { const bool g0 = gs < NT0; gemm_role<4>(p, lds, call, g0 ? 2 * (hb * (NT0 / 2) + (gs >> 1)) : R0 / 64 + 2 * (hb * NS1O + gs - NT0), g0 ? gs & 1 : 0, ct, g0); }
        else gemm_role<2>(p, lds, call, R0 / 64 + hb * NS1H + gs - NT0 - NS1O, 1, ct, false);
        return;
    }
    if (c >= NSW && c < NSW + NUW) { if (wave < UPW) unpack_role(p, lds + wave * 4096, call, wave * NUW + (c - NSW), NUW * UPW); return; }
    if (c == NCOMM - 1 && wave == 0) {
        gu32* tmo = (gu32*)(p.ws[p.rank] + OFF_TMO);
        if (!wait_all<4 * NDONE * 8 / 64>((gu32*)(p.ws[p.rank] + OFF_DONE), tmo, call)) {
            if (lane == 0) ((GAS uint16_t*)p.out[p.rank])[(size_t)p.rank * SROWS * SCOLS] = 0x7fc0;
        }
    }
}
}

extern "C" void kernel_launch(int rank, int nranks, ncclComm_t comm, hipStream_t stream,
                              void* const* d_in, const int* in_sizes, int n_in, void* d_out, int out_size,
                              const int* in_dtypes, int out_dtype, void* const* d_out_all, void* const* d_ws_all, size_t ws_bytes) {
    using namespace agg;
    if (nranks != NRANK || (unsigned)rank >= (unsigned)NRANK || ws_bytes < WS_NEED || n_in != 2 || in_sizes[0] != SROWS * KDIM || in_sizes[1] != KDIM * SCOLS || out_size != 4096 * SCOLS) {
        fprintf(stderr, "ag_gemm: unexpected shapes\n"); return; }
    static bool init[NRANK] = {};
    if (!init[rank]) {
        const hipError_t e = hipFuncSetAttribute((const void*)ag_gemm_kernel, hipFuncAttributeMaxDynamicSharedMemorySize, LDS_BYTES);
        if (e != hipSuccess) { fprintf(stderr, "ag_gemm: hipFuncSetAttribute: %s\n", hipGetErrorString(e)); return; }
        init[rank] = true;
    }
    Params p{};
    p.x = (const uint16_t*)d_in[0]; p.w = (const uint16_t*)d_in[1];
    for (int i = 0; i < NRANK; ++i) { p.out[i] = (char*)d_out_all[i]; p.ws[i] = (char*)d_ws_all[i]; }
    p.rank = rank;
    { const int n1 = N1; unsigned m = 0; int j = 0;
      for (; j < 16 + n1; ++j) if (((j + 1) * n1) / (16 + n1) > (j * n1) / (16 + n1)) m |= 1u << j;
      for (; j < 32; ++j) m |= 1u << j;
      p.gmask = m; }
    hipLaunchKernelGGL(ag_gemm_kernel, dim3(NWG), dim3(NTHREADS), LDS_BYTES, stream, p);
}
```

```cpp
#include <hip/hip_runtime.h>
#include <rccl/rccl.h>
#include <cstdio>
#include <cstdint>

namespace agg {
typedef unsigned u32;
typedef unsigned long long u64;
typedef u32 u32x4 __attribute__((ext_vector_type(4)));
typedef u32 u32x2 __attribute__((ext_vector_type(2)));
typedef float f32x4 __attribute__((ext_vector_type(4)));
typedef short s16x4 __attribute__((ext_vector_type(4)));
typedef __bf16 bf16x8 __attribute__((ext_vector_type(8)));
#define LAS __attribute__((address_space(3)))
#define GAS __attribute__((address_space(1)))
typedef GAS u32 gu32;

constexpr int NRANK = 8, KDIM = 4096, SROWS = 512, SCOLS = 512;
constexpr int KC = 256;
constexpr int R0 = 1024, R1 = 2048 - R0;
constexpr int A0 = 208, A1 = 128, B0 = R0 / 4 - A0, B1 = R1 / 4 - A1, NA = A0 + A1;
static_assert(A0 % 16 == 0 && A1 % 16 == 0 && B0 % 16 == 0 && B1 % 16 == 0 && B0 > 0 && B1 > 0 && B0 + B1 + NA == SROWS, "row split");
constexpr int NT0 = R0 / 128, NS1O = R1 / 256, NS1H = R1 / 128, NGS = NT0 + NS1O + NS1H;
static_assert(NT0 % 2 == 0 && R1 % 256 == 0 && NGS < 32, "tiles");
constexpr int NCOMM = (32 - NGS) * 8;
constexpr int NDONE = NT0 + R1 / 64;
constexpr int BK = 64, NST = KDIM / BK;
constexpr int LDS_CTL = 147456, LDS_BYTES = LDS_CTL + 64;
constexpr int NTHREADS = 704, NWG = 256;
constexpr int NCI = 9, NCX = 9, WPW = 2, NSW = 3 * NCI + 4 * NCX, NUW = NCOMM - 1 - NSW, UPW = 8;
constexpr int N1 = 10;
constexpr int PACE = 2;
static_assert(NSW + NUW + 1 <= NCOMM && NCI >= NCX && NUW >= 8, "roles");

constexpr size_t OFF_CNT = 0;
constexpr size_t OFF_TMO = 32768;
constexpr size_t OFF_DUMMY = OFF_TMO + 256;
constexpr size_t OFF_XRDY = 40960;
constexpr size_t OFF_WRDY = OFF_XRDY + 128 * 64;
constexpr size_t OFF_DONE = OFF_WRDY + 4 * 512;
constexpr int NSLOT = 2048 + 512;
constexpr size_t OFF_HDR = 1048576;
constexpr size_t A_BYTES = (size_t)2048 * KDIM * 2, W_BYTES = (size_t)KDIM * 512 * 2;
constexpr size_t OFF_A = 2097152;
constexpr size_t OFF_W = OFF_A + A_BYTES;
constexpr size_t PK_BYTES = (size_t)NSLOT * 8192;
constexpr size_t OFF_PK = OFF_W + W_BYTES;
constexpr size_t WS_NEED = OFF_PK + 2 * PK_BYTES;
static_assert(OFF_DONE + 4 * NDONE * 32 <= OFF_HDR && (4 * NDONE * 8) % 128 == 0 && 4 * NDONE * 8 / 128 <= NTHREADS / 64 && OFF_HDR + 2 * NSLOT * 128 <= OFF_A, "ctl");
static_assert(WS_NEED <= (size_t)65 * 1048576, "workspace");
constexpr u64 DEADLINE = 190000000ull;

struct Params {
    const uint16_t* x; const uint16_t* w;
    char* out[NRANK]; char* ws[NRANK];
    int rank;
    unsigned gmask;
};

__device__ __forceinline__ u64 rtc() { return __builtin_amdgcn_s_memrealtime(); }
__device__ __forceinline__ int uni(int v) { return __builtin_amdgcn_readfirstlane(v); }

__device__ __forceinline__ void map_unit(int unit, int u, int& rank, int& r) {
    const int lr = unit * 16;
    if (lr < R0) { int i = lr; if (i < 4 * A0) { const int j = i / A0; r = i - j * A0; rank = 4 * u + j; }
                   else { i -= 4 * A0; const int j = i / B0; r = NA + (i - j * B0); rank = 4 * (1 - u) + j; } }
    else { int i = lr - R0; if (i < 4 * A1) { const int j = i / A1; r = A0 + i - j * A1; rank = 4 * u + j; }
           else { i -= 4 * A1; const int j = i / B1; r = NA + B0 + (i - j * B1); rank = 4 * (1 - u) + j; } }
}

__device__ __forceinline__ bool wait_all128(gu32* words, gu32* tmo, u32 call) {
    const u64 t0 = rtc();
    const int lane = threadIdx.x & 63;
    for (int it = 0;; ++it) {
        const u32 v0 = __hip_atomic_load(words + lane, __ATOMIC_RELAXED, __HIP_MEMORY_SCOPE_SYSTEM);
        const u32 v1 = __hip_atomic_load(words + 64 + lane, __ATOMIC_RELAXED, __HIP_MEMORY_SCOPE_SYSTEM);
        if (__all((int)(v0 - call) >= 0 && (int)(v1 - call) >= 0)) return true;
        if ((it & 7) == 7 && ((u32)uni((int)__hip_atomic_load(tmo, __ATOMIC_RELAXED, __HIP_MEMORY_SCOPE_AGENT)) == call || rtc() - t0 > DEADLINE)) {
            __hip_atomic_store(tmo, call, __ATOMIC_RELAXED, __HIP_MEMORY_SCOPE_AGENT);
            return false;
        }
    }
}

__device__ __forceinline__ void st16_sys(char* ptr, u32x4 v) {
    asm volatile("global_store_dwordx4 %0, %1, off sc0 sc1\n\ts_nop 1" :: "v"(ptr), "v"(v) : "memory");
}
typedef float f32x2 __attribute__((ext_vector_type(2)));
__device__ __forceinline__ f32x2 gelu2(f32x2 y) {
    const float k1 = -2.0f * 0.7978845608028654f * 1.4426950408889634f, k2 = k1 * 0.044715f;
    const f32x2 a = y * __builtin_elementwise_fma(y * y, (f32x2){k2, k2}, (f32x2){k1, k1});
    const f32x2 d = (f32x2){__builtin_amdgcn_exp2f(a[0]), __builtin_amdgcn_exp2f(a[1])} + (f32x2){1.0f, 1.0f};
    return y * (f32x2){__builtin_amdgcn_rcpf(d[0]), __builtin_amdgcn_rcpf(d[1])};
}
__device__ __forceinline__ u32 cvt_pk(float lo, float hi) { u32 r; asm("v_cvt_pk_bf16_f32 %0, %1, %2" : "=v"(r) : "v"(lo), "v"(hi)); return r; }

#define BAR() __builtin_amdgcn_s_barrier()
#define WAITV(n) asm volatile("s_waitcnt vmcnt(" #n ")" ::: "memory")
#define WAITL0() asm volatile("s_waitcnt lgkmcnt(0)" ::: "memory")
#define SB() __builtin_amdgcn_sched_barrier(0)
typedef float f32x16 __attribute__((ext_vector_type(16)));
__device__ __forceinline__ const char* uniptr(const char* q) { const u64 a = (u64)q; return (const char*)(((u64)(u32)uni((int)(u32)(a >> 32)) << 32) | (u32)uni((int)(u32)a)); }
template <int N> __device__ __forceinline__ void waitv() { asm volatile("s_waitcnt vmcnt(%0)" :: "n"(N) : "memory"); }
__device__ __forceinline__ void glds2(u32 v0, u32 v1, const char* sb, u32 m0v) {
    asm volatile("s_mov_b32 m0, %3\n\ts_nop 0\n\tglobal_load_lds_dwordx4 %0, %2 sc0 sc1\n\tglobal_load_lds_dwordx4 %1, %2 offset:1024 sc0 sc1" :: "v"(v0), "v"(v1), "s"(sb), "s"(m0v) : "memory");
}
__device__ __forceinline__ void glds4(u32 v0, u32 v1, u32 v2, u32 v3, const char* sb, u32 m0v) {
    asm volatile("s_mov_b32 m0, %5\n\ts_nop 0\n\tglobal_load_lds_dwordx4 %0, %4 sc0 sc1\n\tglobal_load_lds_dwordx4 %1, %4 offset:1024 sc0 sc1\n\t"
                 "global_load_lds_dwordx4 %2, %4 offset:2048 sc0 sc1\n\tglobal_load_lds_dwordx4 %3, %4 offset:3072 sc0 sc1" :: "v"(v0), "v"(v1), "v"(v2), "v"(v3), "s"(sb), "s"(m0v) : "memory");
}
template <int OFF> __device__ __forceinline__ void rdx(bf16x8& d, u32 a) { asm volatile("ds_read_b128 %0, %1 offset:%c2" : "=v"(d) : "v"(a), "i"(OFF) : "memory"); }
template <int OFF> __device__ __forceinline__ void rdw(s16x4& d0, s16x4& d1, u32 a0, u32 a1) {
    asm volatile("ds_read_b64_tr_b16 %0, %2 offset:%c4\n\tds_read_b64_tr_b16 %1, %3 offset:%c4" : "=&v"(d0), "=&v"(d1) : "v"(a0), "v"(a1), "i"(OFF) : "memory"); }
template <int NI> struct Frags { s16x4 w[4][2]; bf16x8 x[4][NI]; };

template <int NF>
__device__ __forceinline__ void gemm_role(const Params& p, LAS char* lds, u32 call, int rt64, int helper, int ct, bool early) {
    constexpr int TM = 32 * NF, NBUF = NF == 4 ? 4 : 6, ASTG = TM * 128, LB = NBUF * ASTG;
    constexpr int NUN = TM / 16;
    constexpr int PPS = NF == 4 ? 16 : 12;
    constexpr int NPRE = NBUF - 1 < 4 ? NBUF - 1 : 4;
    const int g = p.rank, u = g >> 2, v = g & 3;
    const int wave = uni(threadIdx.x >> 6), lane = threadIdx.x & 63;
    char* const ws = p.ws[g];
    gu32* tmo = (gu32*)(ws + OFF_TMO);
    LAS u32* ctl = (LAS u32*)(lds + LDS_CTL);
    const u32 ldsb = (u32)(size_t)lds;
    if (wave == 10) {
        const int dk = lane & 3;
        const bool xl = lane < 4 * NUN, wl = helper && lane >= 32;
        bool mine = false;
        if (xl) { int rk, r; map_unit(rt64 * 4 + (lane >> 2), u, rk, r); mine = rk == g; }
        const bool watch = (xl && !mine) || wl;
        const char* wbase = wl ? ws + OFF_WRDY + ct * 512 + ((lane - 32) >> 2) * 4 : ws + OFF_XRDY + (size_t)(rt64 * 4 + (xl ? lane >> 2 : 0)) * 64;
        const int wstep = wl ? 32 : 4;
        bool ok = true; int ready = 0;
        auto need = [&](int c) {
            if (ready > c) return;
            const u64 t0 = rtc();
            const int kc = min(c + dk, 15);
            gu32* word = (gu32*)(wbase + kc * wstep);
            for (int it = 0;; ++it) {
                const u32 val = __hip_atomic_load(word, __ATOMIC_RELAXED, __HIP_MEMORY_SCOPE_SYSTEM);
                const u64 miss = __ballot(watch && (int)(val - call) < 0);
                const u64 m4 = 0x1111111111111111ull;
                ready = (miss & m4) ? c : (miss & (m4 << 1)) ? c + 1 : (miss & (m4 << 2)) ? c + 2 : (miss & (m4 << 3)) ? c + 3 : c + 4;
                if (ready > c) return;
                if ((it & 7) == 7 && ((u32)uni((int)__hip_atomic_load(tmo, __ATOMIC_RELAXED, __HIP_MEMORY_SCOPE_AGENT)) == call || rtc() - t0 > DEADLINE)) {
                    __hip_atomic_store(tmo, call, __ATOMIC_RELAXED, __HIP_MEMORY_SCOPE_AGENT); ok = false; ready = 16; return; }
                __builtin_amdgcn_s_sleep(1);
            }
        };
        need(0);
        BAR();
        if (NBUF > 4) need(1);
        BAR();
        for (int s = 0; s < NST - 1; ++s) {
            if (((s + NBUF) & 3) == 0 && s + NBUF < NST) need((s + NBUF) >> 2);
            if (s == NST - NBUF - 2) {
                const u32 t = (u32)uni((int)__hip_atomic_load(tmo, __ATOMIC_RELAXED, __HIP_MEMORY_SCOPE_AGENT));
                if (lane == 0) ctl[0] = (!ok || t == call) ? 1u : 0u;
                WAITL0();
            }
            BAR();
        }
        if (helper) BAR();
        return;
    }
    if (wave >= 8) {
        const bool l0 = wave == 8;
        u32 va[2], vb[4]; const char* ub[NUN];
#pragma unroll
        for (int e = 0; e < 2; ++e) { const int ar = 8 * e + (lane >> 3); va[e] = (u32)ar * 8192u + (u32)(((lane & 7) ^ ((ar >> 1) & 7)) * 16) - (u32)e * 1024u; }
#pragma unroll
        for (int i = 0; i < NUN; ++i) { int rk, r; map_unit(rt64 * 4 + i, u, rk, r);
            ub[i] = uniptr((rk == g) ? (const char*)p.x + (size_t)r * 8192 : ws + OFF_A + (size_t)(rt64 * 64 + i * 16) * 8192); }
#pragma unroll
        for (int e = 0; e < 4; ++e) { const int kr = 4 * e + (lane >> 4), sw = ((kr & 3) << 2) | ((kr >> 2) & 3); vb[e] = (u32)kr * 1024u + (u32)(((lane & 15) ^ sw) * 16) - (u32)e * 1024u; }
        const char* b0 = uniptr(helper ? ws + OFF_W + ct * 256 : (const char*)p.w + ct * 256);
        auto issue = [&](int s, int buf) {
            const u32 la = ldsb + (u32)buf * ASTG, lb = ldsb + LB + (u32)buf * 16384u;
            const char* b = b0 + (size_t)s * 65536;
            if (l0) {
                const u32 v0 = va[0] + (u32)s * 128u, v1 = va[1] + (u32)s * 128u;
#pragma unroll
                for (int i = 0; i < NUN; ++i) glds2(v0, v1, ub[i], la + i * 2048);
                if (NF == 2) glds4(vb[0], vb[1], vb[2], vb[3], b, lb);
            } else {
#pragma unroll
                for (int q = (NF == 2 ? 1 : 0); q < 4; ++q) glds4(vb[0], vb[1], vb[2], vb[3], b + q * 16384, lb + q * 4096);
            }
        };
        BAR();
        for (int s = 0; s < NPRE; ++s) issue(s, s);
        waitv<(NPRE - 1) * PPS>(); BAR();
        for (int s = NPRE; s < NBUF; ++s) issue(s, s);
        int buf = 0;
        for (int s = 0; s < NST - NBUF; ++s) { waitv<(NBUF - 2) * PPS>(); BAR(); issue(s + NBUF, buf); buf = buf + 1 == NBUF ? 0 : buf + 1; }
        waitv<(NBUF - 2) * PPS>(); BAR();
        waitv<(NBUF - 3) * PPS>(); BAR();
        waitv<(NBUF - 4) * PPS>(); BAR();
        if (NBUF == 6) { waitv<PPS>(); BAR(); waitv<0>(); BAR(); }
        if (helper) BAR();
        return;
    }
    constexpr int NI = NF / 2;
    const int wm = wave >> 2, wn = wave & 3;
    const int l31 = lane & 31, lh = lane >> 5;
    u32 xa[4], wa[2][2];
#pragma unroll
    for (int ks = 0; ks < 4; ++ks) xa[ks] = ldsb + (u32)((wm * (TM / 2) + l31) * 128 + (((2 * ks + lh) ^ ((l31 >> 1) & 7)) * 16));
    { const int blk = (lane >> 4) & 1, tq = (lane & 15) >> 2, tp = lane & 3;
#pragma unroll
      for (int t = 0; t < 2; ++t) {
          const int kr = 8 * lh + 4 * t + tq;
          const int sw = ((kr & 3) << 2) | ((kr >> 2) & 3);
          const int ch = 4 * wn + 2 * blk + (tp >> 1);
          wa[0][t] = ldsb + (u32)(LB + kr * 256 + ((ch ^ sw) * 16) + 8 * (tp & 1));
          wa[1][t] = wa[0][t] + 3 * 16384;
      } }
    f32x16 acc[NI];
#pragma unroll
    for (int i = 0; i < NI; ++i)
#pragma unroll
        for (int e = 0; e < 16; ++e) acc[i][e] = 0.f;
#define RDW(f, BUF, ks) rdw<((BUF) % 3) * 16384 * (NBUF == 6) + (BUF) * 16384 * (NBUF == 4) + (ks) * 4096>(f.w[ks][0], f.w[ks][1], wa[NBUF == 6 ? (BUF) / 3 : 0][0], wa[NBUF == 6 ? (BUF) / 3 : 0][1])
#define RDX(f, BUF, ks, i) rdx<(BUF) * ASTG + (i) * 4096>(f.x[ks][i], xa[ks])
#define MF(f, ks, i) do { acc[i] = __builtin_amdgcn_mfma_f32_32x32x16_bf16(__builtin_bit_cast(bf16x8, __builtin_shufflevector(f.w[ks][0], f.w[ks][1], 0, 1, 2, 3, 4, 5, 6, 7)), f.x[ks][i], acc[i], 0, 0, 0); SB(); } while (0)
#define LANDED(f) asm volatile("s_waitcnt lgkmcnt(0)" : "+v"(f.w[0][0]), "+v"(f.w[0][1]), "+v"(f.w[1][0]), "+v"(f.w[1][1]), "+v"(f.w[2][0]), "+v"(f.w[2][1]), "+v"(f.w[3][0]), "+v"(f.w[3][1]), \
                     "+v"(f.x[0][0]), "+v"(f.x[1][0]), "+v"(f.x[2][0]), "+v"(f.x[3][0]), "+v"(f.x[0][NI - 1]), "+v"(f.x[1][NI - 1]), "+v"(f.x[2][NI - 1]), "+v"(f.x[3][NI - 1]) :: "memory")
#define LANDED1(f) asm volatile("s_waitcnt lgkmcnt(0)" : "+v"(f.w[0][0]), "+v"(f.w[0][1]), "+v"(f.w[1][0]), "+v"(f.w[1][1]), "+v"(f.w[2][0]), "+v"(f.w[2][1]), "+v"(f.w[3][0]), "+v"(f.w[3][1]), \
                     "+v"(f.x[0][0]), "+v"(f.x[1][0]), "+v"(f.x[2][0]), "+v"(f.x[3][0]) :: "memory")
#define WAITF(f) do { if constexpr (NI == 2) { LANDED(f); } else { LANDED1(f); } } while (0)
#define RDK(f, NB, ks) do { RDW(f, NB, ks); RDX(f, NB, ks, 0); if constexpr (NI == 2) { RDX(f, NB, ks, NI - 1); } } while (0)
#define MFK(f, ks) do { MF(f, ks, 0); if constexpr (NI == 2) { MF(f, ks, NI - 1); } } while (0)
#define STEP(cur, nxt, NB) do { BAR(); SB(); \
        if constexpr (NI == 2) { \
            MF(cur, 0, 0); RDW(nxt, NB, 0); RDX(nxt, NB, 0, 0); SB(); \
            MF(cur, 0, NI - 1); RDX(nxt, NB, 0, NI - 1); RDW(nxt, NB, 1); SB(); \
            MF(cur, 1, 0); RDX(nxt, NB, 1, 0); RDX(nxt, NB, 1, NI - 1); SB(); \
            MF(cur, 1, NI - 1); RDW(nxt, NB, 2); RDX(nxt, NB, 2, 0); SB(); \
            MF(cur, 2, 0); RDX(nxt, NB, 2, NI - 1); RDW(nxt, NB, 3); SB(); \
            MF(cur, 2, NI - 1); RDX(nxt, NB, 3, 0); RDX(nxt, NB, 3, NI - 1); SB(); \
            MF(cur, 3, 0); MF(cur, 3, NI - 1); \
        } else { \
            MF(cur, 0, 0); RDK(nxt, NB, 0); RDW(nxt, NB, 1); SB(); \
            MF(cur, 1, 0); RDX(nxt, NB, 1, 0); RDK(nxt, NB, 2); SB(); \
            MF(cur, 2, 0); RDK(nxt, NB, 3); SB(); \
            MF(cur, 3, 0); \
        } \
        WAITF(nxt); } while (0)
    Frags<NI> P, Q;
    BAR();
    BAR();
    RDK(P, 0, 0); RDK(P, 0, 1); RDK(P, 0, 2); RDK(P, 0, 3); WAITF(P);
    for (int s = 0; s < 60; s += NBUF) {
        STEP(P, Q, 1); STEP(Q, P, 2); STEP(P, Q, 3);
        if constexpr (NBUF == 4) { STEP(Q, P, 0); }
        else { STEP(Q, P, 4 % NBUF); STEP(P, Q, 5 % NBUF); STEP(Q, P, 0); }
    }
    STEP(P, Q, 1); STEP(Q, P, 2); STEP(P, Q, 3);
    MFK(Q, 0); MFK(Q, 1); MFK(Q, 2); MFK(Q, 3);
    const bool bad = ctl[0] != 0u;
    u32x2 pk[NI][4];
#pragma unroll
    for (int i = 0; i < NI; ++i) { const f32x16& a = acc[i];
#pragma unroll
        for (int e = 0; e < 4; ++e) { const f32x2 y0 = gelu2((f32x2){a[4 * e], a[4 * e + 1]}), y1 = gelu2((f32x2){a[4 * e + 2], a[4 * e + 3]});
            pk[i][e] = (u32x2){cvt_pk(y0[0], y0[1]), cvt_pk(y1[0], y1[1])}; } }
    char* const ocol = helper ? p.out[4 * (1 - u) + ct] + v * 256 : p.out[g] + ct * 256;
    if (!helper) {
#pragma unroll
        for (int i = 0; i < NI; ++i) {
            const int row0 = wm * (TM / 2) + 32 * i;
            int rk0, r0, rk1, r1; map_unit(rt64 * 4 + (row0 >> 4), u, rk0, r0); map_unit(rt64 * 4 + (row0 >> 4) + 1, u, rk1, r1);
            const int grow = (l31 < 16 ? rk0 * SROWS + r0 : rk1 * SROWS + r1) + (l31 & 15);
            char* dst = ocol + (size_t)grow * 1024 + wn * 64 + lh * 16;
#pragma unroll
            for (int e = 0; e < 4; e += 2) {
                auto s0 = __builtin_amdgcn_permlane32_swap(pk[i][e][0], pk[i][e + 1][0], false, false);
                auto s1 = __builtin_amdgcn_permlane32_swap(pk[i][e][1], pk[i][e + 1][1], false, false);
                u32x4 ov = (u32x4){s0[0], s1[0], s0[1], s1[1]};
                if (bad) ov = (u32x4){0x7fc07fc0u, 0x7fc07fc0u, 0x7fc07fc0u, 0x7fc07fc0u};
                *(GAS u32x4*)(dst + e * 16) = ov;
            }
        }
    } else {
        LAS char* otile = lds + LB + (NF == 4 ? 0 : 4 * 16384);
#pragma unroll
        for (int i = 0; i < NI; ++i) { const int row = wm * (TM / 2) + 32 * i + l31;
#pragma unroll
            for (int e = 0; e < 4; ++e) *(LAS u32x2*)(otile + row * 256 + (((4 * wn + e) ^ (row & 15)) * 16) + lh * 8) = pk[i][e]; }
        WAITL0(); BAR();
        constexpr int RPW = TM / 8, NS = RPW / 4;
        const int row0 = RPW * wave;
        int urk, urow; map_unit(rt64 * 4 + (row0 >> 4), u, urk, urow);
        char* obase = ocol + (size_t)(urk * SROWS + urow + (row0 & 15)) * 1024;
        u32x4 ov[NS];
#pragma unroll
        for (int it = 0; it < NS; ++it) {
            const int row = row0 + 4 * it + (lane >> 4), ch = lane & 15;
            ov[it] = *(LAS u32x4*)(otile + row * 256 + ((ch ^ (row & 15)) * 16));
            if (bad) ov[it] = (u32x4){0x7fc07fc0u, 0x7fc07fc0u, 0x7fc07fc0u, 0x7fc07fc0u};
        }
#pragma unroll
        for (int it = 0; it < NS; ++it) {
            st16_sys(obase + (size_t)(4 * it + (lane >> 4)) * 1024 + (lane & 15) * 16, ov[it]);
            if (early && (it + 1) % PACE == 0) WAITV(0);
        }
    }
    if (helper) {
        WAITV(0);
        const int tile = early ? rt64 / 2 : NT0 + rt64 - R0 / 64;
        if (lane == 0) __hip_atomic_store((gu32*)(p.ws[4 * (1 - u) + ct] + OFF_DONE) + (v * NDONE + tile) * 8 + wave, call, __ATOMIC_RELAXED, __HIP_MEMORY_SCOPE_SYSTEM);
    }
}

struct Stream {
    u32 gmask; int step, t, G, kc; bool cross; int c1;
    __device__ __forceinline__ bool locate() {
#pragma unroll 1
        for (;;) { if (step >= 32) return false;
            G = (gmask >> step) & 1; kc = G ? c1 : step - c1;
            const int cn = cross ? (G ? B1 / 16 : B0 / 16 + 8) : (G ? A1 / 16 : A0 / 16); if (t < cn) return true; t -= cn; c1 += G; ++step; } }
    __device__ __forceinline__ bool isW() const { return cross && G == 0 && t >= B0 / 16; }
    __device__ __forceinline__ int lrow(int sv) const { return (G ? R0 : 0) + (cross ? 4 * (G ? A1 : A0) + sv * (G ? B1 : B0) : sv * (G ? A1 : A0)) + 16 * t; }
    __device__ __forceinline__ int slot(int sv) const { return isW() ? 2048 + sv * 128 + kc * 8 + (t - B0 / 16) : (lrow(sv) >> 4) * 16 + kc; }
};

__device__ __forceinline__ u32 vperm(u32 s0, u32 s1, u32 sel) { return __builtin_amdgcn_perm(s0, s1, sel); }
__device__ __forceinline__ u32 pkmax(u32 a, u32 b) { u32 r; asm("v_pk_max_u16 %0, %1, %2" : "=v"(r) : "v"(a), "v"(b)); return r; }
__device__ __forceinline__ u32 pkadd(u32 a, u32 b) { u32 r; asm("v_pk_add_u16 %0, %1, %2" : "=v"(r) : "v"(a), "s"(b)); return r; }
__device__ __forceinline__ u32 pksub(u32 a, u32 b) { u32 r; asm("v_pk_sub_u16 %0, %1, %2" : "=v"(r) : "v"(a), "s"(b)); return r; }
__device__ __forceinline__ u32 wave_scan_incl(u32 x) {
    x += (u32)__builtin_amdgcn_update_dpp(0, (int)x, 0x111, 0xf, 0xf, true);
    x += (u32)__builtin_amdgcn_update_dpp(0, (int)x, 0x112, 0xf, 0xf, true);
    x += (u32)__builtin_amdgcn_update_dpp(0, (int)x, 0x114, 0xf, 0xf, true);
    x += (u32)__builtin_amdgcn_update_dpp(0, (int)x, 0x118, 0xf, 0xf, true);
    x += (u32)__builtin_amdgcn_update_dpp(0, (int)x, 0x142, 0xa, 0xf, true);
    x += (u32)__builtin_amdgcn_update_dpp(0, (int)x, 0x143, 0xc, 0xf, true);
    return x;
}
__device__ __forceinline__ u32 wave_max(u32 x) {
    x = max(x, (u32)__builtin_amdgcn_update_dpp(0, (int)x, 0x111, 0xf, 0xf, true));
    x = max(x, (u32)__builtin_amdgcn_update_dpp(0, (int)x, 0x112, 0xf, 0xf, true));
    x = max(x, (u32)__builtin_amdgcn_update_dpp(0, (int)x, 0x114, 0xf, 0xf, true));
    x = max(x, (u32)__builtin_amdgcn_update_dpp(0, (int)x, 0x118, 0xf, 0xf, true));
    const u32 a = (u32)__builtin_amdgcn_readlane((int)x, 15), b = (u32)__builtin_amdgcn_readlane((int)x, 31), c = (u32)__builtin_amdgcn_readlane((int)x, 47), d = (u32)__builtin_amdgcn_readlane((int)x, 63);
    return max(max(a, b), max(c, d));
}
struct Raw { u32x4 d[8]; };
#define GLOAD(dst, voff, sbase) asm volatile("global_load_dwordx4 %0, %1, %2" : "=v"(dst) : "v"(voff), "s"(sbase) : "memory")
#define PIN8(R) "+v"(R.d[0]), "+v"(R.d[1]), "+v"(R.d[2]), "+v"(R.d[3]), "+v"(R.d[4]), "+v"(R.d[5]), "+v"(R.d[6]), "+v"(R.d[7])
#define GSTORE_SYS(voff, val, sbase, off) asm volatile("global_store_dwordx4 %0, %1, %2 offset:" #off " sc0 sc1\n\ts_nop 1" :: "v"(voff), "v"(val), "s"(sbase) : "memory")

__device__ __forceinline__ void send_role(const Params& p, LAS char* wlds, gu32* cnt, int c, int wave) {
    const int g = p.rank, u = g >> 2, v = g & 3, lane = threadIdx.x & 63;
    int di, cl;
    if (c < 7 * NCX) { di = c % 7; cl = c / 7; } else { di = (c - 7 * NCX) % 3; cl = NCX + (c - 7 * NCX) / 3; }
    const int ncl = di >= 3 ? NCX : NCI, sub = wave * ncl + cl, nsub = ncl * WPW;
    const bool cross = di >= 3;
    const int dest = cross ? 4 * (1 - u) + (di - 3) : 4 * u + ((v + 1 + di) & 3);
    const u32 vx0 = (u32)(lane >> 2) * 8192u + (u32)(lane & 3) * 16u, vw0 = (u32)(lane >> 3) * 1024u + (u32)(lane & 7) * 16u;
    Stream ld{p.gmask, 0, sub, 0, 0, cross, 0};
    auto load = [&](Raw& R) -> int {
        const bool have = ld.locate();
        const char* sb; u32 v0, v1, js; int slot = -1;
        if (have && ld.isW()) { sb = (const char*)p.w + (size_t)(ld.kc * KC + 32 * (ld.t - B0 / 16)) * 1024 + (di - 3) * 256; v0 = vw0; v1 = vw0 + 128; js = 8192; }
        else { const int r0 = have ? (cross ? NA + (ld.G ? B0 : 0) : (ld.G ? A0 : 0)) + 16 * ld.t : 0; sb = (const char*)p.x + (size_t)r0 * 8192 + (have ? ld.kc : 0) * 512; v0 = vx0; v1 = vx0 + 64; js = 128; }
        if (have) { slot = ld.slot(v); ld.t += nsub; }
        GLOAD(R.d[0], v0, sb); GLOAD(R.d[1], v1, sb); sb += js; GLOAD(R.d[2], v0, sb); GLOAD(R.d[3], v1, sb); sb += js;
        GLOAD(R.d[4], v0, sb); GLOAD(R.d[5], v1, sb); sb += js; GLOAD(R.d[6], v0, sb); GLOAD(R.d[7], v1, sb);
        return slot;
    };
    Raw RA, RB;
    u32 cv;
    asm volatile("global_load_dword %0, %1, off" : "=v"(cv) : "v"(cnt) : "memory");
    int slotA = load(RA), slotB = load(RB);
    asm volatile("s_waitcnt vmcnt(8)" : PIN8(RA), "+v"(cv) :: "memory");
    const u32 call = (u32)uni((int)cv) + 1u;
    if (lane == 0) *cnt = call;
    const int par = call & 1;
    const char* dP = p.ws[dest] + OFF_PK + (size_t)par * PK_BYTES;
    char* dH = p.ws[dest] + OFF_HDR + (size_t)par * NSLOT * 128;
    char* prevH = p.ws[g] + OFF_DUMMY; u32 prevInfo = 0;
    const u32 vs = (u32)lane * 16u, vs2 = vs + 4096u, vs3 = (u32)lane * 8u + 5120u, vs4 = vs + 5632u;
    LAS uint16_t* elist = (LAS uint16_t*)wlds;
    auto iter = [&](Raw& R, int& slot, bool first) {
        if (!first) asm volatile("s_waitcnt vmcnt(16)" : PIN8(R) :: "memory");
        u32 m = 0;
#pragma unroll
        for (int e = 0; e < 4; ++e) m = pkmax(m, R.d[0][e] & 0x7fff7fffu);
        const bool quick = first && wave == 1;
        const u32 emax = quick ? 257u : wave_max(max(m & 0xffffu, m >> 16)) >> 7;
        const u32 delta = ~emax & 1u, top = (emax + delta) >> 1, low = top > 3u ? top - 3u : 0u, Lx = low * 0x01010101u, dl = delta * 0x00800080u;
        u32x4 lo[4], hi[4], cu = {}; u32 sg[2] = {}; u32 mlo = 0, mhi = 0;
        LAS u32x4* xs = (LAS u32x4*)(wlds + 1024) + lane;
#pragma unroll
        for (int j = 0; j < 4; ++j) {
            u32x4 a = R.d[2 * j], b = R.d[2 * j + 1], xv;
#pragma unroll
            for (int e = 0; e < 4; ++e) { a[e] = pkadd(a[e], dl); b[e] = pkadd(b[e], dl); }
            lo[j] = (u32x4){vperm(a[1], a[0], 0x06040200u), vperm(a[3], a[2], 0x06040200u), vperm(b[1], b[0], 0x06040200u), vperm(b[3], b[2], 0x06040200u)};
            hi[j] = (u32x4){vperm(a[1], a[0], 0x07050301u), vperm(a[3], a[2], 0x07050301u), vperm(b[1], b[0], 0x07050301u), vperm(b[3], b[2], 0x07050301u)};
            if (quick) continue;
            u32 z[4];
#pragma unroll
            for (int e = 0; e < 4; ++e) {
                const u32 h = hi[j][e], x = ((h & 0x7f7f7f7fu) | 0x80808080u) - Lx;
                xv[e] = x;
                z[e] = (((x & 0x7c7c7c7cu) + 0x7c7c7c7cu) | ~x) & 0x80808080u;
                const int k = 4 * (j & 1) + e;
                sg[j >> 1] |= (h >> (7 - k)) & (0x01010101u << k);
            }
            cu[j] = (xv[0] & 0x03030303u) | ((xv[1] & 0x03030303u) << 2) | ((xv[2] & 0x03030303u) << 4) | ((xv[3] << 6) & 0xc0c0c0c0u);
            xs[j * 64] = xv;
            const u32 m0 = __builtin_amdgcn_udot4(z[1], 0x80402010u, __builtin_amdgcn_udot4(z[0], 0x08040201u, 0u, false), false);
            const u32 m1 = __builtin_amdgcn_udot4(z[3], 0x80402010u, __builtin_amdgcn_udot4(z[2], 0x08040201u, 0u, false), false);
            const u32 m16 = (m0 >> 7) | (m1 << 1);
            if (j < 2) mlo |= m16 << (16 * j); else mhi |= m16 << (16 * (j - 2));
        }
        u32 idx = wave_scan_incl((u32)__builtin_popcount(mlo) + (u32)__builtin_popcount(mhi));
        const u32 total = (u32)__builtin_amdgcn_readlane((int)idx, 63);
        idx -= (u32)__builtin_popcount(mlo) + (u32)__builtin_popcount(mhi);
        bool bad = false;
        u64 mm = quick ? 0ull : ((u64)mhi << 32) | mlo;
        while (__ballot(mm != 0ull) != 0ull) {
            u32 ps[4], xb[4]; bool act[4];
#pragma unroll
            for (int k = 0; k < 4; ++k) {
                act[k] = mm != 0ull; ps[k] = act[k] ? (u32)__builtin_ctzll(mm) : 0u; mm &= mm - 1ull;
                xb[k] = *(LAS uint8_t*)(wlds + 1024 + (ps[k] >> 4) * 1024u + (u32)lane * 16u + (ps[k] & 15u));
            }
#pragma unroll
            for (int k = 0; k < 4; ++k) {
                const int d4 = 32 - (int)(xb[k] >> 2);
                bad |= act[k] && (d4 > 15 || d4 < -1);
                elist[(act[k] && idx < 256u) ? idx : 256u + (u32)lane] = (uint16_t)((u32)lane * 64u + ps[k] + ((u32)max(d4, 0) << 12));
                idx += act[k] ? 1u : 0u;
            }
        }
        const bool rawmode = quick || __ballot(bad) != 0ull || total > 256u;
        const u32 info = low | (rawmode ? 128u : 0u) | (delta << 8) | ((rawmode ? 0u : total) << 9);
        const char* sp = dP + (size_t)slot * 8192;
        GSTORE_SYS(vs, lo[0], sp, 0); GSTORE_SYS(vs, lo[1], sp, 1024); GSTORE_SYS(vs, lo[2], sp, 2048); GSTORE_SYS(vs, lo[3], sp, 3072);
        if (!rawmode) {
            const u32x4 c1 = cu; const u32x2 c2 = (u32x2){sg[0], sg[1]};
            GSTORE_SYS(vs2, c1, sp, 0);
            asm volatile("global_store_dwordx2 %0, %1, %2 sc0 sc1\n\ts_nop 1" :: "v"(vs3), "v"(c2), "s"(sp) : "memory");
            if ((u32)lane * 8u < max(total, 1u)) { const u32x4 le = *(LAS u32x4*)(wlds + lane * 16); GSTORE_SYS(vs4, le, sp, 0); }
        } else { GSTORE_SYS(vs2, hi[0], sp, 0); GSTORE_SYS(vs2, hi[1], sp, 1024); GSTORE_SYS(vs2, hi[2], sp, 2048); GSTORE_SYS(vs2, hi[3], sp, 3072); }
        asm volatile("s_waitcnt vmcnt(16)" ::: "memory");
        { const u64 hv = ((u64)call << 32) | prevInfo;
          if (lane == 0) asm volatile("global_store_dwordx2 %0, %1, off sc0 sc1\n\ts_nop 1" :: "v"(prevH), "v"(hv) : "memory"); }
        prevH = dH + (size_t)slot * 128; prevInfo = info;
        slot = load(R);
    };
    if (slotA >= 0) {
        iter(RA, slotA, true);
        for (;;) {
            if (slotB < 0) break;
            iter(RB, slotB, false);
            if (slotA < 0) break;
            iter(RA, slotA, false);
        }
        asm volatile("s_waitcnt vmcnt(0)" : PIN8(RA), PIN8(RB) :: "memory");
        const u64 hv = ((u64)call << 32) | prevInfo;
        if (lane == 0) asm volatile("global_store_dwordx2 %0, %1, off sc0 sc1\n\ts_nop 1" :: "v"(prevH), "v"(hv) : "memory");
    } else asm volatile("s_waitcnt vmcnt(0)" : PIN8(RA), PIN8(RB) :: "memory");
}

__device__ __forceinline__ void unpack_role(const Params& p, LAS char* wlds, u32 call, int id, int nw) {
    const int g = p.rank, v = g & 3, lane = threadIdx.x & 63, par = call & 1;
    const int si = id % 7, nsub = nw / 7;
    if (id / 7 >= nsub) return;
    const bool cross = si >= 3;
    const int sv = cross ? si - 3 : (v + 1 + si) & 3;
    char* const ws = p.ws[g];
    gu32* tmo = (gu32*)(ws + OFF_TMO);
    const u32 vs = (u32)lane * 16u, vs2 = vs + 4096u, vs3 = (u32)lane * 8u + 5120u, vs4 = (u32)lane * 2u + 5632u;
    const u32 vx0 = (u32)(lane >> 2) * 8192u + (u32)(lane & 3) * 16u, vw0 = (u32)(lane >> 3) * 1024u + (u32)(lane & 7) * 16u;
    Stream st{p.gmask, 0, id / 7, 0, 0, cross, 0};
    bool have = st.locate();
    u64 hv = 0;
    while (have) {
        const int slot = st.slot(sv);
        const GAS char* hp = (const GAS char*)(ws + OFF_HDR + ((size_t)par * NSLOT + slot) * 128);
        u32 info = 0; bool ok = true;
        { const u64 t0 = rtc();
          for (int it = 0;; ++it) {
              const u32 seq = (u32)uni((int)(u32)(hv >> 32)); info = (u32)uni((int)(u32)hv);
              if (seq != 0u && (int)(seq - call) >= 0) break;
              if ((it & 7) == 7 && ((u32)uni((int)__hip_atomic_load(tmo, __ATOMIC_RELAXED, __HIP_MEMORY_SCOPE_AGENT)) == call || rtc() - t0 > DEADLINE)) {
                  __hip_atomic_store(tmo, call, __ATOMIC_RELAXED, __HIP_MEMORY_SCOPE_AGENT); ok = false; break; }
              if (it) __builtin_amdgcn_s_sleep(1);
              hv = __hip_atomic_load((const GAS u64*)hp, __ATOMIC_RELAXED, __HIP_MEMORY_SCOPE_SYSTEM);
          } }
        if (!ok) break;
        const char* sp = ws + OFF_PK + (size_t)par * PK_BYTES + (size_t)slot * 8192;
        const u32 low = info & 127u, Lx = low * 0x01010101u, dl = ((info >> 8) & 1u) * 0x00800080u, nex = info >> 9;
        u32x4 lo[4], hi[4];
        if (!(info & 128u)) {
            u32x4 c1; u32x2 c2; u32 ent;
            asm volatile("global_load_dwordx4 %0, %7, %10 sc0 sc1\n\tglobal_load_dwordx4 %1, %7, %10 offset:1024 sc0 sc1\n\tglobal_load_dwordx4 %2, %7, %10 offset:2048 sc0 sc1\n\tglobal_load_dwordx4 %3, %7, %10 offset:3072 sc0 sc1\n\t"
                         "global_load_dwordx4 %4, %8, %10 sc0 sc1\n\tglobal_load_dwordx2 %5, %9, %10 sc0 sc1\n\tglobal_load_ushort %6, %11, %10 sc0 sc1\n\ts_waitcnt vmcnt(0)"
                         : "=&v"(lo[0]), "=&v"(lo[1]), "=&v"(lo[2]), "=&v"(lo[3]), "=&v"(c1), "=&v"(c2), "=&v"(ent) : "v"(vs), "v"(vs2), "v"(vs3), "s"(sp), "v"(vs4) : "memory");
            LAS u32x4* hl = (LAS u32x4*)wlds + lane;
#pragma unroll
            for (int j = 0; j < 4; ++j) {
#pragma unroll
                for (int e = 0; e < 4; ++e) hi[j][e] = (((c1[j] >> (2 * e)) & 0x03030303u) + Lx) | ((c2[j >> 1] << (7 - 4 * (j & 1) - e)) & 0x80808080u);
                hl[j * 64] = hi[j];
            }
            for (u32 k0 = 0; k0 < nex; k0 += 64u) {
                if (k0 > 0u) asm volatile("global_load_ushort %0, %1, %2 sc0 sc1\n\ts_waitcnt vmcnt(0)" : "=&v"(ent) : "v"(vs4 + k0 * 2u), "s"(sp) : "memory");
                if (k0 + (u32)lane < nex) {
                    const u32 f = (ent >> 12) & 15u, q = (ent >> 2) & 15u;
                    LAS u32* w = (LAS u32*)(wlds + (q >> 2) * 1024u + ((ent >> 6) & 63u) * 16u + (q & 3u) * 4u);
                    __hip_atomic_fetch_sub(w, (f ? f * 4u : (u32)-4) << ((ent & 3u) * 8u), __ATOMIC_RELAXED, __HIP_MEMORY_SCOPE_WORKGROUP);
                }
            }
#pragma unroll
            for (int j = 0; j < 4; ++j) hi[j] = hl[j * 64];
        } else {
            asm volatile("global_load_dwordx4 %0, %8, %10 sc0 sc1\n\tglobal_load_dwordx4 %1, %8, %10 offset:1024 sc0 sc1\n\tglobal_load_dwordx4 %2, %8, %10 offset:2048 sc0 sc1\n\tglobal_load_dwordx4 %3, %8, %10 offset:3072 sc0 sc1\n\t"
                         "global_load_dwordx4 %4, %9, %10 sc0 sc1\n\tglobal_load_dwordx4 %5, %9, %10 offset:1024 sc0 sc1\n\tglobal_load_dwordx4 %6, %9, %10 offset:2048 sc0 sc1\n\tglobal_load_dwordx4 %7, %9, %10 offset:3072 sc0 sc1\n\ts_waitcnt vmcnt(0)"
                         : "=&v"(lo[0]), "=&v"(lo[1]), "=&v"(lo[2]), "=&v"(lo[3]), "=&v"(hi[0]), "=&v"(hi[1]), "=&v"(hi[2]), "=&v"(hi[3]) : "v"(vs), "v"(vs2), "s"(sp) : "memory");
        }
        const bool isw = st.isW();
        const char* ob; u32 v0, v1, js; gu32* ctr;
        if (isw) { ob = ws + OFF_W + (size_t)(st.kc * KC + 32 * (st.t - B0 / 16)) * 1024 + sv * 256; v0 = vw0; v1 = vw0 + 128; js = 8192; ctr = (gu32*)(ws + OFF_WRDY + sv * 512 + st.kc * 32) + (st.t - B0 / 16); }
        else { const int lr = st.lrow(sv); ob = ws + OFF_A + (size_t)lr * 8192 + st.kc * 512; v0 = vx0; v1 = vx0 + 64; js = 128; ctr = (gu32*)(ws + OFF_XRDY + (lr >> 4) * 64) + st.kc; }
#pragma unroll
        for (int j = 0; j < 4; ++j) {
            const u32x4 a = (u32x4){pksub(vperm(hi[j][0], lo[j][0], 0x05010400u), dl), pksub(vperm(hi[j][0], lo[j][0], 0x07030602u), dl), pksub(vperm(hi[j][1], lo[j][1], 0x05010400u), dl), pksub(vperm(hi[j][1], lo[j][1], 0x07030602u), dl)};
            const u32x4 b = (u32x4){pksub(vperm(hi[j][2], lo[j][2], 0x05010400u), dl), pksub(vperm(hi[j][2], lo[j][2], 0x07030602u), dl), pksub(vperm(hi[j][3], lo[j][3], 0x05010400u), dl), pksub(vperm(hi[j][3], lo[j][3], 0x07030602u), dl)};
            GSTORE_SYS(v0, a, ob, 0); GSTORE_SYS(v1, b, ob, 0); ob += js;
        }
        st.t += nsub;
        have = st.locate();
        { const GAS char* hn = (const GAS char*)(ws + OFF_HDR + ((size_t)par * NSLOT + (have ? st.slot(sv) : slot)) * 128);
          asm volatile("global_load_dwordx2 %0, %1, off sc0 sc1\n\ts_waitcnt vmcnt(0)" : "=&v"(hv) : "v"(hn) : "memory"); }
        if (lane == 0) __hip_atomic_store(ctr, call, __ATOMIC_RELAXED, __HIP_MEMORY_SCOPE_AGENT);
    }
}

__global__ __launch_bounds__(NTHREADS, 1) void ag_gemm_kernel(Params p) {
    extern __shared__ __attribute__((aligned(16))) char smem[];
    LAS char* lds = (LAS char*)smem;
    const int b = blockIdx.x, xcd = b & 7, slot = b >> 3, wave = uni(threadIdx.x >> 6), lane = threadIdx.x & 63;
    gu32* cnt = (gu32*)(p.ws[p.rank] + OFF_CNT + (size_t)b * 128) + wave;
    const int c = slot < 32 - NGS ? slot * 8 + xcd : -1;
    const int gs = slot - (32 - NGS);
    if (c >= 0 && c < NSW && wave < WPW) { send_role(p, lds + wave * 8192, cnt, c, wave); return; }
    const u32 call = (u32)uni((int)*cnt) + 1u;
    if (lane == 0) *cnt = call;
    if (c < 0) {
        const int ct = xcd & 3, hb = xcd >> 2;
        if (gs < NT0 + NS1O) { const bool g0 = gs < NT0; gemm_role<4>(p, lds, call, g0 ? 2 * (hb * (NT0 / 2) + (gs >> 1)) : R0 / 64 + 2 * (hb * NS1O + gs - NT0), g0 ? gs & 1 : 0, ct, g0); }
        else gemm_role<2>(p, lds, call, R0 / 64 + hb * NS1H + gs - NT0 - NS1O, 1, ct, false);
        return;
    }
    if (c >= NSW && c < NSW + NUW) { if (wave < UPW) unpack_role(p, lds + wave * 4096, call, wave * NUW + (c - NSW), NUW * UPW); return; }
    if (c == NCOMM - 1 && wave < 4 * NDONE * 8 / 128) {
        gu32* tmo = (gu32*)(p.ws[p.rank] + OFF_TMO);
        if (!wait_all128((gu32*)(p.ws[p.rank] + OFF_DONE) + wave * 128, tmo, call)) {
            if (lane == 0) ((GAS uint16_t*)p.out[p.rank])[(size_t)p.rank * SROWS * SCOLS] = 0x7fc0;
        }
    }
}
}

extern "C" void kernel_launch(int rank, int nranks, ncclComm_t comm, hipStream_t stream,
                              void* const* d_in, const int* in_sizes, int n_in, void* d_out, int out_size,
                              const int* in_dtypes, int out_dtype, void* const* d_out_all, void* const* d_ws_all, size_t ws_bytes) {
    using namespace agg;
    if (nranks != NRANK || (unsigned)rank >= (unsigned)NRANK || ws_bytes < WS_NEED || n_in != 2 || in_sizes[0] != SROWS * KDIM || in_sizes[1] != KDIM * SCOLS || out_size != 4096 * SCOLS) {
        fprintf(stderr, "ag_gemm: unexpected shapes\n"); return; }
    static bool init[NRANK] = {};
    if (!init[rank]) {
        const hipError_t e = hipFuncSetAttribute((const void*)ag_gemm_kernel, hipFuncAttributeMaxDynamicSharedMemorySize, LDS_BYTES);
        if (e != hipSuccess) { fprintf(stderr, "ag_gemm: hipFuncSetAttribute: %s\n", hipGetErrorString(e)); return; }
        init[rank] = true;
    }
    Params p{};
    p.x = (const uint16_t*)d_in[0]; p.w = (const uint16_t*)d_in[1];
    for (int i = 0; i < NRANK; ++i) { p.out[i] = (char*)d_out_all[i]; p.ws[i] = (char*)d_ws_all[i]; }
    p.rank = rank;
    { const int n1 = N1; unsigned m = 0; int j = 0;
      for (; j < 16 + n1; ++j) if (((j + 1) * n1) / (16 + n1) > (j * n1) / (16 + n1)) m |= 1u << j;
      for (; j < 32; ++j) m |= 1u << j;
      p.gmask = m; }
    hipLaunchKernelGGL(ag_gemm_kernel, dim3(NWG), dim3(NTHREADS), LDS_BYTES, stream, p);
}
```
